```python
import numpy as np

kernel_name = "gpu_a2a_gemm_m4096_k4096_n4096_bf16_reluqe4m3"


ABSMAX_RTOL = 0.09

COMPUTE_DTYPE = "bfloat16"
EPILOGUE = "relu_quant_e4m3"
IN_QUANT = None
OUT_QUANT = "e4m3"
IN_DTYPES = {"x": "bfloat16", "w_mat": "bfloat16"}
WORLD_SIZE = 8

def _to_bf16(x):
    u = np.ascontiguousarray(x, dtype=np.float32).view(np.uint32)
    return ((u + 0x7FFF + ((u >> 16) & 1)) & 0xFFFF0000).view(np.float32)

def setup_inputs():

    x = _to_bf16(np.random.randn(4096, 4096).astype(np.float32))
    w_mat = _to_bf16((np.random.randn(4096, 4096) / np.sqrt(4096)).astype(np.float32))
    return {"x": x, "w_mat": w_mat}

def _snap_fp8(x, mant_bits, max_abs, min_normal):
    a = np.abs(x.astype(np.float64))
    _m, _e = np.frexp(a)
    q = np.where(a >= min_normal, np.ldexp(1.0, _e - 1 - mant_bits),
                 min_normal * 2.0 ** -mant_bits)
    snapped = np.minimum(np.round(a / q) * q, max_abs)
    return (np.sign(x) * snapped).astype(np.float32)
def _quant_dequant(y, amax):
    scale = amax / 448.0
    q = _snap_fp8(y / scale, 3, 448.0, 2.0 ** -6)
    return (q * scale).astype(np.float32)
def _epilogue(y):
    y = np.maximum(y, 0.0)
    return _quant_dequant(y, np.abs(y).max())

def reference(x, w_mat):
    y = (x @ w_mat).astype(np.float32)
    return _epilogue(y)
```

```cpp
#include <hip/hip_runtime.h>
#include <rccl/rccl.h>
#include <cstdio>
#include <cstdint>

#define HIP_CHECK(expr) do { hipError_t _e = (expr); if (_e != hipSuccess) { \
    fprintf(stderr, "HIP error %s at %s:%d\n", hipGetErrorString(_e), __FILE__, __LINE__); \
    return; } } while (0)

#define PUSH_WIN 2
#define DBG_TS 0

namespace {

typedef unsigned u32x4 __attribute__((ext_vector_type(4)));
typedef unsigned u32x2 __attribute__((ext_vector_type(2)));
typedef short s16x4 __attribute__((ext_vector_type(4)));
typedef short s16x8 __attribute__((ext_vector_type(8)));
typedef __bf16 bf16x8 __attribute__((ext_vector_type(8)));
typedef float f32x16 __attribute__((ext_vector_type(16)));
#define AS3 __attribute__((address_space(3)))

constexpr int NR = 8;
constexpr int NWG = 256;
constexpr int NQ = 16;
constexpr unsigned W_BYTES = 4096u * 4096u * 2u;

constexpr size_t WS_STG = 0;
constexpr size_t STG_BYTES = 4u << 20;
constexpr size_t WS_FLG = STG_BYTES;
constexpr size_t WS_AMX = WS_FLG + 4096;
constexpr size_t WS_CNT = WS_AMX + 16384;
constexpr size_t WS_TMO = WS_CNT + 1024;
constexpr size_t WS_DBG = WS_TMO + 1024;
constexpr size_t WS_END = WS_DBG + 256 * 32 * 8;

constexpr unsigned L_SYNC = 148 * 1024;
constexpr unsigned L_LEVEL = L_SYNC + 0, L_FULL = L_SYNC + 4  , L_TMO = L_SYNC + 12;
constexpr unsigned L_FREE = L_SYNC + 16;
constexpr unsigned L_MAX1 = L_SYNC + 32, L_CNT1 = L_SYNC + 36, L_MAX2 = L_SYNC + 40, L_CNT2 = L_SYNC + 44;
constexpr unsigned LDS_BYTES = L_SYNC + 64;

constexpr unsigned long long DEADLINE = 50000000ull;

struct Params {
    const char* x;
    const char* w;
    char* out;
    char* ws[NR];
    int rank;
    int pad;
};

template <int N> __device__ __forceinline__ void wait_vm() { asm volatile("s_waitcnt vmcnt(%0)" ::"i"(N) : "memory"); }
__device__ __forceinline__ unsigned rfl(unsigned v) { return __builtin_amdgcn_readfirstlane(v); }
__device__ __forceinline__ unsigned slot_base(unsigned c) {
    return c < 2 ? c * 32768u : 65536u + (c - 2) * 28672u;
}
__device__ __forceinline__ u32x4 lds_ld4(unsigned addr) {
    u32x4 v;
    asm volatile("ds_read_b128 %0, %1\n\ts_waitcnt lgkmcnt(0)" : "=&v"(v) : "v"(addr) : "memory");
    return v;
}
__device__ __forceinline__ u32x2 lds_ld2(unsigned addr) {
    u32x2 v;
    asm volatile("ds_read_b64 %0, %1\n\ts_waitcnt lgkmcnt(0)" : "=&v"(v) : "v"(addr) : "memory");
    return v;
}
__device__ __forceinline__ void lds_st(unsigned addr, unsigned v) {
    asm volatile("ds_write_b32 %0, %1" ::"v"(addr), "v"(v) : "memory");
}
__device__ __forceinline__ void st_sys32(void* p, unsigned v) {
    asm volatile("global_store_dword %0, %1, off sc0 sc1" ::"v"(p), "v"(v) : "memory");
}
__device__ __forceinline__ void st_sys128(void* p, u32x4 v) {
    asm volatile("global_store_dwordx4 %0, %1, off sc0 sc1\n\ts_nop 1" ::"v"(p), "v"(v) : "memory");
}
__device__ __forceinline__ void st_sys128_w15(void* p, u32x4 v) {
    asm volatile("s_waitcnt vmcnt(15)\n\tglobal_store_dwordx4 %0, %1, off sc0 sc1\n\ts_nop 1" ::"v"(p), "v"(v) : "memory");
}
__device__ __forceinline__ unsigned wave_max_u32(unsigned v) {
#pragma unroll
    for (int o = 32; o >= 1; o >>= 1) { unsigned t = __shfl_xor(v, o); v = v > t ? v : t; }
    return v;
}
#define STAMP(i) do {} while (0)

template <int S> __device__ __forceinline__ void push_seq(char* dst, const u32x4 (&r)[NQ], unsigned* flag, unsigned base, int lane) {
    constexpr int hi = (S + PUSH_WIN - 1 < NQ - 1) ? (S + PUSH_WIN - 1) : (NQ - 1);
    constexpr int nf = (S < PUSH_WIN - 1) ? S : (PUSH_WIN - 1);
    wait_vm<(hi - S) + nf>();
    if (lane == 0) st_sys32(flag, base + S + 1);
    if constexpr (S + PUSH_WIN < NQ) st_sys128(dst + (size_t)(S + PUSH_WIN) * 32768, r[S + PUSH_WIN]);
    if constexpr (S + 1 < NQ) push_seq<S + 1>(dst, r, flag, base, lane);
}

template <int NU>
__device__ __forceinline__ void compute_batch(const u32x4 (&xb)[16], f32x16 (&acc)[2], unsigned wa0, unsigned wa1) {
#pragma unroll
    for (int u = 0; u < NU; ++u) {
#pragma unroll
        for (int b = 0; b < 2; ++b) {
            const unsigned wa = (b ? wa1 : wa0) + u * 2048;
            s16x4 lo = __builtin_amdgcn_ds_read_tr16_b64_v4i16((AS3 s16x4*)(wa));
            s16x4 hi = __builtin_amdgcn_ds_read_tr16_b64_v4i16((AS3 s16x4*)(wa + 512));
            s16x8 w = __builtin_shufflevector(lo, hi, 0, 1, 2, 3, 4, 5, 6, 7);
            acc[b] = __builtin_amdgcn_mfma_f32_32x32x16_bf16(__builtin_bit_cast(bf16x8, w), __builtin_bit_cast(bf16x8, xb[u]), acc[b], 0, 0, 0);
        }
    }
}

}

__global__ __launch_bounds__(512, 2) void a2a_gemm_kernel(Params P) {
    extern __shared__ __attribute__((aligned(16))) char shm[];
    const int lane = threadIdx.x & 63;
    const int wid = rfl(threadIdx.x >> 6);
    const int b = blockIdx.x;
    const int rank = P.rank;
    char* const my = P.ws[rank];
    const int xcd = b & 7, idx = b >> 3;
    const int nt = xcd * 8 + (idx & 7), mt = idx >> 3;
    unsigned* const cntp = (unsigned*)(my + WS_CNT) + b;
    unsigned* const tmop = (unsigned*)(my + WS_TMO);

    if (wid == 0) {
        STAMP(0);
        const unsigned p = (unsigned)(rank + (b & 7)) & 7u;
        const int g = b >> 3;
        const int pc = lane >> 4, r = lane & 15;
        const char* src = P.x + (size_t)(p * 512 + g * 16 + r) * 1024 + pc * 16;
        unsigned cv;
        u32x4 v[NQ];
        asm volatile("global_load_dword %0, %1, off sc1" : "=&v"(cv) : "v"(cntp) : "memory");
#pragma unroll
        for (int q = 0; q < NQ; ++q)
            asm volatile("global_load_dwordx4 %0, %1, off offset:%2" : "=&v"(v[q]) : "v"(src), "i"(q * 64) : "memory");
        __builtin_amdgcn_s_barrier();
        asm volatile("s_waitcnt vmcnt(16)" : "+v"(cv) :: "memory");
        char* const pw = P.ws[p < (unsigned)NR ? p : 0];
        char* dst = pw + WS_STG + (size_t)rank * (NQ * 32768) + (g >> 1) * 2048 + (pc >> 1) * 1024 + (pc & 1) * 512 + (g & 1) * 256 + r * 16;
        unsigned* flag = (unsigned*)(pw + WS_FLG + ((g >> 3) * 8 + rank) * 128 + (g & 7) * 4);
        if (p == (unsigned)rank) {
#pragma unroll
            for (int q = 0; q < NQ; ++q) st_sys128_w15(dst + (size_t)q * 32768, v[q]);
            wait_vm<0>();
            if (lane == 0) st_sys32(flag, (cv + 1) * 32 + NQ);
        } else {
#pragma unroll
            for (int q = 0; q < PUSH_WIN; ++q) st_sys128_w15(dst + (size_t)q * 32768, v[q]);
            STAMP(1);
            push_seq<0>(dst, v, flag, (cv + 1) * 32, lane);
        }
        STAMP(2);
        return;
    }
    if (wid == 1) {
        if (lane < 16) lds_st(L_SYNC + lane * 4, 0);
        const unsigned call = rfl(__hip_atomic_load(cntp, __ATOMIC_RELAXED, __HIP_MEMORY_SCOPE_AGENT)) + 1;
        asm volatile("s_waitcnt lgkmcnt(0)" ::: "memory");
        __builtin_amdgcn_s_barrier();
        const int s = lane >> 3;
        const unsigned* fp = (const unsigned*)(my + WS_FLG + (mt * 8 + s) * 128 + (lane & 7) * 4);
        const unsigned base = call * 32;
        unsigned level = 0;
        const unsigned long long t0 = __builtin_amdgcn_s_memrealtime();
        while (level <= NQ) {
            const unsigned f = __hip_atomic_load(fp, __ATOMIC_RELAXED, __HIP_MEMORY_SCOPE_SYSTEM);
            const unsigned pr = f > base ? f - base : 0u;
            const unsigned val = (s == rank) ? (pr >= NQ ? NQ + 1 : 0u) : pr + 1;
            unsigned nl = level;
            while (nl <= NQ && __all(val > nl)) ++nl;
            if (nl != level) {
                level = nl;
                if (lane == 0) lds_st(L_LEVEL, level);
            } else {
                if (__hip_atomic_load(tmop, __ATOMIC_RELAXED, __HIP_MEMORY_SCOPE_AGENT) == call ||
                    __builtin_amdgcn_s_memrealtime() - t0 > DEADLINE) {
                    if (lane == 0) {
                        __hip_atomic_store(tmop, call, __ATOMIC_RELAXED, __HIP_MEMORY_SCOPE_AGENT);
                        lds_st(L_TMO, 1);
                        lds_st(L_LEVEL, 99);
                    }
                    break;
                }
                __builtin_amdgcn_s_sleep(1);
            }
        }
        return;
    }
    if (wid < 4) {
        const int lw = wid - 2;
        const auto wr = __builtin_amdgcn_make_buffer_rsrc((void*)P.w, 0, W_BYTES, 0x00020000);
        const int row = lw * 8 + (lane >> 3);
        const unsigned voff = row * 8192 + nt * 128 + (((lane & 7) ^ (row & 2)) * 16);
        auto fill_local = [&](int c) {
            const unsigned lb = slot_base(c) + lw * 1024;
            const unsigned k0 = rank * 512 + c * 256;
#pragma unroll
            for (int u = 0; u < 16; ++u)
                __builtin_amdgcn_raw_ptr_buffer_load_lds(wr, (AS3 void*)(lb + u * 2048), 16, voff, (k0 + u * 16) * 8192, 0, 0);
        };
        auto fill_round = [&](int q, unsigned slot) {
            const unsigned lb = slot_base(slot) + lw * 1024;
#pragma unroll
            for (int j = 0; j < 7; ++j) {
                const unsigned k0 = ((rank + 1 + j) & 7) * 512 + q * 32;
                __builtin_amdgcn_raw_ptr_buffer_load_lds(wr, (AS3 void*)(lb + (2 * j) * 2048), 16, voff, k0 * 8192, 0, 0);
                __builtin_amdgcn_raw_ptr_buffer_load_lds(wr, (AS3 void*)(lb + (2 * j + 1) * 2048), 16, voff, (k0 + 16) * 8192, 0, 0);
            }
        };
        fill_local(0);
        __builtin_amdgcn_s_barrier();
        fill_local(1);
        wait_vm<16>();
        if (lane == 0) lds_st(L_FULL + lw * 4, 1);
        unsigned slot = 2;
        for (int q = 0; q < NQ; ++q) {
            const unsigned c = q + 2;
            if (c >= 5) {
                while (true) {
                    const u32x4 f = lds_ld4(L_FREE);
                    const unsigned m01 = f.x < f.y ? f.x : f.y, m23 = f.z < f.w ? f.z : f.w;
                    if (rfl(m01 < m23 ? m01 : m23) + 5 > c) break;
                    __builtin_amdgcn_s_sleep(1);
                }
            }
            fill_round(q, slot);
            slot = slot == 4 ? 0 : slot + 1;
            wait_vm<14>();
            if (lane == 0) lds_st(L_FULL + lw * 4, c);
        }
        wait_vm<0>();
        if (lane == 0) lds_st(L_FULL + lw * 4, NQ + 2);
        return;
    }

    const int gw = wid - 4;
    const unsigned call = rfl(__hip_atomic_load(cntp, __ATOMIC_RELAXED, __HIP_MEMORY_SCOPE_AGENT)) + 1;
    const auto xr = __builtin_amdgcn_make_buffer_rsrc((void*)(my + WS_STG), 0, (unsigned)STG_BYTES, 0x00020000);
    const unsigned xvoff = (mt * 4 + gw) * 2048 + lane * 16;
    unsigned wl0;
    {
        const int h = lane >> 5, g1 = (lane >> 4) & 1, q4 = (lane & 15) >> 2, p = lane & 3;
        wl0 = (8 * h + q4) * 128 + ((64 * (p & 1) + 16 * g1 + 8 * (p >> 1)) ^ (32 * (q4 >> 1)));
    }
    f32x16 acc[2] = {};
    u32x4 xa[16], xb[16];
    auto issue_local = [&](u32x4 (&d)[16], int c) {
#pragma unroll
        for (int i = 0; i < 8; ++i) {
            const unsigned so = (rank * NQ + c * 8 + i) * 32768;
            d[2 * i] = __builtin_amdgcn_raw_buffer_load_b128(xr, xvoff, so, 17);
            d[2 * i + 1] = __builtin_amdgcn_raw_buffer_load_b128(xr, xvoff + 1024, so, 17);
        }
    };
    auto issue_round = [&](u32x4 (&d)[16], int q) {
#pragma unroll
        for (int j = 0; j < 7; ++j) {
            const unsigned so = (((rank + 1 + j) & 7) * NQ + q) * 32768;
            d[2 * j] = __builtin_amdgcn_raw_buffer_load_b128(xr, xvoff, so, 17);
            d[2 * j + 1] = __builtin_amdgcn_raw_buffer_load_b128(xr, xvoff + 1024, so, 17);
        }
    };
    unsigned level = 0, full = 0;
    auto poll = [&]() {
        const u32x4 s = lds_ld4(L_SYNC);
        level = rfl(s.x);
        full = rfl(s.y < s.z ? s.y : s.z);
    };
    auto wait_level = [&](unsigned need) { while (level < need) { poll(); } };
    auto wait_full = [&](unsigned need) { while (full < need) { poll(); } };
    auto set_free = [&](unsigned n) { if (lane == 0) lds_st(L_FREE + gw * 4, n); };

    __builtin_amdgcn_s_barrier();
    wait_level(1);
    if (gw == 0) STAMP(3);
    issue_local(xa, 0);
    issue_local(xb, 1);
    wait_full(1);
    compute_batch<16>(xa, acc, wl0, wl0 ^ 32);
    set_free(1);
    poll();
#define STEP(NU, cur, nxt, c, slotv, qn)                                                     \
    do {                                                                                     \
        const unsigned wb = slot_base(slotv) + wl0, wb1 = slot_base(slotv) + (wl0 ^ 32);     \
        if (level >= (unsigned)(qn) + 2) {                                                   \
            issue_round(nxt, qn);                                                            \
            wait_full((c) + 1);                                                              \
            compute_batch<NU>(cur, acc, wb, wb1);                                            \
            set_free((c) + 1);                                                               \
            poll();                                                                          \
        } else {                                                                             \
            wait_full((c) + 1);                                                              \
            compute_batch<NU>(cur, acc, wb, wb1);                                            \
            set_free((c) + 1);                                                               \
            wait_level((qn) + 2);                                                            \
            issue_round(nxt, qn);                                                            \
        }                                                                                    \
    } while (0)
    STEP(16, xb, xa, 1, 1u, 0);
    if (gw == 0) STAMP(4);
    unsigned slot = 2;
    for (int q = 0; q < NQ - 2; q += 2) {
        STEP(14, xa, xb, q + 2, slot, q + 1);
        slot = slot == 4 ? 0 : slot + 1;
        STEP(14, xb, xa, q + 3, slot, q + 2);
        slot = slot == 4 ? 0 : slot + 1;
    }
    STEP(14, xa, xb, NQ, slot, NQ - 1);
    slot = slot == 4 ? 0 : slot + 1;
    wait_full(NQ + 2);
    compute_batch<14>(xb, acc, slot_base(slot) + wl0, slot_base(slot) + (wl0 ^ 32));
    if (gw == 0) STAMP(5);

    float mx = 0.f;
#pragma unroll
    for (int i = 0; i < 16; ++i) {
        acc[0][i] = fmaxf(acc[0][i], 0.f);
        acc[1][i] = fmaxf(acc[1][i], 0.f);
        mx = fmaxf(mx, fmaxf(acc[0][i], acc[1][i]));
    }
    unsigned mu = wave_max_u32(__builtin_bit_cast(unsigned, mx));
    {
        unsigned t = 0;
        if (lane == 0)
            asm volatile("ds_max_u32 %1, %2\n\tds_add_rtn_u32 %0, %1, %3 offset:4\n\ts_waitcnt lgkmcnt(0)"
                         : "=&v"(t) : "v"(L_MAX1), "v"(mu), "v"(1u) : "memory");
        if (rfl(t) == 3) {
            unsigned m = lds_ld2(L_MAX1).x;
            if (lds_ld4(L_SYNC).w) m = 0x7fffffffu;
            if (lane < NR) {
                char* gp = P.ws[lane] + WS_AMX + (size_t)(rank * NWG + b) * 8;
                u32x2 gv = {m, call};
                asm volatile("global_store_dwordx2 %0, %1, off sc0 sc1" ::"v"(gp), "v"(gv) : "memory");
            }
        }
    }
    {
        const char* gp = my + WS_AMX + gw * 4096 + lane * 16;
        const unsigned long long t0 = __builtin_amdgcn_s_memrealtime();
        unsigned m;
        while (true) {
            u32x4 g0, g1, g2, g3;
            unsigned tv;
            asm volatile("global_load_dwordx4 %0, %5, off sc0 sc1\n\t"
                         "global_load_dwordx4 %1, %5, off offset:1024 sc0 sc1\n\t"
                         "global_load_dwordx4 %2, %5, off offset:2048 sc0 sc1\n\t"
                         "global_load_dwordx4 %3, %5, off offset:3072 sc0 sc1\n\t"
                         "global_load_dword %4, %6, off sc1\n\t"
                         "s_waitcnt vmcnt(0)"
                         : "=&v"(g0), "=&v"(g1), "=&v"(g2), "=&v"(g3), "=&v"(tv) : "v"(gp), "v"(tmop) : "memory");
            const bool ok = g0.y >= call && g0.w >= call && g1.y >= call && g1.w >= call &&
                            g2.y >= call && g2.w >= call && g3.y >= call && g3.w >= call;
            m = max(max(max(g0.x, g0.z), max(g1.x, g1.z)), max(max(g2.x, g2.z), max(g3.x, g3.z)));
            if (rfl(tv) == call) { m = 0x7fffffffu; break; }
            if (__all(ok)) break;
            if (__builtin_amdgcn_s_memrealtime() - t0 > DEADLINE) {
                if (lane == 0) __hip_atomic_store(tmop, call, __ATOMIC_RELAXED, __HIP_MEMORY_SCOPE_AGENT);
                m = 0x7fffffffu;
                break;
            }
        }
        m = wave_max_u32(m);
        if (lane == 0)
            asm volatile("ds_max_u32 %0, %1\n\tds_add_u32 %0, %2 offset:4" ::"v"(L_MAX2), "v"(m), "v"(1u) : "memory");
        u32x2 r2;
        do { r2 = lds_ld2(L_MAX2); } while (rfl(r2.y) < 4);
        mu = r2.x;
    }
    if (gw == 0) STAMP(6);

    const float amax = __builtin_bit_cast(float, mu);
    const float scale = amax / 448.0f;
    const float inv = scale > 0.f ? 1.0f / scale : 0.f;
    u32x4 o[2];
#pragma unroll
    for (int h2 = 0; h2 < 2; ++h2)
#pragma unroll
        for (int i = 0; i < 4; ++i) {
            int w = __builtin_amdgcn_cvt_pk_fp8_f32(acc[h2][4 * i] * inv, acc[h2][4 * i + 1] * inv, 0, false);
            w = __builtin_amdgcn_cvt_pk_fp8_f32(acc[h2][4 * i + 2] * inv, acc[h2][4 * i + 3] * inv, w, true);
            o[h2][i] = mu > 0x7f800000u ? 0x7f7f7f7fu : (unsigned)w;
        }
    char* op = P.out + (size_t)(mt * 128 + gw * 32 + (lane & 31)) * 4096 + nt * 64 + (lane >> 5) * 32;
    *(u32x4*)op = o[0];
    *(u32x4*)(op + 16) = o[1];
    if (gw == 0 && lane == 0) __hip_atomic_store(cntp, call, __ATOMIC_RELAXED, __HIP_MEMORY_SCOPE_AGENT);
    if (gw == 0) STAMP(7);
}

extern "C" void kernel_launch(int rank, int nranks, ncclComm_t comm, hipStream_t stream,
                              void* const* d_in, const int* in_sizes, int n_in,
                              void* d_out, int out_size,
                              const int* in_dtypes, int out_dtype,
                              void* const* d_out_all,
                              void* const* d_ws_all, size_t ws_bytes) {
    if (nranks != NR || (unsigned)rank >= (unsigned)NR || ws_bytes < WS_END || n_in < 2) {
        fprintf(stderr, "unexpected configuration\n");
        return;
    }
    HIP_CHECK(hipFuncSetAttribute((const void*)a2a_gemm_kernel, hipFuncAttributeMaxDynamicSharedMemorySize, LDS_BYTES));
    Params P;
    P.x = (const char*)d_in[0];
    P.w = (const char*)d_in[1];
    P.out = (char*)d_out;
    for (int i = 0; i < NR; ++i) P.ws[i] = (char*)d_ws_all[i];
    P.rank = rank;
    P.pad = 0;
    a2a_gemm_kernel<<<NWG, 512, LDS_BYTES, stream>>>(P);
    HIP_CHECK(hipGetLastError());
}
```

```cpp
#include <hip/hip_runtime.h>
#include <rccl/rccl.h>
#include <cstdio>
#include <cstdint>

#define HIP_CHECK(expr) do { hipError_t _e = (expr); if (_e != hipSuccess) { \
    fprintf(stderr, "HIP error %s at %s:%d\n", hipGetErrorString(_e), __FILE__, __LINE__); \
    return; } } while (0)

#define PACE 56
#define POFF 60
#define PLEAD 150
#define NP2 4
#define FREP 8
#define WPF 6
#define MSLEEP 1
#define XMAP 1
#define DBG_TS 0

namespace {

typedef unsigned u32x4 __attribute__((ext_vector_type(4)));
typedef unsigned u32x2 __attribute__((ext_vector_type(2)));
typedef short s16x4 __attribute__((ext_vector_type(4)));
typedef short s16x8 __attribute__((ext_vector_type(8)));
typedef __bf16 bf16x8 __attribute__((ext_vector_type(8)));
typedef __bf16 bf16x2 __attribute__((ext_vector_type(2)));
typedef float f32x2 __attribute__((ext_vector_type(2)));
typedef float f32x4 __attribute__((ext_vector_type(4)));
typedef float f32x16 __attribute__((ext_vector_type(16)));
#define AS3 __attribute__((address_space(3)))

constexpr int NR = 8;
constexpr int NWG = 256;
constexpr int NQ = 16;
constexpr int Q1 = NQ - NP2;
constexpr int F0 = 2 + Q1;
constexpr int NF = F0 + 2 * NP2;
constexpr unsigned W_BYTES = 4096u * 4096u * 2u;

constexpr size_t WS_STG = 0;
constexpr size_t STG_BYTES = 4u << 20;
constexpr size_t WS_XCH = STG_BYTES;
constexpr size_t XCH_BYTES = 4u << 20;
constexpr size_t WS_FLG = WS_XCH + XCH_BYTES;
constexpr size_t WS_XFL = WS_FLG + 16384;
constexpr size_t WS_AMX = WS_XFL + 32768;
constexpr size_t WS_AM2 = WS_AMX + 4096;
constexpr size_t WS_CNT = WS_AM2 + 8192;
constexpr size_t WS_TMO = WS_CNT + 1024;
constexpr size_t WS_DBG = WS_TMO + 1024;
constexpr size_t WS_END = WS_DBG + 256 * 128 * 4;

constexpr unsigned L_SYNC = 148 * 1024;
constexpr unsigned L_LEVEL = L_SYNC + 0, L_FULL = L_SYNC + 4  , L_XRDY = L_SYNC + 12;
constexpr unsigned L_FREE = L_SYNC + 16;
constexpr unsigned L_MAX1 = L_SYNC + 32, L_MAX2 = L_SYNC + 40;
constexpr unsigned L_RED = L_SYNC + 48;
constexpr unsigned L_TMO = L_SYNC + 64;
constexpr unsigned L_DBG = L_SYNC + 128;
constexpr unsigned LDS_BYTES = L_SYNC + 128 + (DBG_TS ? 512 : 0);

constexpr unsigned long long DEADLINE = 50000000ull;

struct Params {
    const char* x;
    const char* w;
    char* out;
    char* ws[NR];
    int rank;
    int pad;
};

template <int N> __device__ __forceinline__ void wait_vm() { asm volatile("s_waitcnt vmcnt(%0)" ::"i"(N) : "memory"); }
__device__ __forceinline__ unsigned rfl(unsigned v) { return __builtin_amdgcn_readfirstlane(v); }
__device__ __forceinline__ unsigned slot_base(unsigned c) {
    return c < 2 ? c * 32768u : 8192u + c * 28672u;
}
__device__ __forceinline__ u32x4 lds_ld4(unsigned addr) {
    u32x4 v;
    asm volatile("ds_read_b128 %0, %1\n\ts_waitcnt lgkmcnt(0)" : "=&v"(v) : "v"(addr) : "memory");
    return v;
}
__device__ __forceinline__ u32x2 lds_ld2(unsigned addr) {
    u32x2 v;
    asm volatile("ds_read_b64 %0, %1\n\ts_waitcnt lgkmcnt(0)" : "=&v"(v) : "v"(addr) : "memory");
    return v;
}
__device__ __forceinline__ unsigned lds_ld1(unsigned addr) {
    unsigned v;
    asm volatile("ds_read_b32 %0, %1\n\ts_waitcnt lgkmcnt(0)" : "=&v"(v) : "v"(addr) : "memory");
    return v;
}
__device__ __forceinline__ void lds_st(unsigned addr, unsigned v) {
    asm volatile("ds_write_b32 %0, %1" ::"v"(addr), "v"(v) : "memory");
}
__device__ __forceinline__ void st_sys32(void* p, unsigned v) {
    asm volatile("global_store_dword %0, %1, off sc0 sc1" ::"v"(p), "v"(v) : "memory");
}
__device__ __forceinline__ void st_sys128(void* p, u32x4 v) {
    asm volatile("global_store_dwordx4 %0, %1, off sc0 sc1\n\ts_nop 1" ::"v"(p), "v"(v) : "memory");
}
__device__ __forceinline__ void st_sys128_w15(void* p, u32x4 v) {
    asm volatile("s_waitcnt vmcnt(15)\n\tglobal_store_dwordx4 %0, %1, off sc0 sc1\n\ts_nop 1" ::"v"(p), "v"(v) : "memory");
}
__device__ __forceinline__ unsigned wave_max_u32(unsigned v) {
    v = max(v, (unsigned)__builtin_amdgcn_update_dpp(0, (int)v, 0x111, 0xf, 0xf, true));
    v = max(v, (unsigned)__builtin_amdgcn_update_dpp(0, (int)v, 0x112, 0xf, 0xf, true));
    v = max(v, (unsigned)__builtin_amdgcn_update_dpp(0, (int)v, 0x114, 0xf, 0xf, true));
    v = max(v, (unsigned)__builtin_amdgcn_update_dpp(0, (int)v, 0x118, 0xf, 0xf, true));
    const unsigned a = __builtin_amdgcn_readlane(v, 15), b = __builtin_amdgcn_readlane(v, 31);
    const unsigned c = __builtin_amdgcn_readlane(v, 47), d = __builtin_amdgcn_readlane(v, 63);
    return max(max(a, b), max(c, d));
}
__device__ __forceinline__ void st_gran(void* p, unsigned v, unsigned tag) {
    u32x2 g = {v, tag};
    asm volatile("global_store_dwordx2 %0, %1, off sc0 sc1" ::"v"(p), "v"(g) : "memory");
}
#define STAMP(i) do {} while (0)

__device__ __forceinline__ s16x8 ld_wfrag(unsigned wa) {
    s16x4 lo = __builtin_amdgcn_ds_read_tr16_b64_v4i16((AS3 s16x4*)(wa));
    s16x4 hi = __builtin_amdgcn_ds_read_tr16_b64_v4i16((AS3 s16x4*)(wa + 1024));
    return __builtin_shufflevector(lo, hi, 0, 1, 2, 3, 4, 5, 6, 7);
}
template <int NU>
__device__ __forceinline__ void compute_batch(const u32x4 (&xb)[8], f32x16 (&acc)[4], unsigned wl, unsigned sb) {
    const unsigned wa[4] = {sb + wl, sb + (wl ^ 32), sb + (wl ^ 128), sb + (wl ^ 160)};
    s16x8 wf[WPF + 1];
#pragma unroll
    for (int f = 0; f < WPF; ++f) wf[f] = ld_wfrag(wa[f & 3] + (f >> 2) * 4096);
    __builtin_amdgcn_sched_barrier(0);
#pragma unroll
    for (int f = 0; f < NU * 4; ++f) {
        if (f + WPF < NU * 4) wf[(f + WPF) % (WPF + 1)] = ld_wfrag(wa[(f + WPF) & 3] + ((f + WPF) >> 2) * 4096);
        __builtin_amdgcn_sched_barrier(0);
        acc[f & 3] = __builtin_amdgcn_mfma_f32_32x32x16_bf16(__builtin_bit_cast(bf16x8, wf[f % (WPF + 1)]), __builtin_bit_cast(bf16x8, xb[f >> 2]), acc[f & 3], 0, 0, 0);
        __builtin_amdgcn_sched_barrier(0);
    }
}

}

__global__ __launch_bounds__(512, 2) void a2a_gemm_kernel(Params P) {
    extern __shared__ __attribute__((aligned(16))) char shm[];
    const int lane = threadIdx.x & 63;
    const int wid = rfl(threadIdx.x >> 6);
    const int b = blockIdx.x;
    const int rank = P.rank;
    char* const my = P.ws[rank];
    const int xcd = b & 7, idx = b >> 3;
    const int hh = xcd >> 2, nt = (xcd & 3) * 8 + (idx & 7), mt = idx >> 3, pb = b ^ 4;
    unsigned* const cntp = (unsigned*)(my + WS_CNT) + b;
    unsigned* const tmop = (unsigned*)(my + WS_TMO);

    if (wid == 0) {
        const unsigned long long t0 = __builtin_amdgcn_s_memrealtime();
        STAMP(0);
        __builtin_amdgcn_s_setprio(3);
        const unsigned p = (unsigned)(rank + xcd) & 7u;
        const int q = idx >> 1, G = idx & 1;
        const int pc = lane >> 4, r = lane & 15;
        const unsigned tq = p == (unsigned)rank ? 0u : POFF + q * PACE + G * (PACE / 2);
        const char* src = P.x + (size_t)(p * 512 + G * 256 + r) * 1024 + pc * 16 + q * 64;
        const bool late = tq > PLEAD;
        if (late) {
            __builtin_amdgcn_s_barrier();
            while ((unsigned)(__builtin_amdgcn_s_memrealtime() - t0) < tq - PLEAD) __builtin_amdgcn_s_sleep(4);
        }
        unsigned cv;
        u32x4 v[16];
        asm volatile("global_load_dword %0, %1, off sc1" : "=&v"(cv) : "v"(cntp) : "memory");
#pragma unroll
        for (int i = 0; i < 16; ++i)
            asm volatile("global_load_dwordx4 %0, %1, off" : "=&v"(v[i]) : "v"(src + i * 16384) : "memory");
        if (!late) __builtin_amdgcn_s_barrier();
        asm volatile("s_waitcnt vmcnt(16)" : "+v"(cv) :: "memory");
        char* const pw = P.ws[p < (unsigned)NR ? p : 0];
        char* dst = pw + WS_STG + (size_t)(rank * NQ + q) * 32768 + G * 16384 + (pc >> 1) * 1024 + (pc & 1) * 512 + r * 16;
        unsigned* flag = (unsigned*)(pw + WS_FLG + (lane & 7) * 2048 + (G * 8 + rank) * 128 + q * 4);
        while ((unsigned)(__builtin_amdgcn_s_memrealtime() - t0) < tq) __builtin_amdgcn_s_sleep(1);
        STAMP(28);
#pragma unroll
        for (int i = 0; i < 16; ++i) st_sys128_w15(dst + (i >> 1) * 2048 + (i & 1) * 256, v[i]);
        STAMP(29);
        wait_vm<0>();
        if (lane < FREP) st_sys32(flag, cv + 1);
        STAMP(2);
        return;
    }
    if (wid == 1) {
        if (lane < 32) lds_st(L_SYNC + lane * 4, 0);
        const unsigned call = rfl(__hip_atomic_load(cntp, __ATOMIC_RELAXED, __HIP_MEMORY_SCOPE_AGENT)) + 1;
        asm volatile("s_waitcnt lgkmcnt(0)" ::: "memory");
        __builtin_amdgcn_s_barrier();
        __builtin_amdgcn_s_setprio(2);
        const int s = lane >> 3, j2 = (lane & 7) * 2;
        const char* fp = my + WS_FLG + (xcd % FREP) * 2048 + ((mt >> 1) * 8 + s) * 128 + j2 * 4;
        const char* xp = my + WS_XFL + pb * 128;
        unsigned level = 0, xrdy = 0, spins = 0;
        const unsigned long long t0 = __builtin_amdgcn_s_memrealtime();
        while (level <= NQ || xrdy != 3) {
            u32x2 f, xf = {0u, 0u};
            if (level > Q1) {
                asm volatile("global_load_dwordx2 %0, %2, off sc0 sc1\n\tglobal_load_dwordx2 %1, %3, off sc0 sc1\n\ts_waitcnt vmcnt(0)"
                             : "=&v"(f), "=&v"(xf) : "v"(fp), "v"(xp) : "memory");
            } else {
                asm volatile("global_load_dwordx2 %0, %1, off sc0 sc1\n\ts_waitcnt vmcnt(0)" : "=&v"(f) : "v"(fp) : "memory");
            }
            unsigned val = f.x < call ? j2 + 1 : (f.y < call ? j2 + 2 : NQ + 1);
            if (s == rank && val <= NQ) val = 0;
            unsigned nl = level;
            while (nl <= NQ && __all(val > nl)) ++nl;
            const unsigned nx = (rfl(xf.x) >= call ? 1u : 0u) | (rfl(xf.y) >= call ? 2u : 0u);
            if (nl != level || nx != xrdy) {
                const bool lchg = nl != level, xchg = nx != xrdy;
                level = nl;
                xrdy = nx;
                if (lane == 0) { lds_st(L_LEVEL, level); lds_st(L_XRDY, xrdy); }
            } else {
                if ((++spins & 255u) == 0 &&
                    (__hip_atomic_load(tmop, __ATOMIC_RELAXED, __HIP_MEMORY_SCOPE_AGENT) == call ||
                     __builtin_amdgcn_s_memrealtime() - t0 > DEADLINE)) {
                    if (lane == 0) {
                        __hip_atomic_store(tmop, call, __ATOMIC_RELAXED, __HIP_MEMORY_SCOPE_AGENT);
                        lds_st(L_TMO, 1);
                        lds_st(L_LEVEL, 99);
                        lds_st(L_XRDY, 3);
                    }
                    break;
                }
                __builtin_amdgcn_s_sleep(MSLEEP);
            }
        }
        if (b == 0) {
            char* const dp = P.ws[lane >> 3] + WS_AM2 + (lane & 7) * 1024 + rank * 8;
            const char* gp = my + WS_AMX + lane * 16;
            unsigned m;
            while (true) {
                u32x4 g0, g1;
                asm volatile("global_load_dwordx4 %0, %2, off sc0 sc1\n\t"
                             "global_load_dwordx4 %1, %2, off offset:1024 sc0 sc1\n\t"
                             "s_waitcnt vmcnt(0)"
                             : "=&v"(g0), "=&v"(g1) : "v"(gp) : "memory");
                m = max(max(g0.x, g0.z), max(g1.x, g1.z));
                if (__all(min(min(g0.y, g0.w), min(g1.y, g1.w)) >= call)) break;
                if ((++spins & 255u) == 0 &&
                    (__hip_atomic_load(tmop, __ATOMIC_RELAXED, __HIP_MEMORY_SCOPE_AGENT) == call ||
                     __builtin_amdgcn_s_memrealtime() - t0 > DEADLINE)) {
                    if (lane == 0) __hip_atomic_store(tmop, call, __ATOMIC_RELAXED, __HIP_MEMORY_SCOPE_AGENT);
                    m = 0x7fffffffu;
                    break;
                }
            }
            st_gran(dp, wave_max_u32(m), call);
        }
        return;
    }
    if (wid < 4) {
        const int lw = wid - 2;
        const auto wr = __builtin_amdgcn_make_buffer_rsrc((void*)P.w, 0, W_BYTES, 0x00020000);
        const int r4 = lane >> 4;
        const unsigned voff = (lw * 8 + r4) * 8192 + nt * 256 + (((lane & 15) ^ ((r4 & 1) << 3) ^ (r4 & 2)) * 16);
        auto unit = [&](unsigned lb, unsigned k0) {
            __builtin_amdgcn_raw_ptr_buffer_load_lds(wr, (AS3 void*)(lb), 16, voff, k0 * 8192, 0, 0);
            __builtin_amdgcn_raw_ptr_buffer_load_lds(wr, (AS3 void*)(lb + 1024), 16, voff, (k0 + 4) * 8192, 0, 0);
        };
        auto fill_local = [&](int c) {
#pragma unroll
            for (int u = 0; u < 8; ++u) unit(c * 32768 + lw * 2048 + u * 4096, rank * 512 + (c * 8 + u) * 32 + hh * 16);
        };
        fill_local(0);
        __builtin_amdgcn_s_barrier();
        fill_local(1);
        wait_vm<16>();
        if (lane == 0) lds_st(L_FULL + lw * 4, 1);
        unsigned slot = 2;
        for (unsigned c = 2; c < (unsigned)NF; ++c) {
            if (c >= 5) {
                while (true) {
                    const u32x4 f = lds_ld4(L_FREE);
                    if (rfl(min(min(f.x, f.y), min(f.z, f.w))) + 5 > c) break;
                    __builtin_amdgcn_s_sleep(1);
                }
            }
            const unsigned kq = c < (unsigned)F0 ? (c - 2) * 32 + hh * 16 : (Q1 + ((c - F0) >> 1)) * 32 + ((hh ^ (c - F0)) & 1) * 16;
            const unsigned lb = slot_base(slot) + lw * 2048;
#pragma unroll
            for (int j = 0; j < 7; ++j) unit(lb + j * 4096, ((rank + 1 + j) & 7) * 512 + kq);
            slot = slot == 4 ? 0 : slot + 1;
            wait_vm<14>();
            if (lane == 0) lds_st(L_FULL + lw * 4, c);
            if (lw == 0) STAMP(96 + c);
        }
        wait_vm<0>();
        if (lane == 0) lds_st(L_FULL + lw * 4, NF);
        return;
    }

    const int gw = wid - 4;
    const unsigned call = rfl(__hip_atomic_load(cntp, __ATOMIC_RELAXED, __HIP_MEMORY_SCOPE_AGENT)) + 1;
    const auto xr = __builtin_amdgcn_make_buffer_rsrc((void*)(my + WS_STG), 0, (unsigned)(STG_BYTES + XCH_BYTES), 0x00020000);
    const unsigned swp = (unsigned)((gw >> 1) ^ hh);
    const unsigned xv1 = (mt * 4 + gw) * 2048 + hh * 1024 + lane * 16;
    const unsigned xv2 = xv1 ^ (swp * (4096u + 1024u));
    unsigned wl;
    {
        const int h = lane >> 5, g1 = (lane >> 4) & 1, q4 = (lane & 15) >> 2, p = lane & 3;
        wl = (8 * h + q4) * 256 + ((128 * (gw >> 1) + 64 * (p & 1) + 16 * g1 + 8 * (p >> 1)) ^ (128 * (q4 & 1)) ^ (32 * (q4 >> 1)));
    }
    f32x16 acc[4] = {};
    u32x4 xa[8], xb[8];
    auto issue_local = [&](u32x4 (&d)[8], int c) {
#pragma unroll
        for (int i = 0; i < 8; ++i) d[i] = __builtin_amdgcn_raw_buffer_load_b128(xr, xv1, (rank * NQ + c * 8 + i) * 32768, 17);
    };
    auto issue_round = [&](u32x4 (&d)[8], int q, unsigned xv) {
#pragma unroll
        for (int j = 0; j < 7; ++j) d[j] = __builtin_amdgcn_raw_buffer_load_b128(xr, xv, (((rank + 1 + j) & 7) * NQ + q) * 32768, 17);
    };
    unsigned level = 0, full = 0, xrdy = 0;
    auto poll = [&]() {
        const u32x4 s = lds_ld4(L_SYNC);
        level = rfl(s.x);
        full = rfl(min(s.y, s.z));
        xrdy = rfl(s.w);
    };
    auto wait_level = [&](unsigned need) { while (level < need) { __builtin_amdgcn_s_sleep(1); poll(); } };
    auto wait_full = [&](unsigned need) { while (full < need) { __builtin_amdgcn_s_sleep(1); poll(); } };
    auto set_free = [&](unsigned n) { if (lane == 0) lds_st(L_FREE + gw * 4, n); };

    __builtin_amdgcn_s_barrier();
    wait_level(1);
    if (gw == 0) STAMP(3);
    issue_local(xa, 0);
    issue_local(xb, 1);
    wait_full(1);
    compute_batch<8>(xa, acc, wl, 0);
    set_free(1);
    poll();
#define STEP(NU, cur, nxt, c, sl, fr, qn, xv)                                                \
    do {                                                                                     \
        if (level >= (unsigned)(qn) + 2) {                                                   \
            issue_round(nxt, qn, xv);                                                        \
            wait_full((c) + 1);                                                              \
            compute_batch<NU>(cur, acc, wl, slot_base(sl));                                  \
            set_free(fr);                                                                    \
            poll();                                                                          \
        } else {                                                                             \
            wait_full((c) + 1);                                                              \
            compute_batch<NU>(cur, acc, wl, slot_base(sl));                                  \
            set_free(fr);                                                                    \
            wait_level((qn) + 2);                                                            \
            issue_round(nxt, qn, xv);                                                        \
        }                                                                                    \
    } while (0)
    STEP(8, xb, xa, 1, 1u, 2, 0, xv1);
    if (gw == 0) STAMP(4);
    unsigned c = 2, slot = 2, cstep = 1;
    u32x4 part[8];
    for (int q = 0; q < NQ; q += 2) {
        if (q == Q1) {
            if (swp) {
                char* xd = my + WS_XCH + (size_t)(b * 2 + (gw & 1)) * 8192 + lane * 16;
#pragma unroll
                for (int i = 0; i < 8; ++i) {
                    u32x4 o;
#pragma unroll
                    for (int e = 0; e < 4; ++e) {
                        const f32x2 t = {acc[i >> 1][(i & 1) * 8 + 2 * e], acc[i >> 1][(i & 1) * 8 + 2 * e + 1]};
                        o[e] = __builtin_bit_cast(unsigned, __builtin_convertvector(t, bf16x2));
                    }
                    st_sys128(xd + i * 1024, o);
                }
#pragma unroll
                for (int i = 0; i < 4; ++i) acc[i] = f32x16{};
                wait_vm<0>();
                if (lane == 0) st_sys32(my + WS_XFL + b * 128 + (gw & 1) * 4, call);
                slot = slot == 4 ? 0 : slot + 1;
                ++c;
            }
            cstep = 2;
            if (gw == 0) STAMP(1);
        }
        const unsigned fadd = q >= Q1 ? 2 - swp : 1;
        STEP(7, xa, xb, c, slot, c + fadd, q + 1, (q + 1 >= Q1 ? xv2 : xv1));
        STAMP(32 + gw * 16 + q);
        c += cstep;
        slot += cstep;
        slot = slot >= 5 ? slot - 5 : slot;
        if (q + 2 < NQ) {
            STEP(7, xb, xa, c, slot, c + fadd, q + 2, (q + 2 >= Q1 ? xv2 : xv1));
            STAMP(32 + gw * 16 + q + 1);
            c += cstep;
            slot += cstep;
            slot = slot >= 5 ? slot - 5 : slot;
        }
    }
    if (swp) {
        while (!(xrdy >> (gw & 1) & 1)) { __builtin_amdgcn_s_sleep(1); poll(); }
#pragma unroll
        for (int i = 0; i < 8; ++i)
            part[i] = __builtin_amdgcn_raw_buffer_load_b128(xr, lane * 16 + (i ^ 4) * 1024, (unsigned)STG_BYTES + (pb * 2 + (gw & 1)) * 8192, 17);
        wait_full(c + 1);
        compute_batch<7>(xb, acc, wl, slot_base(slot));
#pragma unroll
        for (int i = 0; i < 8; ++i)
#pragma unroll
            for (int e = 0; e < 4; ++e) {
                acc[i >> 1][(i & 1) * 8 + 2 * e] += __builtin_bit_cast(float, part[i][e] << 16);
                acc[i >> 1][(i & 1) * 8 + 2 * e + 1] += __builtin_bit_cast(float, part[i][e] & 0xffff0000u);
            }
    } else {
        wait_full(c + 1);
        compute_batch<7>(xb, acc, wl, slot_base(slot));
    }
    STAMP(32 + gw * 16 + 15);
    {
        while (true) {
            const u32x4 f = lds_ld4(L_FREE);
            if (rfl(min(min(f.x, f.y), min(f.z, f.w))) >= (unsigned)NF - 2) break;
        }
        const unsigned mine = slot_base((NF + (gw >> 1)) % 5) + (gw & 1) * 8192 + lane * 16;
        const unsigned other = slot_base((NF + ((gw >> 1) ^ 1)) % 5) + (gw & 1) * 8192 + lane * 16;
#pragma unroll
        for (int i = 0; i < 8; ++i) {
            const f32x4 t = {acc[2 + (i >> 2)][(i & 3) * 4], acc[2 + (i >> 2)][(i & 3) * 4 + 1], acc[2 + (i >> 2)][(i & 3) * 4 + 2], acc[2 + (i >> 2)][(i & 3) * 4 + 3]};
            *(AS3 f32x4*)(mine + i * 1024) = t;
        }
        if (lane == 0) lds_st(L_RED + gw * 4, 1);
        while (rfl(lds_ld1(L_RED + (gw ^ 2) * 4)) == 0) {}
#pragma unroll
        for (int i = 0; i < 8; ++i) {
            const f32x4 t = *(AS3 f32x4*)(other + i * 1024);
#pragma unroll
            for (int e = 0; e < 4; ++e) acc[i >> 2][(i & 3) * 4 + e] += t[e];
        }
    }
    if (gw == 0) STAMP(5);

    float mx = 0.f;
#pragma unroll
    for (int i = 0; i < 16; ++i) {
        acc[0][i] = fmaxf(acc[0][i], 0.f);
        acc[1][i] = fmaxf(acc[1][i], 0.f);
        mx = fmaxf(mx, fmaxf(acc[0][i], acc[1][i]));
    }
    unsigned mu = wave_max_u32(__builtin_bit_cast(unsigned, mx));
    {
        unsigned t = 0;
        if (lane == 0)
            asm volatile("ds_max_u32 %1, %2\n\tds_add_rtn_u32 %0, %1, %3 offset:4\n\ts_waitcnt lgkmcnt(0)"
                         : "=&v"(t) : "v"(L_MAX1), "v"(mu), "v"(1u) : "memory");
        if (rfl(t) == 3) {
            unsigned m = lds_ld1(L_MAX1);
            if (lds_ld1(L_TMO)) m = 0x7fffffffu;
            if (lane == 0) st_gran(my + WS_AMX + b * 8, m, call);
            STAMP(27);
            const char* gp = my + WS_AM2 + xcd * 1024 + (lane & 7) * 8;
            const unsigned long long t0 = __builtin_amdgcn_s_memrealtime();
            unsigned spins = 0;
            while (true) {
                u32x2 g;
                asm volatile("global_load_dwordx2 %0, %1, off sc0 sc1\n\ts_waitcnt vmcnt(0)" : "=&v"(g) : "v"(gp) : "memory");
                m = g.x;
                if (__all(g.y >= call)) break;
                if ((++spins & 255u) == 0 &&
                    (__hip_atomic_load(tmop, __ATOMIC_RELAXED, __HIP_MEMORY_SCOPE_AGENT) == call ||
                     __builtin_amdgcn_s_memrealtime() - t0 > DEADLINE)) {
                    if (lane == 0) __hip_atomic_store(tmop, call, __ATOMIC_RELAXED, __HIP_MEMORY_SCOPE_AGENT);
                    m = 0x7fffffffu;
                    break;
                }
            }
            mu = wave_max_u32(m);
            if (lane == 0) {
                u32x2 r2 = {mu, 1u};
                asm volatile("ds_write_b64 %0, %1" ::"v"(L_MAX2), "v"(r2) : "memory");
            }
        } else {
            u32x2 r2;
            do { r2 = lds_ld2(L_MAX2); } while (rfl(r2.y) == 0);
            mu = r2.x;
        }
    }
    if (gw == 0) STAMP(6);

    const float amax = __builtin_bit_cast(float, mu);
    const float scale = amax / 448.0f;
    const float inv = scale > 0.f ? 1.0f / scale : 0.f;
    u32x4 o[2];
#pragma unroll
    for (int h2 = 0; h2 < 2; ++h2)
#pragma unroll
        for (int i = 0; i < 4; ++i) {
            int w = __builtin_amdgcn_cvt_pk_fp8_f32(acc[h2][4 * i] * inv, acc[h2][4 * i + 1] * inv, 0, false);
            w = __builtin_amdgcn_cvt_pk_fp8_f32(acc[h2][4 * i + 2] * inv, acc[h2][4 * i + 3] * inv, w, true);
            o[h2][i] = mu > 0x7f800000u ? 0x7f7f7f7fu : (unsigned)w;
        }
    char* op = P.out + (size_t)(mt * 128 + (2 * hh + (gw & 1)) * 32 + (lane & 31)) * 4096 + nt * 128 + (gw >> 1) * 64 + (lane >> 5) * 32;
    *(u32x4*)op = o[0];
    *(u32x4*)(op + 16) = o[1];
    if (gw == 0 && lane == 0) __hip_atomic_store(cntp, call, __ATOMIC_RELAXED, __HIP_MEMORY_SCOPE_AGENT);
}

extern "C" void kernel_launch(int rank, int nranks, ncclComm_t comm, hipStream_t stream,
                              void* const* d_in, const int* in_sizes, int n_in,
                              void* d_out, int out_size,
                              const int* in_dtypes, int out_dtype,
                              void* const* d_out_all,
                              void* const* d_ws_all, size_t ws_bytes) {
    if (nranks != NR || (unsigned)rank >= (unsigned)NR || ws_bytes < WS_END || n_in < 2) {
        fprintf(stderr, "unexpected configuration\n");
        return;
    }
    HIP_CHECK(hipFuncSetAttribute((const void*)a2a_gemm_kernel, hipFuncAttributeMaxDynamicSharedMemorySize, LDS_BYTES));
    Params P;
    P.x = (const char*)d_in[0];
    P.w = (const char*)d_in[1];
    P.out = (char*)d_out;
    for (int i = 0; i < NR; ++i) P.ws[i] = (char*)d_ws_all[i];
    P.rank = rank;
    P.pad = 0;
    a2a_gemm_kernel<<<NWG, 512, LDS_BYTES, stream>>>(P);
    HIP_CHECK(hipGetLastError());
}
```

```cpp
#include <hip/hip_runtime.h>
#include <rccl/rccl.h>
#include <cstdio>
#include <cstdint>
#include <type_traits>

#define HIP_CHECK(expr) do { hipError_t _e = (expr); if (_e != hipSuccess) { \
    fprintf(stderr, "HIP error %s at %s:%d\n", hipGetErrorString(_e), __FILE__, __LINE__); \
    return; } } while (0)

#define PACE 62
#define POFF 95
#define PLEAD 300
#define FREP 16
#define FREP2 32
#define LDEPTH 4
#define LSTAG 1
#define CODEPF 0
#define LPRIO 0
#define TWARM 2
#define PLOAD 200
#define GPRIO 2
#define LSLEEP 2
#define XSKIP 1
#define XROT 1
#define GREL 2
#define GREL1 2
#define OST 1
#define FINE 1
#define SLOW0 6
#define MPRIO 2
#define KROT 5
#define PPAUSE 0
#define PHOLD 70
#define PREL 1
#define GHOLD 190
#define GDELAY 0
#define LDELAY 0
#define LSB 0x4444444444448ull
#define LST 0x000064200ull
#define LIF 4
#define LMODE 0
#define L7B 0x77777ull
#define L7T 0x08630ull
#define L7TOUCH 2
#define FDELAY 0
#define FLEAD 9
#define FBATCH 2
#define FRAMP 10
#define LTOUCH 1
#define MSTART 260
#define MSLEEP 12
#define MLOOK 6
#define XMAP 0
#define DBG_TS 0

namespace {

typedef unsigned u32x4 __attribute__((ext_vector_type(4)));
typedef unsigned u32x2 __attribute__((ext_vector_type(2)));
typedef unsigned u32x8 __attribute__((ext_vector_type(8)));
typedef unsigned u32x16 __attribute__((ext_vector_type(16)));
typedef unsigned u32x32 __attribute__((ext_vector_type(32)));
typedef float f32x32 __attribute__((ext_vector_type(32)));
#define AS3 __attribute__((address_space(3)))
#define AS1 __attribute__((address_space(1)))

constexpr int NR = 8;
constexpr int NWG = 256;
constexpr int NQ = 16;
constexpr int NA = 26;
constexpr int IA = 4 * (NA - 4);
constexpr int N2 = 3;
constexpr int NB0 = NA + 2 * N2;
constexpr int NST = NB0 + 112 - IA - 4 * N2;
constexpr int NS = 19;
constexpr unsigned SLOT = 8448;
constexpr unsigned CODE_BYTES = 24704;
constexpr unsigned W_BYTES = 4096u * 4096u * 2u;

constexpr unsigned FRAG = XSKIP ? 2048 : 1024, BLK = 32 * FRAG;
constexpr size_t WS_STG = 0;
constexpr size_t WS_XCH = 8u << 20;
constexpr size_t WS_XFL = WS_XCH + (12u << 20);
constexpr size_t WS_FLG = WS_XFL + 32768;
constexpr size_t WS_AMX = WS_FLG + 131072;
constexpr size_t WS_AM2 = WS_AMX + 4096;
constexpr size_t WS_CNT = WS_AM2 + 8192;
constexpr size_t WS_TMO = WS_CNT + 1024;
constexpr size_t WS_DBG = WS_TMO + 1024;
constexpr size_t WS_END = WS_DBG + 256 * 2048;

constexpr unsigned L_XL = 81920;
constexpr unsigned XL_SLOT = L_XL / SLOT;
constexpr unsigned L_SYNC = NS * SLOT;
constexpr unsigned L_FULL = L_SYNC + 8, L_XRDY = L_SYNC + 16;
constexpr unsigned L_MAX1 = L_SYNC + 96  , L_MAX2 = L_SYNC + 40  ;
constexpr unsigned L_TMO = L_SYNC + 48, L_PST = L_SYNC + 52, L_SHD = L_SYNC + 56  , L_PAU = L_SYNC + 64, L_CALL = L_SYNC + 68, L_HLD = L_SYNC + 72;
constexpr unsigned L_FREE = L_SYNC + 128;
constexpr unsigned L_DBG = L_FREE + 1024;
constexpr unsigned LDS_BYTES = L_DBG + (DBG_TS ? 2048 : 0);
static_assert(LDS_BYTES <= 163840, "LDS");

constexpr unsigned long long DEADLINE = 50000000ull;

struct Params {
    const char* x;
    const char* w;
    char* out;
    char* ws[NR];
    int rank;
    int pad;
};

template <int N> __device__ __forceinline__ void wait_vm() { asm volatile("s_waitcnt vmcnt(%0)" ::"i"(N) : "memory"); }
__device__ __forceinline__ unsigned rfl(unsigned v) { return __builtin_amdgcn_readfirstlane(v); }
__device__ __forceinline__ u32x4 lds_ld4(unsigned addr) {
    u32x4 v;
    asm volatile("ds_read_b128 %0, %1\n\ts_waitcnt lgkmcnt(0)" : "=&v"(v) : "v"(addr) : "memory");
    return v;
}
__device__ __forceinline__ u32x2 lds_ld2(unsigned addr) {
    u32x2 v;
    asm volatile("ds_read_b64 %0, %1\n\ts_waitcnt lgkmcnt(0)" : "=&v"(v) : "v"(addr) : "memory");
    return v;
}
__device__ __forceinline__ unsigned lds_ld1(unsigned addr) {
    unsigned v;
    asm volatile("ds_read_b32 %0, %1\n\ts_waitcnt lgkmcnt(0)" : "=&v"(v) : "v"(addr) : "memory");
    return v;
}
__device__ __forceinline__ void lds_st(unsigned addr, unsigned v) {
    asm volatile("ds_write_b32 %0, %1" ::"v"(addr), "v"(v) : "memory");
}
__device__ __forceinline__ void lds_st2(unsigned addr, unsigned a, unsigned b) {
    u32x2 v = {a, b};
    asm volatile("ds_write_b64 %0, %1" ::"v"(addr), "v"(v) : "memory");
}
__device__ __forceinline__ void lds_max(unsigned addr, unsigned a) {
    asm volatile("ds_max_u32 %0, %1" ::"v"(addr), "v"(a) : "memory");
}
__device__ __forceinline__ void lds_max2(unsigned addr, unsigned a, unsigned b) {
    asm volatile("ds_max_u32 %0, %1\n\tds_max_u32 %0, %2 offset:4" ::"v"(addr), "v"(a), "v"(b) : "memory");
}
__device__ __forceinline__ void st_sys32(void* p, unsigned v) {
    asm volatile("global_store_dword %0, %1, off sc0 sc1" ::"v"(p), "v"(v) : "memory");
}
__device__ __forceinline__ void st_sys128_w15(void* p, u32x4 v) {
    asm volatile("s_waitcnt vmcnt(15)\n\tglobal_store_dwordx4 %0, %1, off sc0 sc1\n\ts_nop 1" ::"v"(p), "v"(v) : "memory");
}
__device__ __forceinline__ unsigned wave_max_u32(unsigned v) {
    v = max(v, (unsigned)__builtin_amdgcn_update_dpp(0, (int)v, 0x111, 0xf, 0xf, true));
    v = max(v, (unsigned)__builtin_amdgcn_update_dpp(0, (int)v, 0x112, 0xf, 0xf, true));
    v = max(v, (unsigned)__builtin_amdgcn_update_dpp(0, (int)v, 0x114, 0xf, 0xf, true));
    v = max(v, (unsigned)__builtin_amdgcn_update_dpp(0, (int)v, 0x118, 0xf, 0xf, true));
    const unsigned a = __builtin_amdgcn_readlane(v, 15), b = __builtin_amdgcn_readlane(v, 31);
    const unsigned c = __builtin_amdgcn_readlane(v, 47), d = __builtin_amdgcn_readlane(v, 63);
    return max(max(a, b), max(c, d));
}
__device__ __forceinline__ void st_gran(void* p, unsigned v, unsigned tag) {
    u32x2 g = {v, tag};
    asm volatile("global_store_dwordx2 %0, %1, off sc0 sc1" ::"v"(p), "v"(g) : "memory");
}
__device__ __forceinline__ unsigned lane_off(int lane) { return lane * 16 + (XSKIP ? (lane < 56 ? 512 : 640) : 0); }
__device__ __forceinline__ unsigned step_block(int k, int c, int rank) {
    if (k < 4) return rank * 16 + 4 * c + k;
    int i = k < NA ? 4 * (k - 4) + c : (k < NB0 ? IA + 2 * (k - NA) + (c & 1) : IA + 4 * N2 + (k - NB0));
    i = i < 111 ? i : 111;
    const int q = i / 7, j = i - 7 * q;
    const int src = (rank + 1 + j) & 7;
    return src * 16 + ((q + KROT * src) & 15);
}
#define STAMP(i) do {} while (0)

#define GEMM_ASM \
    "buffer_load_dword v25, off, s[36:39], s57 sc1\n\t" \
    "v_xor_b32 v28, 0x0, v2\n\t" \
    "v_xor_b32 v29, 0x1000, v2\n\t" \
    "v_xor_b32 v30, 0x2000, v2\n\t" \
    "v_xor_b32 v31, 0x3000, v2\n\t" \
    "v_xor_b32 v32, 0x40, v2\n\t" \
    "v_xor_b32 v33, 0x1040, v2\n\t" \
    "v_xor_b32 v34, 0x2040, v2\n\t" \
    "v_xor_b32 v35, 0x3040, v2\n\t" \
    "v_xor_b32 v36, 0x80, v2\n\t" \
    "v_xor_b32 v37, 0x1080, v2\n\t" \
    "v_xor_b32 v38, 0x2080, v2\n\t" \
    "v_xor_b32 v39, 0x3080, v2\n\t" \
    "v_xor_b32 v40, 0xc0, v2\n\t" \
    "v_xor_b32 v41, 0x10c0, v2\n\t" \
    "v_xor_b32 v42, 0x20c0, v2\n\t" \
    "v_xor_b32 v43, 0x30c0, v2\n\t" \
    "v_mov_b32 v10, 1\n\t" \
    "s_mov_b64 s[64:65], s[36:37]\n\t" \
    "s_mov_b64 s[66:67], s[38:39]\n\t" \
    "s_mov_b32 s46, s38\n\t" \
    "s_mov_b32 s56, 0\n\t" \
    "s_mov_b32 s43, 0\n\t" \
    "s_mov_b32 s40, 4\n\t" \
    "s_mov_b32 s41, 4\n\t" \
    "s_mov_b32 s45, 8448\n\t" \
    "s_mov_b32 s55, 0\n\t" \
    "s_mov_b32 s44, 0\n\t" \
    "v_readlane_b32 s60, v4, s41\n\t" \
    "v_readlane_b32 s61, v5, s41\n\t" \
    "v_readlane_b32 s62, v6, s41\n\t" \
    "v_readlane_b32 s63, v7, s41\n\t" \
    "s_waitcnt vmcnt(0)\n\t" \
    "ds_read_b128 v[48:51], v28\n\t" \
    "ds_read_b128 v[52:55], v29\n\t" \
    "ds_read_b128 v[56:59], v30\n\t" \
    "ds_read_b128 v[60:63], v31\n\t" \
    "ds_read_b128 v[64:67], v32\n\t" \
    "ds_read_b128 v[68:71], v33\n\t" \
    "ds_read_b128 v[72:75], v34\n\t" \
    "ds_read_b128 v[76:79], v35\n\t" \
    "ds_read_b128 v[80:83], v36\n\t" \
    "ds_read_b128 v[84:87], v37\n\t" \
    "ds_read_b128 v[88:91], v38\n\t" \
    "ds_read_b128 v[92:95], v39\n\t" \
    "ds_read_b128 v[96:99], v40\n\t" \
    "ds_read_b128 v[100:103], v41\n\t" \
    "ds_read_b128 v[104:107], v42\n\t" \
    "ds_read_b128 v[108:111], v43\n\t" \
    "v_readfirstlane_b32 s49, v25\n\t" \
    "ds_write_b32 v9, v10\n\t" \
    "s_add_u32 s49, s49, 1\n\t" \
    "s_cmp_gt_u32 s43, s56\n\t" \
    "s_cbranch_scc0 Lwf1_%=\n\t" \
    "Lwb1_%=:\n\t" \
    "v_mov_b32 v2, v1\n\t" \
    "ds_read_b64_tr_b16 v[28:29], v2 offset:0\n\t" \
    "ds_read_b64_tr_b16 v[30:31], v2 offset:256\n\t" \
    "ds_read_b64_tr_b16 v[32:33], v2 offset:32\n\t" \
    "ds_read_b64_tr_b16 v[34:35], v2 offset:288\n\t" \
    "ds_read_b64_tr_b16 v[36:37], v2 offset:64\n\t" \
    "ds_read_b64_tr_b16 v[38:39], v2 offset:320\n\t" \
    "ds_read_b64_tr_b16 v[40:41], v2 offset:96\n\t" \
    "ds_read_b64_tr_b16 v[42:43], v2 offset:352\n\t" \
    "s_mov_b32 s56, 1\n\t" \
    "v_readfirstlane_b32 s42, v12\n\t" \
    "s_waitcnt lgkmcnt(0)\n\t" \
    "v_mfma_f32_16x16x32_bf16 v[128:131], v[28:31], v[48:51], 0\n\t" \
    "ds_read_b64_tr_b16 v[44:45], v2 offset:128\n\t" \
    "s_cmp_eq_u32 s41, s40\n\t" \
    "v_mfma_f32_16x16x32_bf16 v[160:163], v[28:31], v[52:55], 0\n\t" \
    "ds_read_b64_tr_b16 v[46:47], v2 offset:384\n\t" \
    "s_cselect_b32 s47, s42, 0\n\t" \
    "v_mfma_f32_16x16x32_bf16 v[192:195], v[28:31], v[56:59], 0\n\t" \
    "s_cmp_gt_u32 s47, s41\n\t" \
    "s_cselect_b32 s66, s46, 0\n\t" \
    "s_addc_u32 s41, s41, 0\n\t" \
    "v_mfma_f32_16x16x32_bf16 v[224:227], v[28:31], v[60:63], 0\n\t" \
    "buffer_load_dwordx4 v[112:115], v3, s[64:67], s60 offen sc0 sc1\n\t" \
    "s_waitcnt lgkmcnt(6)\n\t" \
    "v_mfma_f32_16x16x32_bf16 v[228:231], v[32:35], v[60:63], 0\n\t" \
    "ds_read_b64_tr_b16 v[28:29], v2 offset:160\n\t" \
    "s_add_u32 s40, s40, 1\n\t" \
    "v_mfma_f32_16x16x32_bf16 v[196:199], v[32:35], v[56:59], 0\n\t" \
    "ds_read_b64_tr_b16 v[30:31], v2 offset:416\n\t" \
    "v_mfma_f32_16x16x32_bf16 v[164:167], v[32:35], v[52:55], 0\n\t" \
    "buffer_load_dwordx4 v[116:119], v3, s[64:67], s61 offen sc0 sc1\n\t" \
    "v_mfma_f32_16x16x32_bf16 v[132:135], v[32:35], v[48:51], 0\n\t" \
    "s_waitcnt lgkmcnt(6)\n\t" \
    "v_mfma_f32_16x16x32_bf16 v[136:139], v[36:39], v[48:51], 0\n\t" \
    "ds_read_b64_tr_b16 v[32:33], v2 offset:192\n\t" \
    "v_mfma_f32_16x16x32_bf16 v[168:171], v[36:39], v[52:55], 0\n\t" \
    "ds_read_b64_tr_b16 v[34:35], v2 offset:448\n\t" \
    "v_mfma_f32_16x16x32_bf16 v[200:203], v[36:39], v[56:59], 0\n\t" \
    "buffer_load_dwordx4 v[120:123], v3, s[64:67], s62 offen sc0 sc1\n\t" \
    "v_mfma_f32_16x16x32_bf16 v[232:235], v[36:39], v[60:63], 0\n\t" \
    "buffer_load_dwordx4 v[124:127], v3, s[64:67], s63 offen sc0 sc1\n\t" \
    "s_waitcnt lgkmcnt(6)\n\t" \
    "v_mfma_f32_16x16x32_bf16 v[236:239], v[40:43], v[60:63], 0\n\t" \
    "ds_read_b64_tr_b16 v[36:37], v2 offset:224\n\t" \
    "v_add_u32 v10, 1, v10\n\t" \
    "v_mfma_f32_16x16x32_bf16 v[204:207], v[40:43], v[56:59], 0\n\t" \
    "ds_read_b64_tr_b16 v[38:39], v2 offset:480\n\t" \
    "s_cmp_gt_u32 s43, s56\n\t" \
    "v_mfma_f32_16x16x32_bf16 v[172:175], v[40:43], v[52:55], 0\n\t" \
    "s_cbranch_scc0 Lwf4_%=\n\t" \
    "Lwb4_%=:\n\t" \
    "v_mfma_f32_16x16x32_bf16 v[140:143], v[40:43], v[48:51], 0\n\t" \
    "v_add_u32 v2, s45, v1\n\t" \
    "ds_read_b128 v[12:15], v8\n\t" \
    "s_waitcnt lgkmcnt(7)\n\t" \
    "v_mfma_f32_16x16x32_bf16 v[144:147], v[44:47], v[48:51], 0\n\t" \
    "ds_read_b64_tr_b16 v[40:41], v2 offset:0\n\t" \
    "s_add_u32 s45, s45, 8448\n\t" \
    "v_mfma_f32_16x16x32_bf16 v[176:179], v[44:47], v[52:55], 0\n\t" \
    "ds_read_b64_tr_b16 v[42:43], v2 offset:256\n\t" \
    "s_cmp_eq_u32 s45, 160512\n\t" \
    "v_mfma_f32_16x16x32_bf16 v[208:211], v[44:47], v[56:59], 0\n\t" \
    "s_cselect_b32 s45, 0, s45\n\t" \
    "s_add_u32 s56, s56, 1\n\t" \
    "v_mfma_f32_16x16x32_bf16 v[240:243], v[44:47], v[60:63], 0\n\t" \
    "ds_write_b32 v9, v10\n\t" \
    "s_waitcnt lgkmcnt(8)\n\t" \
    "v_mfma_f32_16x16x32_bf16 v[244:247], v[28:31], v[60:63], 0\n\t" \
    "ds_read_b64_tr_b16 v[44:45], v2 offset:32\n\t" \
    "v_readlane_b32 s60, v4, s41\n\t" \
    "v_mfma_f32_16x16x32_bf16 v[212:215], v[28:31], v[56:59], 0\n\t" \
    "ds_read_b64_tr_b16 v[46:47], v2 offset:288\n\t" \
    "v_readlane_b32 s61, v5, s41\n\t" \
    "v_mfma_f32_16x16x32_bf16 v[180:183], v[28:31], v[52:55], 0\n\t" \
    "v_readlane_b32 s62, v6, s41\n\t" \
    "v_readlane_b32 s63, v7, s41\n\t" \
    "v_mfma_f32_16x16x32_bf16 v[148:151], v[28:31], v[48:51], 0\n\t" \
    "s_waitcnt lgkmcnt(5)\n\t" \
    "v_mfma_f32_16x16x32_bf16 v[152:155], v[32:35], v[48:51], 0\n\t" \
    "ds_read_b64_tr_b16 v[28:29], v2 offset:64\n\t" \
    "v_mfma_f32_16x16x32_bf16 v[184:187], v[32:35], v[52:55], 0\n\t" \
    "ds_read_b64_tr_b16 v[30:31], v2 offset:320\n\t" \
    "v_min_u32 v14, v14, v15\n\t" \
    "v_mfma_f32_16x16x32_bf16 v[216:219], v[32:35], v[56:59], 0\n\t" \
    "v_readfirstlane_b32 s42, v12\n\t" \
    "v_readfirstlane_b32 s43, v14\n\t" \
    "v_mfma_f32_16x16x32_bf16 v[248:251], v[32:35], v[60:63], 0\n\t" \
    "s_waitcnt lgkmcnt(8)\n\t" \
    "v_mfma_f32_16x16x32_bf16 v[252:255], v[36:39], v[60:63], 0\n\t" \
    "ds_read_b64_tr_b16 v[32:33], v2 offset:96\n\t" \
    "v_mfma_f32_16x16x32_bf16 v[220:223], v[36:39], v[56:59], 0\n\t" \
    "ds_read_b64_tr_b16 v[34:35], v2 offset:352\n\t" \
    "v_mfma_f32_16x16x32_bf16 v[188:191], v[36:39], v[52:55], 0\n\t" \
    "v_mfma_f32_16x16x32_bf16 v[156:159], v[36:39], v[48:51], 0\n\t" \
    "s_waitcnt lgkmcnt(7)\n\t" \
    "s_branch LA1_%=\n\t" \
    "LA0_%=:\n\t" \
    "v_mfma_f32_16x16x32_bf16 v[128:131], v[28:31], v[48:51], v[128:131]\n\t" \
    "ds_read_b64_tr_b16 v[44:45], v2 offset:128\n\t" \
    "s_cmp_eq_u32 s41, s40\n\t" \
    "v_mfma_f32_16x16x32_bf16 v[160:163], v[28:31], v[52:55], v[160:163]\n\t" \
    "ds_read_b64_tr_b16 v[46:47], v2 offset:384\n\t" \
    "s_cselect_b32 s47, s42, 0\n\t" \
    "v_mfma_f32_16x16x32_bf16 v[192:195], v[28:31], v[56:59], v[192:195]\n\t" \
    "s_cmp_gt_u32 s47, s41\n\t" \
    "s_cselect_b32 s66, s46, 0\n\t" \
    "s_addc_u32 s41, s41, 0\n\t" \
    "v_mfma_f32_16x16x32_bf16 v[224:227], v[28:31], v[60:63], v[224:227]\n\t" \
    "buffer_load_dwordx4 v[112:115], v3, s[64:67], s60 offen sc0 sc1\n\t" \
    "s_waitcnt lgkmcnt(6)\n\t" \
    "v_mfma_f32_16x16x32_bf16 v[228:231], v[32:35], v[60:63], v[228:231]\n\t" \
    "ds_read_b64_tr_b16 v[28:29], v2 offset:160\n\t" \
    "s_add_u32 s40, s40, 1\n\t" \
    "v_mfma_f32_16x16x32_bf16 v[196:199], v[32:35], v[56:59], v[196:199]\n\t" \
    "ds_read_b64_tr_b16 v[30:31], v2 offset:416\n\t" \
    "v_mfma_f32_16x16x32_bf16 v[164:167], v[32:35], v[52:55], v[164:167]\n\t" \
    "buffer_load_dwordx4 v[116:119], v3, s[64:67], s61 offen sc0 sc1\n\t" \
    "v_mfma_f32_16x16x32_bf16 v[132:135], v[32:35], v[48:51], v[132:135]\n\t" \
    "s_waitcnt lgkmcnt(6)\n\t" \
    "v_mfma_f32_16x16x32_bf16 v[136:139], v[36:39], v[48:51], v[136:139]\n\t" \
    "ds_read_b64_tr_b16 v[32:33], v2 offset:192\n\t" \
    "v_mfma_f32_16x16x32_bf16 v[168:171], v[36:39], v[52:55], v[168:171]\n\t" \
    "ds_read_b64_tr_b16 v[34:35], v2 offset:448\n\t" \
    "v_mfma_f32_16x16x32_bf16 v[200:203], v[36:39], v[56:59], v[200:203]\n\t" \
    "buffer_load_dwordx4 v[120:123], v3, s[64:67], s62 offen sc0 sc1\n\t" \
    "v_mfma_f32_16x16x32_bf16 v[232:235], v[36:39], v[60:63], v[232:235]\n\t" \
    "buffer_load_dwordx4 v[124:127], v3, s[64:67], s63 offen sc0 sc1\n\t" \
    "s_waitcnt lgkmcnt(6)\n\t" \
    "v_mfma_f32_16x16x32_bf16 v[236:239], v[40:43], v[60:63], v[236:239]\n\t" \
    "ds_read_b64_tr_b16 v[36:37], v2 offset:224\n\t" \
    "v_add_u32 v10, 1, v10\n\t" \
    "v_mfma_f32_16x16x32_bf16 v[204:207], v[40:43], v[56:59], v[204:207]\n\t" \
    "ds_read_b64_tr_b16 v[38:39], v2 offset:480\n\t" \
    "s_cmp_gt_u32 s43, s56\n\t" \
    "v_mfma_f32_16x16x32_bf16 v[172:175], v[40:43], v[52:55], v[172:175]\n\t" \
    "s_cbranch_scc0 Lwf5_%=\n\t" \
    "Lwb5_%=:\n\t" \
    "v_mfma_f32_16x16x32_bf16 v[140:143], v[40:43], v[48:51], v[140:143]\n\t" \
    "v_add_u32 v2, s45, v1\n\t" \
    "ds_read_b128 v[12:15], v8\n\t" \
    "s_waitcnt lgkmcnt(7)\n\t" \
    "v_mfma_f32_16x16x32_bf16 v[144:147], v[44:47], v[48:51], v[144:147]\n\t" \
    "ds_read_b64_tr_b16 v[40:41], v2 offset:0\n\t" \
    "s_add_u32 s45, s45, 8448\n\t" \
    "v_mfma_f32_16x16x32_bf16 v[176:179], v[44:47], v[52:55], v[176:179]\n\t" \
    "ds_read_b64_tr_b16 v[42:43], v2 offset:256\n\t" \
    "s_cmp_eq_u32 s45, 160512\n\t" \
    "v_mfma_f32_16x16x32_bf16 v[208:211], v[44:47], v[56:59], v[208:211]\n\t" \
    "s_cselect_b32 s45, 0, s45\n\t" \
    "s_add_u32 s56, s56, 1\n\t" \
    "v_mfma_f32_16x16x32_bf16 v[240:243], v[44:47], v[60:63], v[240:243]\n\t" \
    "ds_write_b32 v9, v10\n\t" \
    "s_waitcnt lgkmcnt(8)\n\t" \
    "v_mfma_f32_16x16x32_bf16 v[244:247], v[28:31], v[60:63], v[244:247]\n\t" \
    "ds_read_b64_tr_b16 v[44:45], v2 offset:32\n\t" \
    "v_readlane_b32 s60, v4, s41\n\t" \
    "v_mfma_f32_16x16x32_bf16 v[212:215], v[28:31], v[56:59], v[212:215]\n\t" \
    "ds_read_b64_tr_b16 v[46:47], v2 offset:288\n\t" \
    "v_readlane_b32 s61, v5, s41\n\t" \
    "v_mfma_f32_16x16x32_bf16 v[180:183], v[28:31], v[52:55], v[180:183]\n\t" \
    "v_readlane_b32 s62, v6, s41\n\t" \
    "v_readlane_b32 s63, v7, s41\n\t" \
    "v_mfma_f32_16x16x32_bf16 v[148:151], v[28:31], v[48:51], v[148:151]\n\t" \
    "s_waitcnt lgkmcnt(5)\n\t" \
    "v_mfma_f32_16x16x32_bf16 v[152:155], v[32:35], v[48:51], v[152:155]\n\t" \
    "ds_read_b64_tr_b16 v[28:29], v2 offset:64\n\t" \
    "v_mfma_f32_16x16x32_bf16 v[184:187], v[32:35], v[52:55], v[184:187]\n\t" \
    "ds_read_b64_tr_b16 v[30:31], v2 offset:320\n\t" \
    "v_min_u32 v14, v14, v15\n\t" \
    "v_mfma_f32_16x16x32_bf16 v[216:219], v[32:35], v[56:59], v[216:219]\n\t" \
    "v_readfirstlane_b32 s42, v12\n\t" \
    "v_readfirstlane_b32 s43, v14\n\t" \
    "v_mfma_f32_16x16x32_bf16 v[248:251], v[32:35], v[60:63], v[248:251]\n\t" \
    "s_waitcnt lgkmcnt(8)\n\t" \
    "v_mfma_f32_16x16x32_bf16 v[252:255], v[36:39], v[60:63], v[252:255]\n\t" \
    "ds_read_b64_tr_b16 v[32:33], v2 offset:96\n\t" \
    "v_mfma_f32_16x16x32_bf16 v[220:223], v[36:39], v[56:59], v[220:223]\n\t" \
    "ds_read_b64_tr_b16 v[34:35], v2 offset:352\n\t" \
    "v_mfma_f32_16x16x32_bf16 v[188:191], v[36:39], v[52:55], v[188:191]\n\t" \
    "s_cmp_ge_u32 s41, s40\n\t" \
    "s_cbranch_scc0 Lrp1_%=\n\t" \
    "Lbk1_%=:\n\t" \
    "s_waitcnt vmcnt(12)\n\t" \
    "Lbj1_%=:\n\t" \
    "v_mfma_f32_16x16x32_bf16 v[156:159], v[36:39], v[48:51], v[156:159]\n\t" \
    "s_waitcnt lgkmcnt(7)\n\t" \
    "s_cmp_eq_u32 s56, 27\n\t" \
    "s_cbranch_scc1 LAX_%=\n\t" \
    "LA1_%=:\n\t" \
    "v_mfma_f32_16x16x32_bf16 v[128:131], v[40:43], v[64:67], v[128:131]\n\t" \
    "ds_read_b64_tr_b16 v[36:37], v2 offset:128\n\t" \
    "s_cmp_eq_u32 s41, s40\n\t" \
    "v_mfma_f32_16x16x32_bf16 v[160:163], v[40:43], v[68:71], v[160:163]\n\t" \
    "ds_read_b64_tr_b16 v[38:39], v2 offset:384\n\t" \
    "s_cselect_b32 s47, s42, 0\n\t" \
    "v_mfma_f32_16x16x32_bf16 v[192:195], v[40:43], v[72:75], v[192:195]\n\t" \
    "s_cmp_gt_u32 s47, s41\n\t" \
    "s_cselect_b32 s66, s46, 0\n\t" \
    "s_addc_u32 s41, s41, 0\n\t" \
    "v_mfma_f32_16x16x32_bf16 v[224:227], v[40:43], v[76:79], v[224:227]\n\t" \
    "buffer_load_dwordx4 v[48:51], v3, s[64:67], s60 offen sc0 sc1\n\t" \
    "s_waitcnt lgkmcnt(6)\n\t" \
    "v_mfma_f32_16x16x32_bf16 v[228:231], v[44:47], v[76:79], v[228:231]\n\t" \
    "ds_read_b64_tr_b16 v[40:41], v2 offset:160\n\t" \
    "s_add_u32 s40, s40, 1\n\t" \
    "v_mfma_f32_16x16x32_bf16 v[196:199], v[44:47], v[72:75], v[196:199]\n\t" \
    "ds_read_b64_tr_b16 v[42:43], v2 offset:416\n\t" \
    "v_mfma_f32_16x16x32_bf16 v[164:167], v[44:47], v[68:71], v[164:167]\n\t" \
    "buffer_load_dwordx4 v[52:55], v3, s[64:67], s61 offen sc0 sc1\n\t" \
    "v_mfma_f32_16x16x32_bf16 v[132:135], v[44:47], v[64:67], v[132:135]\n\t" \
    "s_waitcnt lgkmcnt(6)\n\t" \
    "v_mfma_f32_16x16x32_bf16 v[136:139], v[28:31], v[64:67], v[136:139]\n\t" \
    "ds_read_b64_tr_b16 v[44:45], v2 offset:192\n\t" \
    "v_mfma_f32_16x16x32_bf16 v[168:171], v[28:31], v[68:71], v[168:171]\n\t" \
    "ds_read_b64_tr_b16 v[46:47], v2 offset:448\n\t" \
    "v_mfma_f32_16x16x32_bf16 v[200:203], v[28:31], v[72:75], v[200:203]\n\t" \
    "buffer_load_dwordx4 v[56:59], v3, s[64:67], s62 offen sc0 sc1\n\t" \
    "v_mfma_f32_16x16x32_bf16 v[232:235], v[28:31], v[76:79], v[232:235]\n\t" \
    "buffer_load_dwordx4 v[60:63], v3, s[64:67], s63 offen sc0 sc1\n\t" \
    "s_waitcnt lgkmcnt(6)\n\t" \
    "v_mfma_f32_16x16x32_bf16 v[236:239], v[32:35], v[76:79], v[236:239]\n\t" \
    "ds_read_b64_tr_b16 v[28:29], v2 offset:224\n\t" \
    "v_add_u32 v10, 1, v10\n\t" \
    "v_mfma_f32_16x16x32_bf16 v[204:207], v[32:35], v[72:75], v[204:207]\n\t" \
    "ds_read_b64_tr_b16 v[30:31], v2 offset:480\n\t" \
    "s_cmp_gt_u32 s43, s56\n\t" \
    "v_mfma_f32_16x16x32_bf16 v[172:175], v[32:35], v[68:71], v[172:175]\n\t" \
    "s_cbranch_scc0 Lwf6_%=\n\t" \
    "Lwb6_%=:\n\t" \
    "v_mfma_f32_16x16x32_bf16 v[140:143], v[32:35], v[64:67], v[140:143]\n\t" \
    "v_add_u32 v2, s45, v1\n\t" \
    "ds_read_b128 v[12:15], v8\n\t" \
    "s_waitcnt lgkmcnt(7)\n\t" \
    "v_mfma_f32_16x16x32_bf16 v[144:147], v[36:39], v[64:67], v[144:147]\n\t" \
    "ds_read_b64_tr_b16 v[32:33], v2 offset:0\n\t" \
    "s_add_u32 s45, s45, 8448\n\t" \
    "v_mfma_f32_16x16x32_bf16 v[176:179], v[36:39], v[68:71], v[176:179]\n\t" \
    "ds_read_b64_tr_b16 v[34:35], v2 offset:256\n\t" \
    "s_cmp_eq_u32 s45, 160512\n\t" \
    "v_mfma_f32_16x16x32_bf16 v[208:211], v[36:39], v[72:75], v[208:211]\n\t" \
    "s_cselect_b32 s45, 0, s45\n\t" \
    "s_add_u32 s56, s56, 1\n\t" \
    "v_mfma_f32_16x16x32_bf16 v[240:243], v[36:39], v[76:79], v[240:243]\n\t" \
    "ds_write_b32 v9, v10\n\t" \
    "s_waitcnt lgkmcnt(8)\n\t" \
    "v_mfma_f32_16x16x32_bf16 v[244:247], v[40:43], v[76:79], v[244:247]\n\t" \
    "ds_read_b64_tr_b16 v[36:37], v2 offset:32\n\t" \
    "v_readlane_b32 s60, v4, s41\n\t" \
    "v_mfma_f32_16x16x32_bf16 v[212:215], v[40:43], v[72:75], v[212:215]\n\t" \
    "ds_read_b64_tr_b16 v[38:39], v2 offset:288\n\t" \
    "v_readlane_b32 s61, v5, s41\n\t" \
    "v_mfma_f32_16x16x32_bf16 v[180:183], v[40:43], v[68:71], v[180:183]\n\t" \
    "v_readlane_b32 s62, v6, s41\n\t" \
    "v_readlane_b32 s63, v7, s41\n\t" \
    "v_mfma_f32_16x16x32_bf16 v[148:151], v[40:43], v[64:67], v[148:151]\n\t" \
    "s_waitcnt lgkmcnt(5)\n\t" \
    "v_mfma_f32_16x16x32_bf16 v[152:155], v[44:47], v[64:67], v[152:155]\n\t" \
    "ds_read_b64_tr_b16 v[40:41], v2 offset:64\n\t" \
    "v_mfma_f32_16x16x32_bf16 v[184:187], v[44:47], v[68:71], v[184:187]\n\t" \
    "ds_read_b64_tr_b16 v[42:43], v2 offset:320\n\t" \
    "v_min_u32 v14, v14, v15\n\t" \
    "v_mfma_f32_16x16x32_bf16 v[216:219], v[44:47], v[72:75], v[216:219]\n\t" \
    "v_readfirstlane_b32 s42, v12\n\t" \
    "v_readfirstlane_b32 s43, v14\n\t" \
    "v_mfma_f32_16x16x32_bf16 v[248:251], v[44:47], v[76:79], v[248:251]\n\t" \
    "s_waitcnt lgkmcnt(8)\n\t" \
    "v_mfma_f32_16x16x32_bf16 v[252:255], v[28:31], v[76:79], v[252:255]\n\t" \
    "ds_read_b64_tr_b16 v[44:45], v2 offset:96\n\t" \
    "v_mfma_f32_16x16x32_bf16 v[220:223], v[28:31], v[72:75], v[220:223]\n\t" \
    "ds_read_b64_tr_b16 v[46:47], v2 offset:352\n\t" \
    "v_mfma_f32_16x16x32_bf16 v[188:191], v[28:31], v[68:71], v[188:191]\n\t" \
    "s_cmp_ge_u32 s41, s40\n\t" \
    "s_cbranch_scc0 Lrp2_%=\n\t" \
    "Lbk2_%=:\n\t" \
    "s_waitcnt vmcnt(12)\n\t" \
    "Lbj2_%=:\n\t" \
    "v_mfma_f32_16x16x32_bf16 v[156:159], v[28:31], v[64:67], v[156:159]\n\t" \
    "s_waitcnt lgkmcnt(7)\n\t" \
    "LA2_%=:\n\t" \
    "v_mfma_f32_16x16x32_bf16 v[128:131], v[32:35], v[80:83], v[128:131]\n\t" \
    "ds_read_b64_tr_b16 v[28:29], v2 offset:128\n\t" \
    "s_cmp_eq_u32 s41, s40\n\t" \
    "v_mfma_f32_16x16x32_bf16 v[160:163], v[32:35], v[84:87], v[160:163]\n\t" \
    "ds_read_b64_tr_b16 v[30:31], v2 offset:384\n\t" \
    "s_cselect_b32 s47, s42, 0\n\t" \
    "v_mfma_f32_16x16x32_bf16 v[192:195], v[32:35], v[88:91], v[192:195]\n\t" \
    "s_cmp_gt_u32 s47, s41\n\t" \
    "s_cselect_b32 s66, s46, 0\n\t" \
    "s_addc_u32 s41, s41, 0\n\t" \
    "v_mfma_f32_16x16x32_bf16 v[224:227], v[32:35], v[92:95], v[224:227]\n\t" \
    "buffer_load_dwordx4 v[64:67], v3, s[64:67], s60 offen sc0 sc1\n\t" \
    "s_waitcnt lgkmcnt(6)\n\t" \
    "v_mfma_f32_16x16x32_bf16 v[228:231], v[36:39], v[92:95], v[228:231]\n\t" \
    "ds_read_b64_tr_b16 v[32:33], v2 offset:160\n\t" \
    "s_add_u32 s40, s40, 1\n\t" \
    "v_mfma_f32_16x16x32_bf16 v[196:199], v[36:39], v[88:91], v[196:199]\n\t" \
    "ds_read_b64_tr_b16 v[34:35], v2 offset:416\n\t" \
    "v_mfma_f32_16x16x32_bf16 v[164:167], v[36:39], v[84:87], v[164:167]\n\t" \
    "buffer_load_dwordx4 v[68:71], v3, s[64:67], s61 offen sc0 sc1\n\t" \
    "v_mfma_f32_16x16x32_bf16 v[132:135], v[36:39], v[80:83], v[132:135]\n\t" \
    "s_waitcnt lgkmcnt(6)\n\t" \
    "v_mfma_f32_16x16x32_bf16 v[136:139], v[40:43], v[80:83], v[136:139]\n\t" \
    "ds_read_b64_tr_b16 v[36:37], v2 offset:192\n\t" \
    "v_mfma_f32_16x16x32_bf16 v[168:171], v[40:43], v[84:87], v[168:171]\n\t" \
    "ds_read_b64_tr_b16 v[38:39], v2 offset:448\n\t" \
    "v_mfma_f32_16x16x32_bf16 v[200:203], v[40:43], v[88:91], v[200:203]\n\t" \
    "buffer_load_dwordx4 v[72:75], v3, s[64:67], s62 offen sc0 sc1\n\t" \
    "v_mfma_f32_16x16x32_bf16 v[232:235], v[40:43], v[92:95], v[232:235]\n\t" \
    "buffer_load_dwordx4 v[76:79], v3, s[64:67], s63 offen sc0 sc1\n\t" \
    "s_waitcnt lgkmcnt(6)\n\t" \
    "v_mfma_f32_16x16x32_bf16 v[236:239], v[44:47], v[92:95], v[236:239]\n\t" \
    "ds_read_b64_tr_b16 v[40:41], v2 offset:224\n\t" \
    "v_add_u32 v10, 1, v10\n\t" \
    "v_mfma_f32_16x16x32_bf16 v[204:207], v[44:47], v[88:91], v[204:207]\n\t" \
    "ds_read_b64_tr_b16 v[42:43], v2 offset:480\n\t" \
    "s_cmp_gt_u32 s43, s56\n\t" \
    "v_mfma_f32_16x16x32_bf16 v[172:175], v[44:47], v[84:87], v[172:175]\n\t" \
    "s_cbranch_scc0 Lwf7_%=\n\t" \
    "Lwb7_%=:\n\t" \
    "v_mfma_f32_16x16x32_bf16 v[140:143], v[44:47], v[80:83], v[140:143]\n\t" \
    "v_add_u32 v2, s45, v1\n\t" \
    "ds_read_b128 v[12:15], v8\n\t" \
    "s_waitcnt lgkmcnt(7)\n\t" \
    "v_mfma_f32_16x16x32_bf16 v[144:147], v[28:31], v[80:83], v[144:147]\n\t" \
    "ds_read_b64_tr_b16 v[44:45], v2 offset:0\n\t" \
    "s_add_u32 s45, s45, 8448\n\t" \
    "v_mfma_f32_16x16x32_bf16 v[176:179], v[28:31], v[84:87], v[176:179]\n\t" \
    "ds_read_b64_tr_b16 v[46:47], v2 offset:256\n\t" \
    "s_cmp_eq_u32 s45, 160512\n\t" \
    "v_mfma_f32_16x16x32_bf16 v[208:211], v[28:31], v[88:91], v[208:211]\n\t" \
    "s_cselect_b32 s45, 0, s45\n\t" \
    "s_add_u32 s56, s56, 1\n\t" \
    "v_mfma_f32_16x16x32_bf16 v[240:243], v[28:31], v[92:95], v[240:243]\n\t" \
    "ds_write_b32 v9, v10\n\t" \
    "s_waitcnt lgkmcnt(8)\n\t" \
    "v_mfma_f32_16x16x32_bf16 v[244:247], v[32:35], v[92:95], v[244:247]\n\t" \
    "ds_read_b64_tr_b16 v[28:29], v2 offset:32\n\t" \
    "v_readlane_b32 s60, v4, s41\n\t" \
    "v_mfma_f32_16x16x32_bf16 v[212:215], v[32:35], v[88:91], v[212:215]\n\t" \
    "ds_read_b64_tr_b16 v[30:31], v2 offset:288\n\t" \
    "v_readlane_b32 s61, v5, s41\n\t" \
    "v_mfma_f32_16x16x32_bf16 v[180:183], v[32:35], v[84:87], v[180:183]\n\t" \
    "v_readlane_b32 s62, v6, s41\n\t" \
    "v_readlane_b32 s63, v7, s41\n\t" \
    "v_mfma_f32_16x16x32_bf16 v[148:151], v[32:35], v[80:83], v[148:151]\n\t" \
    "s_waitcnt lgkmcnt(5)\n\t" \
    "v_mfma_f32_16x16x32_bf16 v[152:155], v[36:39], v[80:83], v[152:155]\n\t" \
    "ds_read_b64_tr_b16 v[32:33], v2 offset:64\n\t" \
    "v_mfma_f32_16x16x32_bf16 v[184:187], v[36:39], v[84:87], v[184:187]\n\t" \
    "ds_read_b64_tr_b16 v[34:35], v2 offset:320\n\t" \
    "v_min_u32 v14, v14, v15\n\t" \
    "v_mfma_f32_16x16x32_bf16 v[216:219], v[36:39], v[88:91], v[216:219]\n\t" \
    "v_readfirstlane_b32 s42, v12\n\t" \
    "v_readfirstlane_b32 s43, v14\n\t" \
    "v_mfma_f32_16x16x32_bf16 v[248:251], v[36:39], v[92:95], v[248:251]\n\t" \
    "s_waitcnt lgkmcnt(8)\n\t" \
    "v_mfma_f32_16x16x32_bf16 v[252:255], v[40:43], v[92:95], v[252:255]\n\t" \
    "ds_read_b64_tr_b16 v[36:37], v2 offset:96\n\t" \
    "v_mfma_f32_16x16x32_bf16 v[220:223], v[40:43], v[88:91], v[220:223]\n\t" \
    "ds_read_b64_tr_b16 v[38:39], v2 offset:352\n\t" \
    "v_mfma_f32_16x16x32_bf16 v[188:191], v[40:43], v[84:87], v[188:191]\n\t" \
    "s_cmp_ge_u32 s41, s40\n\t" \
    "s_cbranch_scc0 Lrp3_%=\n\t" \
    "Lbk3_%=:\n\t" \
    "s_waitcnt vmcnt(12)\n\t" \
    "Lbj3_%=:\n\t" \
    "v_mfma_f32_16x16x32_bf16 v[156:159], v[40:43], v[80:83], v[156:159]\n\t" \
    "s_waitcnt lgkmcnt(7)\n\t" \
    "LA3_%=:\n\t" \
    "v_mfma_f32_16x16x32_bf16 v[128:131], v[44:47], v[96:99], v[128:131]\n\t" \
    "ds_read_b64_tr_b16 v[40:41], v2 offset:128\n\t" \
    "s_cmp_eq_u32 s41, s40\n\t" \
    "v_mfma_f32_16x16x32_bf16 v[160:163], v[44:47], v[100:103], v[160:163]\n\t" \
    "ds_read_b64_tr_b16 v[42:43], v2 offset:384\n\t" \
    "s_cselect_b32 s47, s42, 0\n\t" \
    "v_mfma_f32_16x16x32_bf16 v[192:195], v[44:47], v[104:107], v[192:195]\n\t" \
    "s_cmp_gt_u32 s47, s41\n\t" \
    "s_cselect_b32 s66, s46, 0\n\t" \
    "s_addc_u32 s41, s41, 0\n\t" \
    "v_mfma_f32_16x16x32_bf16 v[224:227], v[44:47], v[108:111], v[224:227]\n\t" \
    "buffer_load_dwordx4 v[80:83], v3, s[64:67], s60 offen sc0 sc1\n\t" \
    "s_waitcnt lgkmcnt(6)\n\t" \
    "v_mfma_f32_16x16x32_bf16 v[228:231], v[28:31], v[108:111], v[228:231]\n\t" \
    "ds_read_b64_tr_b16 v[44:45], v2 offset:160\n\t" \
    "s_add_u32 s40, s40, 1\n\t" \
    "v_mfma_f32_16x16x32_bf16 v[196:199], v[28:31], v[104:107], v[196:199]\n\t" \
    "ds_read_b64_tr_b16 v[46:47], v2 offset:416\n\t" \
    "v_mfma_f32_16x16x32_bf16 v[164:167], v[28:31], v[100:103], v[164:167]\n\t" \
    "buffer_load_dwordx4 v[84:87], v3, s[64:67], s61 offen sc0 sc1\n\t" \
    "v_mfma_f32_16x16x32_bf16 v[132:135], v[28:31], v[96:99], v[132:135]\n\t" \
    "s_waitcnt lgkmcnt(6)\n\t" \
    "v_mfma_f32_16x16x32_bf16 v[136:139], v[32:35], v[96:99], v[136:139]\n\t" \
    "ds_read_b64_tr_b16 v[28:29], v2 offset:192\n\t" \
    "v_mfma_f32_16x16x32_bf16 v[168:171], v[32:35], v[100:103], v[168:171]\n\t" \
    "ds_read_b64_tr_b16 v[30:31], v2 offset:448\n\t" \
    "v_mfma_f32_16x16x32_bf16 v[200:203], v[32:35], v[104:107], v[200:203]\n\t" \
    "buffer_load_dwordx4 v[88:91], v3, s[64:67], s62 offen sc0 sc1\n\t" \
    "v_mfma_f32_16x16x32_bf16 v[232:235], v[32:35], v[108:111], v[232:235]\n\t" \
    "buffer_load_dwordx4 v[92:95], v3, s[64:67], s63 offen sc0 sc1\n\t" \
    "s_waitcnt lgkmcnt(6)\n\t" \
    "v_mfma_f32_16x16x32_bf16 v[236:239], v[36:39], v[108:111], v[236:239]\n\t" \
    "ds_read_b64_tr_b16 v[32:33], v2 offset:224\n\t" \
    "v_add_u32 v10, 1, v10\n\t" \
    "v_mfma_f32_16x16x32_bf16 v[204:207], v[36:39], v[104:107], v[204:207]\n\t" \
    "ds_read_b64_tr_b16 v[34:35], v2 offset:480\n\t" \
    "s_cmp_gt_u32 s43, s56\n\t" \
    "v_mfma_f32_16x16x32_bf16 v[172:175], v[36:39], v[100:103], v[172:175]\n\t" \
    "s_cbranch_scc0 Lwf8_%=\n\t" \
    "Lwb8_%=:\n\t" \
    "v_mfma_f32_16x16x32_bf16 v[140:143], v[36:39], v[96:99], v[140:143]\n\t" \
    "v_add_u32 v2, s45, v1\n\t" \
    "ds_read_b128 v[12:15], v8\n\t" \
    "s_waitcnt lgkmcnt(7)\n\t" \
    "v_mfma_f32_16x16x32_bf16 v[144:147], v[40:43], v[96:99], v[144:147]\n\t" \
    "ds_read_b64_tr_b16 v[36:37], v2 offset:0\n\t" \
    "s_add_u32 s45, s45, 8448\n\t" \
    "v_mfma_f32_16x16x32_bf16 v[176:179], v[40:43], v[100:103], v[176:179]\n\t" \
    "ds_read_b64_tr_b16 v[38:39], v2 offset:256\n\t" \
    "s_cmp_eq_u32 s45, 160512\n\t" \
    "v_mfma_f32_16x16x32_bf16 v[208:211], v[40:43], v[104:107], v[208:211]\n\t" \
    "s_cselect_b32 s45, 0, s45\n\t" \
    "s_add_u32 s56, s56, 1\n\t" \
    "v_mfma_f32_16x16x32_bf16 v[240:243], v[40:43], v[108:111], v[240:243]\n\t" \
    "ds_write_b32 v9, v10\n\t" \
    "s_waitcnt lgkmcnt(8)\n\t" \
    "v_mfma_f32_16x16x32_bf16 v[244:247], v[44:47], v[108:111], v[244:247]\n\t" \
    "ds_read_b64_tr_b16 v[40:41], v2 offset:32\n\t" \
    "v_readlane_b32 s60, v4, s41\n\t" \
    "v_mfma_f32_16x16x32_bf16 v[212:215], v[44:47], v[104:107], v[212:215]\n\t" \
    "ds_read_b64_tr_b16 v[42:43], v2 offset:288\n\t" \
    "v_readlane_b32 s61, v5, s41\n\t" \
    "v_mfma_f32_16x16x32_bf16 v[180:183], v[44:47], v[100:103], v[180:183]\n\t" \
    "v_readlane_b32 s62, v6, s41\n\t" \
    "v_readlane_b32 s63, v7, s41\n\t" \
    "v_mfma_f32_16x16x32_bf16 v[148:151], v[44:47], v[96:99], v[148:151]\n\t" \
    "s_waitcnt lgkmcnt(5)\n\t" \
    "v_mfma_f32_16x16x32_bf16 v[152:155], v[28:31], v[96:99], v[152:155]\n\t" \
    "ds_read_b64_tr_b16 v[44:45], v2 offset:64\n\t" \
    "v_mfma_f32_16x16x32_bf16 v[184:187], v[28:31], v[100:103], v[184:187]\n\t" \
    "ds_read_b64_tr_b16 v[46:47], v2 offset:320\n\t" \
    "v_min_u32 v14, v14, v15\n\t" \
    "v_mfma_f32_16x16x32_bf16 v[216:219], v[28:31], v[104:107], v[216:219]\n\t" \
    "v_readfirstlane_b32 s42, v12\n\t" \
    "v_readfirstlane_b32 s43, v14\n\t" \
    "v_mfma_f32_16x16x32_bf16 v[248:251], v[28:31], v[108:111], v[248:251]\n\t" \
    "s_waitcnt lgkmcnt(8)\n\t" \
    "v_mfma_f32_16x16x32_bf16 v[252:255], v[32:35], v[108:111], v[252:255]\n\t" \
    "ds_read_b64_tr_b16 v[28:29], v2 offset:96\n\t" \
    "v_mfma_f32_16x16x32_bf16 v[220:223], v[32:35], v[104:107], v[220:223]\n\t" \
    "ds_read_b64_tr_b16 v[30:31], v2 offset:352\n\t" \
    "v_mfma_f32_16x16x32_bf16 v[188:191], v[32:35], v[100:103], v[188:191]\n\t" \
    "s_cmp_ge_u32 s41, s40\n\t" \
    "s_cbranch_scc0 Lrp4_%=\n\t" \
    "Lbk4_%=:\n\t" \
    "s_waitcnt vmcnt(12)\n\t" \
    "Lbj4_%=:\n\t" \
    "v_mfma_f32_16x16x32_bf16 v[156:159], v[32:35], v[96:99], v[156:159]\n\t" \
    "s_waitcnt lgkmcnt(7)\n\t" \
    "LA4_%=:\n\t" \
    "v_mfma_f32_16x16x32_bf16 v[128:131], v[36:39], v[112:115], v[128:131]\n\t" \
    "ds_read_b64_tr_b16 v[32:33], v2 offset:128\n\t" \
    "s_cmp_eq_u32 s41, s40\n\t" \
    "v_mfma_f32_16x16x32_bf16 v[160:163], v[36:39], v[116:119], v[160:163]\n\t" \
    "ds_read_b64_tr_b16 v[34:35], v2 offset:384\n\t" \
    "s_cselect_b32 s47, s42, 0\n\t" \
    "v_mfma_f32_16x16x32_bf16 v[192:195], v[36:39], v[120:123], v[192:195]\n\t" \
    "s_cmp_gt_u32 s47, s41\n\t" \
    "s_cselect_b32 s66, s46, 0\n\t" \
    "s_addc_u32 s41, s41, 0\n\t" \
    "v_mfma_f32_16x16x32_bf16 v[224:227], v[36:39], v[124:127], v[224:227]\n\t" \
    "buffer_load_dwordx4 v[96:99], v3, s[64:67], s60 offen sc0 sc1\n\t" \
    "s_waitcnt lgkmcnt(6)\n\t" \
    "v_mfma_f32_16x16x32_bf16 v[228:231], v[40:43], v[124:127], v[228:231]\n\t" \
    "ds_read_b64_tr_b16 v[36:37], v2 offset:160\n\t" \
    "s_add_u32 s40, s40, 1\n\t" \
    "v_mfma_f32_16x16x32_bf16 v[196:199], v[40:43], v[120:123], v[196:199]\n\t" \
    "ds_read_b64_tr_b16 v[38:39], v2 offset:416\n\t" \
    "v_mfma_f32_16x16x32_bf16 v[164:167], v[40:43], v[116:119], v[164:167]\n\t" \
    "buffer_load_dwordx4 v[100:103], v3, s[64:67], s61 offen sc0 sc1\n\t" \
    "v_mfma_f32_16x16x32_bf16 v[132:135], v[40:43], v[112:115], v[132:135]\n\t" \
    "s_waitcnt lgkmcnt(6)\n\t" \
    "v_mfma_f32_16x16x32_bf16 v[136:139], v[44:47], v[112:115], v[136:139]\n\t" \
    "ds_read_b64_tr_b16 v[40:41], v2 offset:192\n\t" \
    "v_mfma_f32_16x16x32_bf16 v[168:171], v[44:47], v[116:119], v[168:171]\n\t" \
    "ds_read_b64_tr_b16 v[42:43], v2 offset:448\n\t" \
    "v_mfma_f32_16x16x32_bf16 v[200:203], v[44:47], v[120:123], v[200:203]\n\t" \
    "buffer_load_dwordx4 v[104:107], v3, s[64:67], s62 offen sc0 sc1\n\t" \
    "v_mfma_f32_16x16x32_bf16 v[232:235], v[44:47], v[124:127], v[232:235]\n\t" \
    "buffer_load_dwordx4 v[108:111], v3, s[64:67], s63 offen sc0 sc1\n\t" \
    "s_waitcnt lgkmcnt(6)\n\t" \
    "v_mfma_f32_16x16x32_bf16 v[236:239], v[28:31], v[124:127], v[236:239]\n\t" \
    "ds_read_b64_tr_b16 v[44:45], v2 offset:224\n\t" \
    "v_add_u32 v10, 1, v10\n\t" \
    "v_mfma_f32_16x16x32_bf16 v[204:207], v[28:31], v[120:123], v[204:207]\n\t" \
    "ds_read_b64_tr_b16 v[46:47], v2 offset:480\n\t" \
    "s_cmp_gt_u32 s43, s56\n\t" \
    "v_mfma_f32_16x16x32_bf16 v[172:175], v[28:31], v[116:119], v[172:175]\n\t" \
    "s_cbranch_scc0 Lwf9_%=\n\t" \
    "Lwb9_%=:\n\t" \
    "v_mfma_f32_16x16x32_bf16 v[140:143], v[28:31], v[112:115], v[140:143]\n\t" \
    "v_add_u32 v2, s45, v1\n\t" \
    "ds_read_b128 v[12:15], v8\n\t" \
    "s_waitcnt lgkmcnt(7)\n\t" \
    "v_mfma_f32_16x16x32_bf16 v[144:147], v[32:35], v[112:115], v[144:147]\n\t" \
    "ds_read_b64_tr_b16 v[28:29], v2 offset:0\n\t" \
    "s_add_u32 s45, s45, 8448\n\t" \
    "v_mfma_f32_16x16x32_bf16 v[176:179], v[32:35], v[116:119], v[176:179]\n\t" \
    "ds_read_b64_tr_b16 v[30:31], v2 offset:256\n\t" \
    "s_cmp_eq_u32 s45, 160512\n\t" \
    "v_mfma_f32_16x16x32_bf16 v[208:211], v[32:35], v[120:123], v[208:211]\n\t" \
    "s_cselect_b32 s45, 0, s45\n\t" \
    "s_add_u32 s56, s56, 1\n\t" \
    "v_mfma_f32_16x16x32_bf16 v[240:243], v[32:35], v[124:127], v[240:243]\n\t" \
    "ds_write_b32 v9, v10\n\t" \
    "s_waitcnt lgkmcnt(8)\n\t" \
    "v_mfma_f32_16x16x32_bf16 v[244:247], v[36:39], v[124:127], v[244:247]\n\t" \
    "ds_read_b64_tr_b16 v[32:33], v2 offset:32\n\t" \
    "v_readlane_b32 s60, v4, s41\n\t" \
    "v_mfma_f32_16x16x32_bf16 v[212:215], v[36:39], v[120:123], v[212:215]\n\t" \
    "ds_read_b64_tr_b16 v[34:35], v2 offset:288\n\t" \
    "v_readlane_b32 s61, v5, s41\n\t" \
    "v_mfma_f32_16x16x32_bf16 v[180:183], v[36:39], v[116:119], v[180:183]\n\t" \
    "v_readlane_b32 s62, v6, s41\n\t" \
    "v_readlane_b32 s63, v7, s41\n\t" \
    "v_mfma_f32_16x16x32_bf16 v[148:151], v[36:39], v[112:115], v[148:151]\n\t" \
    "s_waitcnt lgkmcnt(5)\n\t" \
    "v_mfma_f32_16x16x32_bf16 v[152:155], v[40:43], v[112:115], v[152:155]\n\t" \
    "ds_read_b64_tr_b16 v[36:37], v2 offset:64\n\t" \
    "v_mfma_f32_16x16x32_bf16 v[184:187], v[40:43], v[116:119], v[184:187]\n\t" \
    "ds_read_b64_tr_b16 v[38:39], v2 offset:320\n\t" \
    "v_min_u32 v14, v14, v15\n\t" \
    "v_mfma_f32_16x16x32_bf16 v[216:219], v[40:43], v[120:123], v[216:219]\n\t" \
    "v_readfirstlane_b32 s42, v12\n\t" \
    "v_readfirstlane_b32 s43, v14\n\t" \
    "v_mfma_f32_16x16x32_bf16 v[248:251], v[40:43], v[124:127], v[248:251]\n\t" \
    "s_waitcnt lgkmcnt(8)\n\t" \
    "v_mfma_f32_16x16x32_bf16 v[252:255], v[44:47], v[124:127], v[252:255]\n\t" \
    "ds_read_b64_tr_b16 v[40:41], v2 offset:96\n\t" \
    "v_mfma_f32_16x16x32_bf16 v[220:223], v[44:47], v[120:123], v[220:223]\n\t" \
    "ds_read_b64_tr_b16 v[42:43], v2 offset:352\n\t" \
    "v_mfma_f32_16x16x32_bf16 v[188:191], v[44:47], v[116:119], v[188:191]\n\t" \
    "s_cmp_ge_u32 s41, s40\n\t" \
    "s_cbranch_scc0 Lrp0_%=\n\t" \
    "Lbk0_%=:\n\t" \
    "s_waitcnt vmcnt(12)\n\t" \
    "Lbj0_%=:\n\t" \
    "v_mfma_f32_16x16x32_bf16 v[156:159], v[44:47], v[112:115], v[156:159]\n\t" \
    "s_waitcnt lgkmcnt(7)\n\t" \
    "s_branch LA0_%=\n\t" \
    "LAX_%=:\n\t" \
    "s_waitcnt lgkmcnt(7)\n\t" \
    "v_mfma_f32_16x16x32_bf16 v[128:131], v[40:43], v[64:67], v[128:131]\n\t" \
    "ds_read_b64_tr_b16 v[36:37], v2 offset:128\n\t" \
    "v_mfma_f32_16x16x32_bf16 v[160:163], v[40:43], v[68:71], v[160:163]\n\t" \
    "ds_read_b64_tr_b16 v[38:39], v2 offset:384\n\t" \
    "s_cmp_eq_u32 s41, 30\n\t" \
    "s_cselect_b32 s47, s42, 0\n\t" \
    "s_cmp_gt_u32 s47, s41\n\t" \
    "s_cselect_b32 s66, s46, 0\n\t" \
    "s_addc_u32 s41, s41, 0\n\t" \
    "s_waitcnt lgkmcnt(6)\n\t" \
    "v_mfma_f32_16x16x32_bf16 v[132:135], v[44:47], v[64:67], v[132:135]\n\t" \
    "ds_read_b64_tr_b16 v[40:41], v2 offset:160\n\t" \
    "buffer_load_dwordx4 v[48:51], v3, s[64:67], s60 offen sc0 sc1\n\t" \
    "buffer_load_dwordx4 v[52:55], v3, s[64:67], s61 offen sc0 sc1\n\t" \
    "v_mfma_f32_16x16x32_bf16 v[164:167], v[44:47], v[68:71], v[164:167]\n\t" \
    "ds_read_b64_tr_b16 v[42:43], v2 offset:416\n\t" \
    "s_waitcnt lgkmcnt(6)\n\t" \
    "v_mfma_f32_16x16x32_bf16 v[136:139], v[28:31], v[64:67], v[136:139]\n\t" \
    "ds_read_b64_tr_b16 v[44:45], v2 offset:192\n\t" \
    "buffer_load_dwordx4 v[56:59], v3, s[64:67], s62 offen sc0 sc1\n\t" \
    "buffer_load_dwordx4 v[60:63], v3, s[64:67], s63 offen sc0 sc1\n\t" \
    "v_mfma_f32_16x16x32_bf16 v[168:171], v[28:31], v[68:71], v[168:171]\n\t" \
    "ds_read_b64_tr_b16 v[46:47], v2 offset:448\n\t" \
    "v_cvt_pk_bf16_f32 v224, v224, v225\n\t" \
    "v_cvt_pk_bf16_f32 v225, v226, v227\n\t" \
    "s_waitcnt lgkmcnt(6)\n\t" \
    "v_mfma_f32_16x16x32_bf16 v[140:143], v[32:35], v[64:67], v[140:143]\n\t" \
    "ds_read_b64_tr_b16 v[28:29], v2 offset:224\n\t" \
    "v_cvt_pk_bf16_f32 v226, v228, v229\n\t" \
    "v_cvt_pk_bf16_f32 v227, v230, v231\n\t" \
    "v_mfma_f32_16x16x32_bf16 v[172:175], v[32:35], v[68:71], v[172:175]\n\t" \
    "ds_read_b64_tr_b16 v[30:31], v2 offset:480\n\t" \
    "s_waitcnt lgkmcnt(6)\n\t" \
    "v_mfma_f32_16x16x32_bf16 v[144:147], v[36:39], v[64:67], v[144:147]\n\t" \
    "s_cmp_gt_u32 s43, 28\n\t" \
    "s_cbranch_scc0 Lwf10_%=\n\t" \
    "Lwb10_%=:\n\t" \
    "v_add_u32 v2, 67584, v1\n\t" \
    "ds_read_b64_tr_b16 v[32:33], v2 offset:0\n\t" \
    "global_store_dwordx4 v11, v[224:227], s[72:73] offset:0 sc0 sc1\n\t" \
    "v_cvt_pk_bf16_f32 v232, v232, v233\n\t" \
    "v_mfma_f32_16x16x32_bf16 v[176:179], v[36:39], v[68:71], v[176:179]\n\t" \
    "ds_read_b64_tr_b16 v[34:35], v2 offset:256\n\t" \
    "v_cvt_pk_bf16_f32 v233, v234, v235\n\t" \
    "v_cvt_pk_bf16_f32 v234, v236, v237\n\t" \
    "s_waitcnt lgkmcnt(6)\n\t" \
    "v_mfma_f32_16x16x32_bf16 v[148:151], v[40:43], v[64:67], v[148:151]\n\t" \
    "ds_read_b64_tr_b16 v[228:229], v2 offset:32\n\t" \
    "v_cvt_pk_bf16_f32 v235, v238, v239\n\t" \
    "global_store_dwordx4 v11, v[232:235], s[72:73] offset:1024 sc0 sc1\n\t" \
    "v_mfma_f32_16x16x32_bf16 v[180:183], v[40:43], v[68:71], v[180:183]\n\t" \
    "ds_read_b64_tr_b16 v[230:231], v2 offset:288\n\t" \
    "ds_read_b64_tr_b16 v[36:37], v2 offset:64\n\t" \
    "s_waitcnt lgkmcnt(7)\n\t" \
    "v_mfma_f32_16x16x32_bf16 v[152:155], v[44:47], v[64:67], v[152:155]\n\t" \
    "ds_read_b64_tr_b16 v[38:39], v2 offset:320\n\t" \
    "ds_read_b64_tr_b16 v[236:237], v2 offset:96\n\t" \
    "v_cvt_pk_bf16_f32 v240, v240, v241\n\t" \
    "v_cvt_pk_bf16_f32 v241, v242, v243\n\t" \
    "v_mfma_f32_16x16x32_bf16 v[184:187], v[44:47], v[68:71], v[184:187]\n\t" \
    "ds_read_b64_tr_b16 v[238:239], v2 offset:352\n\t" \
    "v_cvt_pk_bf16_f32 v242, v244, v245\n\t" \
    "v_cvt_pk_bf16_f32 v243, v246, v247\n\t" \
    "s_waitcnt lgkmcnt(8)\n\t" \
    "v_mfma_f32_16x16x32_bf16 v[156:159], v[28:31], v[64:67], v[156:159]\n\t" \
    "ds_read_b64_tr_b16 v[40:41], v2 offset:128\n\t" \
    "global_store_dwordx4 v11, v[240:243], s[72:73] offset:2048 sc0 sc1\n\t" \
    "v_cvt_pk_bf16_f32 v248, v248, v249\n\t" \
    "v_mfma_f32_16x16x32_bf16 v[188:191], v[28:31], v[68:71], v[188:191]\n\t" \
    "ds_read_b64_tr_b16 v[42:43], v2 offset:384\n\t" \
    "ds_read_b64_tr_b16 v[44:45], v2 offset:160\n\t" \
    "s_waitcnt lgkmcnt(9)\n\t" \
    "v_mfma_f32_16x16x32_bf16 v[128:131], v[32:35], v[72:75], v[128:131]\n\t" \
    "ds_read_b64_tr_b16 v[46:47], v2 offset:416\n\t" \
    "ds_read_b64_tr_b16 v[244:245], v2 offset:192\n\t" \
    "v_cvt_pk_bf16_f32 v249, v250, v251\n\t" \
    "v_cvt_pk_bf16_f32 v250, v252, v253\n\t" \
    "v_mfma_f32_16x16x32_bf16 v[160:163], v[32:35], v[76:79], v[160:163]\n\t" \
    "ds_read_b64_tr_b16 v[246:247], v2 offset:448\n\t" \
    "v_cvt_pk_bf16_f32 v251, v254, v255\n\t" \
    "global_store_dwordx4 v11, v[248:251], s[72:73] offset:3072 sc0 sc1\n\t" \
    "s_waitcnt lgkmcnt(8)\n\t" \
    "v_mfma_f32_16x16x32_bf16 v[132:135], v[228:231], v[72:75], v[132:135]\n\t" \
    "ds_read_b64_tr_b16 v[28:29], v2 offset:224\n\t" \
    "v_mfma_f32_16x16x32_bf16 v[164:167], v[228:231], v[76:79], v[164:167]\n\t" \
    "ds_read_b64_tr_b16 v[30:31], v2 offset:480\n\t" \
    "v_mfma_f32_16x16x32_bf16 v[136:139], v[36:39], v[72:75], v[136:139]\n\t" \
    "v_add_u32 v2, 76032, v1\n\t" \
    "ds_read_b64_tr_b16 v[32:33], v2 offset:0\n\t" \
    "v_mfma_f32_16x16x32_bf16 v[168:171], v[36:39], v[76:79], v[168:171]\n\t" \
    "ds_read_b64_tr_b16 v[34:35], v2 offset:256\n\t" \
    "ds_read_b128 v[12:15], v8\n\t" \
    "s_waitcnt lgkmcnt(9)\n\t" \
    "v_mfma_f32_16x16x32_bf16 v[140:143], v[236:239], v[72:75], v[140:143]\n\t" \
    "ds_read_b64_tr_b16 v[228:229], v2 offset:32\n\t" \
    "v_readlane_b32 s60, v4, s41\n\t" \
    "v_mfma_f32_16x16x32_bf16 v[172:175], v[236:239], v[76:79], v[172:175]\n\t" \
    "ds_read_b64_tr_b16 v[230:231], v2 offset:288\n\t" \
    "v_mfma_f32_16x16x32_bf16 v[144:147], v[40:43], v[72:75], v[144:147]\n\t" \
    "ds_read_b64_tr_b16 v[36:37], v2 offset:64\n\t" \
    "v_readlane_b32 s61, v5, s41\n\t" \
    "v_mfma_f32_16x16x32_bf16 v[176:179], v[40:43], v[76:79], v[176:179]\n\t" \
    "ds_read_b64_tr_b16 v[38:39], v2 offset:320\n\t" \
    "v_readlane_b32 s62, v6, s41\n\t" \
    "s_waitcnt lgkmcnt(9)\n\t" \
    "v_mfma_f32_16x16x32_bf16 v[148:151], v[44:47], v[72:75], v[148:151]\n\t" \
    "ds_read_b64_tr_b16 v[236:237], v2 offset:96\n\t" \
    "v_readlane_b32 s63, v7, s41\n\t" \
    "v_mfma_f32_16x16x32_bf16 v[180:183], v[44:47], v[76:79], v[180:183]\n\t" \
    "ds_read_b64_tr_b16 v[238:239], v2 offset:352\n\t" \
    "v_mfma_f32_16x16x32_bf16 v[152:155], v[244:247], v[72:75], v[152:155]\n\t" \
    "ds_read_b64_tr_b16 v[40:41], v2 offset:128\n\t" \
    "v_mfma_f32_16x16x32_bf16 v[184:187], v[244:247], v[76:79], v[184:187]\n\t" \
    "ds_read_b64_tr_b16 v[42:43], v2 offset:384\n\t" \
    "s_waitcnt lgkmcnt(8)\n\t" \
    "v_min_u32 v14, v14, v15\n\t" \
    "v_readfirstlane_b32 s42, v12\n\t" \
    "v_readfirstlane_b32 s43, v14\n\t" \
    "s_waitcnt lgkmcnt(9)\n\t" \
    "v_mfma_f32_16x16x32_bf16 v[156:159], v[28:31], v[72:75], v[156:159]\n\t" \
    "ds_read_b64_tr_b16 v[44:45], v2 offset:160\n\t" \
    "v_mfma_f32_16x16x32_bf16 v[188:191], v[28:31], v[76:79], v[188:191]\n\t" \
    "ds_read_b64_tr_b16 v[46:47], v2 offset:416\n\t" \
    "s_cmp_ge_u32 s41, 31\n\t" \
    "s_cselect_b32 s47, s55, 1\n\t" \
    "s_mov_b32 s55, 0\n\t" \
    "s_cmp_eq_u32 s47, 0\n\t" \
    "s_cbranch_scc0 LfB1_%=\n\t" \
    "s_waitcnt vmcnt(16)\n\t" \
    "LmB1_%=:\n\t" \
    "s_waitcnt lgkmcnt(8)\n\t" \
    "v_mfma_f32_16x16x32_bf16 v[128:131], v[32:35], v[80:83], v[128:131]\n\t" \
    "ds_read_b64_tr_b16 v[244:245], v2 offset:192\n\t" \
    "v_mfma_f32_16x16x32_bf16 v[160:163], v[32:35], v[84:87], v[160:163]\n\t" \
    "ds_read_b64_tr_b16 v[246:247], v2 offset:448\n\t" \
    "s_cmp_eq_u32 s41, 31\n\t" \
    "s_cselect_b32 s47, s42, 0\n\t" \
    "s_cmp_gt_u32 s47, s41\n\t" \
    "s_cselect_b32 s66, s46, 0\n\t" \
    "s_addc_u32 s41, s41, 0\n\t" \
    "v_mfma_f32_16x16x32_bf16 v[132:135], v[228:231], v[80:83], v[132:135]\n\t" \
    "ds_read_b64_tr_b16 v[28:29], v2 offset:224\n\t" \
    "buffer_load_dwordx4 v[64:67], v3, s[64:67], s60 offen sc0 sc1\n\t" \
    "buffer_load_dwordx4 v[68:71], v3, s[64:67], s61 offen sc0 sc1\n\t" \
    "v_mfma_f32_16x16x32_bf16 v[164:167], v[228:231], v[84:87], v[164:167]\n\t" \
    "ds_read_b64_tr_b16 v[30:31], v2 offset:480\n\t" \
    "s_waitcnt lgkmcnt(8)\n\t" \
    "v_mfma_f32_16x16x32_bf16 v[136:139], v[36:39], v[80:83], v[136:139]\n\t" \
    "s_cmp_gt_u32 s43, 30\n\t" \
    "s_cbranch_scc0 Lwf11_%=\n\t" \
    "Lwb11_%=:\n\t" \
    "v_add_u32 v2, 84480, v1\n\t" \
    "ds_read_b64_tr_b16 v[32:33], v2 offset:0\n\t" \
    "buffer_load_dwordx4 v[72:75], v3, s[64:67], s62 offen sc0 sc1\n\t" \
    "buffer_load_dwordx4 v[76:79], v3, s[64:67], s63 offen sc0 sc1\n\t" \
    "v_mfma_f32_16x16x32_bf16 v[168:171], v[36:39], v[84:87], v[168:171]\n\t" \
    "ds_read_b64_tr_b16 v[34:35], v2 offset:256\n\t" \
    "v_cvt_pk_bf16_f32 v192, v192, v193\n\t" \
    "v_cvt_pk_bf16_f32 v193, v194, v195\n\t" \
    "v_mfma_f32_16x16x32_bf16 v[140:143], v[236:239], v[80:83], v[140:143]\n\t" \
    "ds_read_b64_tr_b16 v[228:229], v2 offset:32\n\t" \
    "v_cvt_pk_bf16_f32 v194, v196, v197\n\t" \
    "v_cvt_pk_bf16_f32 v195, v198, v199\n\t" \
    "v_mfma_f32_16x16x32_bf16 v[172:175], v[236:239], v[84:87], v[172:175]\n\t" \
    "ds_read_b64_tr_b16 v[230:231], v2 offset:288\n\t" \
    "s_waitcnt lgkmcnt(8)\n\t" \
    "v_mfma_f32_16x16x32_bf16 v[144:147], v[40:43], v[80:83], v[144:147]\n\t" \
    "ds_read_b64_tr_b16 v[36:37], v2 offset:64\n\t" \
    "global_store_dwordx4 v11, v[192:195], s[70:71] offset:0 sc0 sc1\n\t" \
    "v_cvt_pk_bf16_f32 v200, v200, v201\n\t" \
    "v_mfma_f32_16x16x32_bf16 v[176:179], v[40:43], v[84:87], v[176:179]\n\t" \
    "ds_read_b64_tr_b16 v[38:39], v2 offset:320\n\t" \
    "v_cvt_pk_bf16_f32 v201, v202, v203\n\t" \
    "v_cvt_pk_bf16_f32 v202, v204, v205\n\t" \
    "v_mfma_f32_16x16x32_bf16 v[148:151], v[44:47], v[80:83], v[148:151]\n\t" \
    "ds_read_b64_tr_b16 v[236:237], v2 offset:96\n\t" \
    "v_cvt_pk_bf16_f32 v203, v206, v207\n\t" \
    "global_store_dwordx4 v11, v[200:203], s[70:71] offset:1024 sc0 sc1\n\t" \
    "v_mfma_f32_16x16x32_bf16 v[180:183], v[44:47], v[84:87], v[180:183]\n\t" \
    "ds_read_b64_tr_b16 v[238:239], v2 offset:352\n\t" \
    "s_waitcnt lgkmcnt(8)\n\t" \
    "v_mfma_f32_16x16x32_bf16 v[152:155], v[244:247], v[80:83], v[152:155]\n\t" \
    "ds_read_b64_tr_b16 v[40:41], v2 offset:128\n\t" \
    "v_cvt_pk_bf16_f32 v208, v208, v209\n\t" \
    "v_cvt_pk_bf16_f32 v209, v210, v211\n\t" \
    "v_mfma_f32_16x16x32_bf16 v[184:187], v[244:247], v[84:87], v[184:187]\n\t" \
    "ds_read_b64_tr_b16 v[42:43], v2 offset:384\n\t" \
    "v_cvt_pk_bf16_f32 v210, v212, v213\n\t" \
    "v_cvt_pk_bf16_f32 v211, v214, v215\n\t" \
    "v_mfma_f32_16x16x32_bf16 v[156:159], v[28:31], v[80:83], v[156:159]\n\t" \
    "ds_read_b64_tr_b16 v[44:45], v2 offset:160\n\t" \
    "global_store_dwordx4 v11, v[208:211], s[70:71] offset:2048 sc0 sc1\n\t" \
    "v_cvt_pk_bf16_f32 v216, v216, v217\n\t" \
    "v_mfma_f32_16x16x32_bf16 v[188:191], v[28:31], v[84:87], v[188:191]\n\t" \
    "ds_read_b64_tr_b16 v[46:47], v2 offset:416\n\t" \
    "s_waitcnt lgkmcnt(8)\n\t" \
    "v_mfma_f32_16x16x32_bf16 v[128:131], v[32:35], v[88:91], v[128:131]\n\t" \
    "ds_read_b64_tr_b16 v[244:245], v2 offset:192\n\t" \
    "v_cvt_pk_bf16_f32 v217, v218, v219\n\t" \
    "v_cvt_pk_bf16_f32 v218, v220, v221\n\t" \
    "v_mfma_f32_16x16x32_bf16 v[160:163], v[32:35], v[92:95], v[160:163]\n\t" \
    "ds_read_b64_tr_b16 v[246:247], v2 offset:448\n\t" \
    "v_cvt_pk_bf16_f32 v219, v222, v223\n\t" \
    "global_store_dwordx4 v11, v[216:219], s[70:71] offset:3072 sc0 sc1\n\t" \
    "v_mfma_f32_16x16x32_bf16 v[132:135], v[228:231], v[88:91], v[132:135]\n\t" \
    "ds_read_b64_tr_b16 v[28:29], v2 offset:224\n\t" \
    "v_mfma_f32_16x16x32_bf16 v[164:167], v[228:231], v[92:95], v[164:167]\n\t" \
    "ds_read_b64_tr_b16 v[30:31], v2 offset:480\n\t" \
    "s_waitcnt lgkmcnt(8)\n\t" \
    "v_mfma_f32_16x16x32_bf16 v[136:139], v[36:39], v[88:91], v[136:139]\n\t" \
    "v_add_u32 v2, 92928, v1\n\t" \
    "ds_read_b64_tr_b16 v[32:33], v2 offset:0\n\t" \
    "v_mfma_f32_16x16x32_bf16 v[168:171], v[36:39], v[92:95], v[168:171]\n\t" \
    "ds_read_b64_tr_b16 v[34:35], v2 offset:256\n\t" \
    "v_mfma_f32_16x16x32_bf16 v[140:143], v[236:239], v[88:91], v[140:143]\n\t" \
    "ds_read_b64_tr_b16 v[228:229], v2 offset:32\n\t" \
    "v_mfma_f32_16x16x32_bf16 v[172:175], v[236:239], v[92:95], v[172:175]\n\t" \
    "ds_read_b64_tr_b16 v[230:231], v2 offset:288\n\t" \
    "s_waitcnt lgkmcnt(8)\n\t" \
    "v_mfma_f32_16x16x32_bf16 v[144:147], v[40:43], v[88:91], v[144:147]\n\t" \
    "ds_read_b64_tr_b16 v[36:37], v2 offset:64\n\t" \
    "ds_read_b128 v[12:15], v8\n\t" \
    "v_mfma_f32_16x16x32_bf16 v[176:179], v[40:43], v[92:95], v[176:179]\n\t" \
    "ds_read_b64_tr_b16 v[38:39], v2 offset:320\n\t" \
    "v_mfma_f32_16x16x32_bf16 v[148:151], v[44:47], v[88:91], v[148:151]\n\t" \
    "ds_read_b64_tr_b16 v[236:237], v2 offset:96\n\t" \
    "v_mfma_f32_16x16x32_bf16 v[180:183], v[44:47], v[92:95], v[180:183]\n\t" \
    "ds_read_b64_tr_b16 v[238:239], v2 offset:352\n\t" \
    "s_waitcnt lgkmcnt(9)\n\t" \
    "v_mfma_f32_16x16x32_bf16 v[152:155], v[244:247], v[88:91], v[152:155]\n\t" \
    "ds_read_b64_tr_b16 v[40:41], v2 offset:128\n\t" \
    "v_mfma_f32_16x16x32_bf16 v[184:187], v[244:247], v[92:95], v[184:187]\n\t" \
    "ds_read_b64_tr_b16 v[42:43], v2 offset:384\n\t" \
    "s_waitcnt lgkmcnt(5)\n\t" \
    "v_min_u32 v14, v14, v15\n\t" \
    "v_readfirstlane_b32 s42, v12\n\t" \
    "v_readfirstlane_b32 s43, v14\n\t" \
    "v_mfma_f32_16x16x32_bf16 v[156:159], v[28:31], v[88:91], v[156:159]\n\t" \
    "ds_read_b64_tr_b16 v[44:45], v2 offset:160\n\t" \
    "v_mfma_f32_16x16x32_bf16 v[188:191], v[28:31], v[92:95], v[188:191]\n\t" \
    "ds_read_b64_tr_b16 v[46:47], v2 offset:416\n\t" \
    "s_cmp_ge_u32 s41, 32\n\t" \
    "s_cselect_b32 s47, s55, 1\n\t" \
    "s_mov_b32 s55, 0\n\t" \
    "s_cmp_eq_u32 s47, 0\n\t" \
    "s_cbranch_scc0 LfB2_%=\n\t" \
    "s_waitcnt vmcnt(20)\n\t" \
    "LmB2_%=:\n\t" \
    "s_waitcnt lgkmcnt(9)\n\t" \
    "v_mfma_f32_16x16x32_bf16 v[128:131], v[32:35], v[96:99], v[128:131]\n\t" \
    "ds_read_b64_tr_b16 v[244:245], v2 offset:192\n\t" \
    "v_mfma_f32_16x16x32_bf16 v[160:163], v[32:35], v[100:103], v[160:163]\n\t" \
    "ds_read_b64_tr_b16 v[246:247], v2 offset:448\n\t" \
    "v_mfma_f32_16x16x32_bf16 v[132:135], v[228:231], v[96:99], v[132:135]\n\t" \
    "ds_read_b64_tr_b16 v[28:29], v2 offset:224\n\t" \
    "v_mfma_f32_16x16x32_bf16 v[164:167], v[228:231], v[100:103], v[164:167]\n\t" \
    "ds_read_b64_tr_b16 v[30:31], v2 offset:480\n\t" \
    "s_waitcnt lgkmcnt(8)\n\t" \
    "v_mfma_f32_16x16x32_bf16 v[136:139], v[36:39], v[96:99], v[136:139]\n\t" \
    "s_cmp_gt_u32 s43, 32\n\t" \
    "s_cbranch_scc0 Lwf12_%=\n\t" \
    "Lwb12_%=:\n\t" \
    "v_add_u32 v2, 101376, v1\n\t" \
    "ds_read_b64_tr_b16 v[32:33], v2 offset:0\n\t" \
    "v_mfma_f32_16x16x32_bf16 v[168:171], v[36:39], v[100:103], v[168:171]\n\t" \
    "ds_read_b64_tr_b16 v[34:35], v2 offset:256\n\t" \
    "v_mfma_f32_16x16x32_bf16 v[140:143], v[236:239], v[96:99], v[140:143]\n\t" \
    "ds_read_b64_tr_b16 v[228:229], v2 offset:32\n\t" \
    "v_mfma_f32_16x16x32_bf16 v[172:175], v[236:239], v[100:103], v[172:175]\n\t" \
    "ds_read_b64_tr_b16 v[230:231], v2 offset:288\n\t" \
    "s_waitcnt lgkmcnt(8)\n\t" \
    "v_mfma_f32_16x16x32_bf16 v[144:147], v[40:43], v[96:99], v[144:147]\n\t" \
    "ds_read_b64_tr_b16 v[36:37], v2 offset:64\n\t" \
    "v_mfma_f32_16x16x32_bf16 v[176:179], v[40:43], v[100:103], v[176:179]\n\t" \
    "ds_read_b64_tr_b16 v[38:39], v2 offset:320\n\t" \
    "v_mfma_f32_16x16x32_bf16 v[148:151], v[44:47], v[96:99], v[148:151]\n\t" \
    "ds_read_b64_tr_b16 v[236:237], v2 offset:96\n\t" \
    "v_mfma_f32_16x16x32_bf16 v[180:183], v[44:47], v[100:103], v[180:183]\n\t" \
    "ds_read_b64_tr_b16 v[238:239], v2 offset:352\n\t" \
    "s_waitcnt lgkmcnt(8)\n\t" \
    "v_mfma_f32_16x16x32_bf16 v[152:155], v[244:247], v[96:99], v[152:155]\n\t" \
    "ds_read_b64_tr_b16 v[40:41], v2 offset:128\n\t" \
    "v_mfma_f32_16x16x32_bf16 v[184:187], v[244:247], v[100:103], v[184:187]\n\t" \
    "ds_read_b64_tr_b16 v[42:43], v2 offset:384\n\t" \
    "v_mfma_f32_16x16x32_bf16 v[156:159], v[28:31], v[96:99], v[156:159]\n\t" \
    "ds_read_b64_tr_b16 v[44:45], v2 offset:160\n\t" \
    "v_mfma_f32_16x16x32_bf16 v[188:191], v[28:31], v[100:103], v[188:191]\n\t" \
    "ds_read_b64_tr_b16 v[46:47], v2 offset:416\n\t" \
    "s_waitcnt lgkmcnt(8)\n\t" \
    "v_mfma_f32_16x16x32_bf16 v[128:131], v[32:35], v[104:107], v[128:131]\n\t" \
    "ds_read_b64_tr_b16 v[244:245], v2 offset:192\n\t" \
    "v_mfma_f32_16x16x32_bf16 v[160:163], v[32:35], v[108:111], v[160:163]\n\t" \
    "ds_read_b64_tr_b16 v[246:247], v2 offset:448\n\t" \
    "v_mfma_f32_16x16x32_bf16 v[132:135], v[228:231], v[104:107], v[132:135]\n\t" \
    "ds_read_b64_tr_b16 v[28:29], v2 offset:224\n\t" \
    "v_mfma_f32_16x16x32_bf16 v[164:167], v[228:231], v[108:111], v[164:167]\n\t" \
    "ds_read_b64_tr_b16 v[30:31], v2 offset:480\n\t" \
    "s_waitcnt lgkmcnt(8)\n\t" \
    "v_mfma_f32_16x16x32_bf16 v[136:139], v[36:39], v[104:107], v[136:139]\n\t" \
    "v_add_u32 v2, 109824, v1\n\t" \
    "ds_read_b64_tr_b16 v[32:33], v2 offset:0\n\t" \
    "v_mfma_f32_16x16x32_bf16 v[168:171], v[36:39], v[108:111], v[168:171]\n\t" \
    "ds_read_b64_tr_b16 v[34:35], v2 offset:256\n\t" \
    "v_mfma_f32_16x16x32_bf16 v[140:143], v[236:239], v[104:107], v[140:143]\n\t" \
    "ds_read_b64_tr_b16 v[228:229], v2 offset:32\n\t" \
    "v_mfma_f32_16x16x32_bf16 v[172:175], v[236:239], v[108:111], v[172:175]\n\t" \
    "ds_read_b64_tr_b16 v[230:231], v2 offset:288\n\t" \
    "s_waitcnt lgkmcnt(8)\n\t" \
    "v_mfma_f32_16x16x32_bf16 v[144:147], v[40:43], v[104:107], v[144:147]\n\t" \
    "ds_read_b64_tr_b16 v[36:37], v2 offset:64\n\t" \
    "ds_read_b128 v[12:15], v8\n\t" \
    "v_mfma_f32_16x16x32_bf16 v[176:179], v[40:43], v[108:111], v[176:179]\n\t" \
    "ds_read_b64_tr_b16 v[38:39], v2 offset:320\n\t" \
    "v_mfma_f32_16x16x32_bf16 v[148:151], v[44:47], v[104:107], v[148:151]\n\t" \
    "ds_read_b64_tr_b16 v[236:237], v2 offset:96\n\t" \
    "v_mfma_f32_16x16x32_bf16 v[180:183], v[44:47], v[108:111], v[180:183]\n\t" \
    "ds_read_b64_tr_b16 v[238:239], v2 offset:352\n\t" \
    "s_waitcnt lgkmcnt(9)\n\t" \
    "v_mfma_f32_16x16x32_bf16 v[152:155], v[244:247], v[104:107], v[152:155]\n\t" \
    "ds_read_b64_tr_b16 v[40:41], v2 offset:128\n\t" \
    "v_mfma_f32_16x16x32_bf16 v[184:187], v[244:247], v[108:111], v[184:187]\n\t" \
    "ds_read_b64_tr_b16 v[42:43], v2 offset:384\n\t" \
    "s_waitcnt lgkmcnt(5)\n\t" \
    "v_min_u32 v14, v14, v15\n\t" \
    "v_readfirstlane_b32 s42, v12\n\t" \
    "v_readfirstlane_b32 s43, v14\n\t" \
    "v_mfma_f32_16x16x32_bf16 v[156:159], v[28:31], v[104:107], v[156:159]\n\t" \
    "ds_read_b64_tr_b16 v[44:45], v2 offset:160\n\t" \
    "v_mfma_f32_16x16x32_bf16 v[188:191], v[28:31], v[108:111], v[188:191]\n\t" \
    "ds_read_b64_tr_b16 v[46:47], v2 offset:416\n\t" \
    "s_waitcnt vmcnt(0)\n\t" \
    "v_mov_b32 v25, s49\n\t" \
    "s_mov_b64 exec, 6\n\t" \
    "buffer_store_dword v25, v24, s[36:39], 0 offen sc0 sc1\n\t" \
    "s_mov_b64 exec, -1\n\t" \
    "s_cmp_ge_u32 s41, 32\n\t" \
    "s_cselect_b32 s47, s55, 1\n\t" \
    "s_mov_b32 s55, 0\n\t" \
    "s_cmp_eq_u32 s47, 0\n\t" \
    "s_cbranch_scc0 LfB3_%=\n\t" \
    "s_waitcnt vmcnt(17)\n\t" \
    "LmB3_%=:\n\t" \
    "s_waitcnt lgkmcnt(9)\n\t" \
    "v_mfma_f32_16x16x32_bf16 v[128:131], v[32:35], v[112:115], v[128:131]\n\t" \
    "ds_read_b64_tr_b16 v[244:245], v2 offset:192\n\t" \
    "ds_read_b64_tr_b16 v[246:247], v2 offset:448\n\t" \
    "v_mfma_f32_16x16x32_bf16 v[132:135], v[228:231], v[112:115], v[132:135]\n\t" \
    "ds_read_b64_tr_b16 v[28:29], v2 offset:224\n\t" \
    "ds_read_b64_tr_b16 v[30:31], v2 offset:480\n\t" \
    "v_cvt_pk_bf16_f32 v160, v160, v161\n\t" \
    "v_cvt_pk_bf16_f32 v161, v162, v163\n\t" \
    "v_cvt_pk_bf16_f32 v162, v164, v165\n\t" \
    "v_cvt_pk_bf16_f32 v163, v166, v167\n\t" \
    "global_store_dwordx4 v11, v[160:163], s[68:69] offset:0 sc0 sc1\n\t" \
    "s_waitcnt lgkmcnt(8)\n\t" \
    "v_mfma_f32_16x16x32_bf16 v[136:139], v[36:39], v[112:115], v[136:139]\n\t" \
    "s_cmp_gt_u32 s43, 36\n\t" \
    "s_cbranch_scc0 Lwf13_%=\n\t" \
    "Lwb13_%=:\n\t" \
    "v_add_u32 v2, 118272, v1\n\t" \
    "ds_read_b64_tr_b16 v[32:33], v2 offset:0\n\t" \
    "ds_read_b64_tr_b16 v[34:35], v2 offset:256\n\t" \
    "v_cvt_pk_bf16_f32 v168, v168, v169\n\t" \
    "v_cvt_pk_bf16_f32 v169, v170, v171\n\t" \
    "v_cvt_pk_bf16_f32 v170, v172, v173\n\t" \
    "v_cvt_pk_bf16_f32 v171, v174, v175\n\t" \
    "global_store_dwordx4 v11, v[168:171], s[68:69] offset:1024 sc0 sc1\n\t" \
    "v_mfma_f32_16x16x32_bf16 v[140:143], v[236:239], v[112:115], v[140:143]\n\t" \
    "ds_read_b64_tr_b16 v[228:229], v2 offset:32\n\t" \
    "ds_read_b64_tr_b16 v[230:231], v2 offset:288\n\t" \
    "v_cvt_pk_bf16_f32 v176, v176, v177\n\t" \
    "v_cvt_pk_bf16_f32 v177, v178, v179\n\t" \
    "v_cvt_pk_bf16_f32 v178, v180, v181\n\t" \
    "v_cvt_pk_bf16_f32 v179, v182, v183\n\t" \
    "global_store_dwordx4 v11, v[176:179], s[68:69] offset:2048 sc0 sc1\n\t" \
    "s_waitcnt lgkmcnt(8)\n\t" \
    "v_mfma_f32_16x16x32_bf16 v[144:147], v[40:43], v[112:115], v[144:147]\n\t" \
    "ds_read_b64_tr_b16 v[36:37], v2 offset:64\n\t" \
    "ds_read_b64_tr_b16 v[38:39], v2 offset:320\n\t" \
    "v_cvt_pk_bf16_f32 v184, v184, v185\n\t" \
    "v_cvt_pk_bf16_f32 v185, v186, v187\n\t" \
    "v_cvt_pk_bf16_f32 v186, v188, v189\n\t" \
    "v_cvt_pk_bf16_f32 v187, v190, v191\n\t" \
    "global_store_dwordx4 v11, v[184:187], s[68:69] offset:3072 sc0 sc1\n\t" \
    "v_mfma_f32_16x16x32_bf16 v[148:151], v[44:47], v[112:115], v[148:151]\n\t" \
    "ds_read_b64_tr_b16 v[236:237], v2 offset:96\n\t" \
    "ds_read_b64_tr_b16 v[238:239], v2 offset:352\n\t" \
    "s_waitcnt lgkmcnt(8)\n\t" \
    "v_mfma_f32_16x16x32_bf16 v[152:155], v[244:247], v[112:115], v[152:155]\n\t" \
    "ds_read_b64_tr_b16 v[40:41], v2 offset:128\n\t" \
    "ds_read_b64_tr_b16 v[42:43], v2 offset:384\n\t" \
    "v_mfma_f32_16x16x32_bf16 v[156:159], v[28:31], v[112:115], v[156:159]\n\t" \
    "ds_read_b64_tr_b16 v[44:45], v2 offset:160\n\t" \
    "ds_read_b64_tr_b16 v[46:47], v2 offset:416\n\t" \
    "s_waitcnt lgkmcnt(8)\n\t" \
    "v_mfma_f32_16x16x32_bf16 v[128:131], v[32:35], v[116:119], v[128:131]\n\t" \
    "ds_read_b64_tr_b16 v[244:245], v2 offset:192\n\t" \
    "ds_read_b64_tr_b16 v[246:247], v2 offset:448\n\t" \
    "v_mfma_f32_16x16x32_bf16 v[132:135], v[228:231], v[116:119], v[132:135]\n\t" \
    "ds_read_b64_tr_b16 v[28:29], v2 offset:224\n\t" \
    "ds_read_b64_tr_b16 v[30:31], v2 offset:480\n\t" \
    "s_waitcnt lgkmcnt(8)\n\t" \
    "v_mfma_f32_16x16x32_bf16 v[136:139], v[36:39], v[116:119], v[136:139]\n\t" \
    "v_add_u32 v2, 126720, v1\n\t" \
    "ds_read_b64_tr_b16 v[32:33], v2 offset:0\n\t" \
    "ds_read_b64_tr_b16 v[34:35], v2 offset:256\n\t" \
    "v_mfma_f32_16x16x32_bf16 v[140:143], v[236:239], v[116:119], v[140:143]\n\t" \
    "ds_read_b64_tr_b16 v[228:229], v2 offset:32\n\t" \
    "ds_read_b64_tr_b16 v[230:231], v2 offset:288\n\t" \
    "s_waitcnt lgkmcnt(8)\n\t" \
    "v_mfma_f32_16x16x32_bf16 v[144:147], v[40:43], v[116:119], v[144:147]\n\t" \
    "ds_read_b64_tr_b16 v[36:37], v2 offset:64\n\t" \
    "ds_read_b64_tr_b16 v[38:39], v2 offset:320\n\t" \
    "v_mfma_f32_16x16x32_bf16 v[148:151], v[44:47], v[116:119], v[148:151]\n\t" \
    "ds_read_b64_tr_b16 v[236:237], v2 offset:96\n\t" \
    "ds_read_b64_tr_b16 v[238:239], v2 offset:352\n\t" \
    "s_waitcnt lgkmcnt(8)\n\t" \
    "v_mfma_f32_16x16x32_bf16 v[152:155], v[244:247], v[116:119], v[152:155]\n\t" \
    "ds_read_b64_tr_b16 v[40:41], v2 offset:128\n\t" \
    "ds_read_b64_tr_b16 v[42:43], v2 offset:384\n\t" \
    "v_mfma_f32_16x16x32_bf16 v[156:159], v[28:31], v[116:119], v[156:159]\n\t" \
    "ds_read_b64_tr_b16 v[44:45], v2 offset:160\n\t" \
    "ds_read_b64_tr_b16 v[46:47], v2 offset:416\n\t" \
    "s_waitcnt lgkmcnt(8)\n\t" \
    "v_mfma_f32_16x16x32_bf16 v[128:131], v[32:35], v[120:123], v[128:131]\n\t" \
    "ds_read_b64_tr_b16 v[244:245], v2 offset:192\n\t" \
    "ds_read_b64_tr_b16 v[246:247], v2 offset:448\n\t" \
    "v_mfma_f32_16x16x32_bf16 v[132:135], v[228:231], v[120:123], v[132:135]\n\t" \
    "ds_read_b64_tr_b16 v[28:29], v2 offset:224\n\t" \
    "ds_read_b64_tr_b16 v[30:31], v2 offset:480\n\t" \
    "s_waitcnt lgkmcnt(8)\n\t" \
    "v_mfma_f32_16x16x32_bf16 v[136:139], v[36:39], v[120:123], v[136:139]\n\t" \
    "v_add_u32 v2, 135168, v1\n\t" \
    "ds_read_b64_tr_b16 v[32:33], v2 offset:0\n\t" \
    "ds_read_b64_tr_b16 v[34:35], v2 offset:256\n\t" \
    "v_mfma_f32_16x16x32_bf16 v[140:143], v[236:239], v[120:123], v[140:143]\n\t" \
    "ds_read_b64_tr_b16 v[228:229], v2 offset:32\n\t" \
    "ds_read_b64_tr_b16 v[230:231], v2 offset:288\n\t" \
    "s_waitcnt lgkmcnt(8)\n\t" \
    "v_mfma_f32_16x16x32_bf16 v[144:147], v[40:43], v[120:123], v[144:147]\n\t" \
    "ds_read_b64_tr_b16 v[36:37], v2 offset:64\n\t" \
    "ds_read_b64_tr_b16 v[38:39], v2 offset:320\n\t" \
    "v_mfma_f32_16x16x32_bf16 v[148:151], v[44:47], v[120:123], v[148:151]\n\t" \
    "ds_read_b64_tr_b16 v[236:237], v2 offset:96\n\t" \
    "ds_read_b64_tr_b16 v[238:239], v2 offset:352\n\t" \
    "s_waitcnt lgkmcnt(8)\n\t" \
    "v_mfma_f32_16x16x32_bf16 v[152:155], v[244:247], v[120:123], v[152:155]\n\t" \
    "ds_read_b64_tr_b16 v[40:41], v2 offset:128\n\t" \
    "ds_read_b64_tr_b16 v[42:43], v2 offset:384\n\t" \
    "v_mfma_f32_16x16x32_bf16 v[156:159], v[28:31], v[120:123], v[156:159]\n\t" \
    "ds_read_b64_tr_b16 v[44:45], v2 offset:160\n\t" \
    "ds_read_b64_tr_b16 v[46:47], v2 offset:416\n\t" \
    "s_waitcnt lgkmcnt(8)\n\t" \
    "v_mfma_f32_16x16x32_bf16 v[128:131], v[32:35], v[124:127], v[128:131]\n\t" \
    "ds_read_b64_tr_b16 v[244:245], v2 offset:192\n\t" \
    "ds_read_b64_tr_b16 v[246:247], v2 offset:448\n\t" \
    "ds_read_b128 v[12:15], v8\n\t" \
    "v_mfma_f32_16x16x32_bf16 v[132:135], v[228:231], v[124:127], v[132:135]\n\t" \
    "ds_read_b64_tr_b16 v[28:29], v2 offset:224\n\t" \
    "ds_read_b64_tr_b16 v[30:31], v2 offset:480\n\t" \
    "s_waitcnt lgkmcnt(9)\n\t" \
    "v_mfma_f32_16x16x32_bf16 v[136:139], v[36:39], v[124:127], v[136:139]\n\t" \
    "v_add_u32 v2, 143616, v1\n\t" \
    "ds_read_b64_tr_b16 v[32:33], v2 offset:0\n\t" \
    "ds_read_b64_tr_b16 v[34:35], v2 offset:256\n\t" \
    "v_mfma_f32_16x16x32_bf16 v[140:143], v[236:239], v[124:127], v[140:143]\n\t" \
    "ds_read_b64_tr_b16 v[228:229], v2 offset:32\n\t" \
    "ds_read_b64_tr_b16 v[230:231], v2 offset:288\n\t" \
    "s_waitcnt lgkmcnt(9)\n\t" \
    "v_mfma_f32_16x16x32_bf16 v[144:147], v[40:43], v[124:127], v[144:147]\n\t" \
    "ds_read_b64_tr_b16 v[36:37], v2 offset:64\n\t" \
    "ds_read_b64_tr_b16 v[38:39], v2 offset:320\n\t" \
    "v_mfma_f32_16x16x32_bf16 v[148:151], v[44:47], v[124:127], v[148:151]\n\t" \
    "ds_read_b64_tr_b16 v[236:237], v2 offset:96\n\t" \
    "ds_read_b64_tr_b16 v[238:239], v2 offset:352\n\t" \
    "s_waitcnt lgkmcnt(10)\n\t" \
    "v_min_u32 v14, v14, v15\n\t" \
    "v_readfirstlane_b32 s42, v12\n\t" \
    "v_readfirstlane_b32 s43, v14\n\t" \
    "s_waitcnt lgkmcnt(8)\n\t" \
    "v_mfma_f32_16x16x32_bf16 v[152:155], v[244:247], v[124:127], v[152:155]\n\t" \
    "ds_read_b64_tr_b16 v[40:41], v2 offset:128\n\t" \
    "ds_read_b64_tr_b16 v[42:43], v2 offset:384\n\t" \
    "v_mfma_f32_16x16x32_bf16 v[156:159], v[28:31], v[124:127], v[156:159]\n\t" \
    "ds_read_b64_tr_b16 v[44:45], v2 offset:160\n\t" \
    "ds_read_b64_tr_b16 v[46:47], v2 offset:416\n\t" \
    "s_waitcnt vmcnt(0)\n\t" \
    "v_mov_b32 v25, s49\n\t" \
    "s_mov_b64 exec, 1\n\t" \
    "buffer_store_dword v25, v24, s[36:39], 0 offen sc0 sc1\n\t" \
    "s_mov_b64 exec, -1\n\t" \
    "s_cmp_ge_u32 s41, 32\n\t" \
    "s_cselect_b32 s47, s55, 1\n\t" \
    "s_mov_b32 s55, 0\n\t" \
    "s_cmp_eq_u32 s47, 0\n\t" \
    "s_cbranch_scc0 LfB4_%=\n\t" \
    "s_waitcnt vmcnt(18)\n\t" \
    "LmB4_%=:\n\t" \
    "s_bitcmp1_b32 s44, 0\n\t" \
    "s_cbranch_scc1 Ly14_%=\n\t" \
    "Lx14_%=:\n\t" \
    "ds_read_b32 v25, v8 offset:16\n\t" \
    "s_waitcnt lgkmcnt(0)\n\t" \
    "v_readfirstlane_b32 s47, v25\n\t" \
    "s_and_b32 s47, s47, s54\n\t" \
    "s_cbranch_scc0 Ly14_%=\n\t" \
    "s_bitset1_b32 s44, 0\n\t" \
    "s_add_u32 s47, s53, 16384\n\t" \
    "buffer_load_dwordx4 v[192:195], v11, s[36:39], s47 offen sc0 sc1 nt\n\t" \
    "buffer_load_dwordx4 v[196:199], v11, s[36:39], s47 offen offset:1024 sc0 sc1 nt\n\t" \
    "buffer_load_dwordx4 v[200:203], v11, s[36:39], s47 offen offset:2048 sc0 sc1 nt\n\t" \
    "buffer_load_dwordx4 v[204:207], v11, s[36:39], s47 offen offset:3072 sc0 sc1 nt\n\t" \
    "s_add_u32 s47, s53, 32768\n\t" \
    "buffer_load_dwordx4 v[208:211], v11, s[36:39], s47 offen sc0 sc1 nt\n\t" \
    "buffer_load_dwordx4 v[212:215], v11, s[36:39], s47 offen offset:1024 sc0 sc1 nt\n\t" \
    "buffer_load_dwordx4 v[216:219], v11, s[36:39], s47 offen offset:2048 sc0 sc1 nt\n\t" \
    "buffer_load_dwordx4 v[220:223], v11, s[36:39], s47 offen offset:3072 sc0 sc1 nt\n\t" \
    "Ly14_%=:\n\t" \
    "s_waitcnt lgkmcnt(8)\n\t" \
    "v_mfma_f32_16x16x32_bf16 v[128:131], v[32:35], v[48:51], v[128:131]\n\t" \
    "ds_read_b64_tr_b16 v[244:245], v2 offset:192\n\t" \
    "ds_read_b64_tr_b16 v[246:247], v2 offset:448\n\t" \
    "v_mfma_f32_16x16x32_bf16 v[132:135], v[228:231], v[48:51], v[132:135]\n\t" \
    "ds_read_b64_tr_b16 v[28:29], v2 offset:224\n\t" \
    "ds_read_b64_tr_b16 v[30:31], v2 offset:480\n\t" \
    "s_waitcnt lgkmcnt(8)\n\t" \
    "v_mfma_f32_16x16x32_bf16 v[136:139], v[36:39], v[48:51], v[136:139]\n\t" \
    "s_cmp_gt_u32 s43, 40\n\t" \
    "s_cbranch_scc0 Lwf15_%=\n\t" \
    "Lwb15_%=:\n\t" \
    "v_add_u32 v2, 152064, v1\n\t" \
    "ds_read_b64_tr_b16 v[32:33], v2 offset:0\n\t" \
    "ds_read_b64_tr_b16 v[34:35], v2 offset:256\n\t" \
    "v_mfma_f32_16x16x32_bf16 v[140:143], v[236:239], v[48:51], v[140:143]\n\t" \
    "ds_read_b64_tr_b16 v[228:229], v2 offset:32\n\t" \
    "ds_read_b64_tr_b16 v[230:231], v2 offset:288\n\t" \
    "s_waitcnt lgkmcnt(8)\n\t" \
    "v_mfma_f32_16x16x32_bf16 v[144:147], v[40:43], v[48:51], v[144:147]\n\t" \
    "ds_read_b64_tr_b16 v[36:37], v2 offset:64\n\t" \
    "ds_read_b64_tr_b16 v[38:39], v2 offset:320\n\t" \
    "v_mfma_f32_16x16x32_bf16 v[148:151], v[44:47], v[48:51], v[148:151]\n\t" \
    "ds_read_b64_tr_b16 v[236:237], v2 offset:96\n\t" \
    "ds_read_b64_tr_b16 v[238:239], v2 offset:352\n\t" \
    "s_waitcnt lgkmcnt(8)\n\t" \
    "v_mfma_f32_16x16x32_bf16 v[152:155], v[244:247], v[48:51], v[152:155]\n\t" \
    "ds_read_b64_tr_b16 v[40:41], v2 offset:128\n\t" \
    "ds_read_b64_tr_b16 v[42:43], v2 offset:384\n\t" \
    "v_mfma_f32_16x16x32_bf16 v[156:159], v[28:31], v[48:51], v[156:159]\n\t" \
    "ds_read_b64_tr_b16 v[44:45], v2 offset:160\n\t" \
    "ds_read_b64_tr_b16 v[46:47], v2 offset:416\n\t" \
    "s_waitcnt lgkmcnt(8)\n\t" \
    "v_mfma_f32_16x16x32_bf16 v[128:131], v[32:35], v[52:55], v[128:131]\n\t" \
    "ds_read_b64_tr_b16 v[244:245], v2 offset:192\n\t" \
    "ds_read_b64_tr_b16 v[246:247], v2 offset:448\n\t" \
    "v_mfma_f32_16x16x32_bf16 v[132:135], v[228:231], v[52:55], v[132:135]\n\t" \
    "ds_read_b64_tr_b16 v[28:29], v2 offset:224\n\t" \
    "ds_read_b64_tr_b16 v[30:31], v2 offset:480\n\t" \
    "s_waitcnt lgkmcnt(8)\n\t" \
    "v_mfma_f32_16x16x32_bf16 v[136:139], v[36:39], v[52:55], v[136:139]\n\t" \
    "v_add_u32 v2, 0, v1\n\t" \
    "ds_read_b64_tr_b16 v[32:33], v2 offset:0\n\t" \
    "ds_read_b64_tr_b16 v[34:35], v2 offset:256\n\t" \
    "v_mfma_f32_16x16x32_bf16 v[140:143], v[236:239], v[52:55], v[140:143]\n\t" \
    "ds_read_b64_tr_b16 v[228:229], v2 offset:32\n\t" \
    "ds_read_b64_tr_b16 v[230:231], v2 offset:288\n\t" \
    "s_waitcnt lgkmcnt(8)\n\t" \
    "v_mfma_f32_16x16x32_bf16 v[144:147], v[40:43], v[52:55], v[144:147]\n\t" \
    "ds_read_b64_tr_b16 v[36:37], v2 offset:64\n\t" \
    "ds_read_b64_tr_b16 v[38:39], v2 offset:320\n\t" \
    "v_mfma_f32_16x16x32_bf16 v[148:151], v[44:47], v[52:55], v[148:151]\n\t" \
    "ds_read_b64_tr_b16 v[236:237], v2 offset:96\n\t" \
    "ds_read_b64_tr_b16 v[238:239], v2 offset:352\n\t" \
    "s_waitcnt lgkmcnt(8)\n\t" \
    "v_mfma_f32_16x16x32_bf16 v[152:155], v[244:247], v[52:55], v[152:155]\n\t" \
    "ds_read_b64_tr_b16 v[40:41], v2 offset:128\n\t" \
    "ds_read_b64_tr_b16 v[42:43], v2 offset:384\n\t" \
    "v_mfma_f32_16x16x32_bf16 v[156:159], v[28:31], v[52:55], v[156:159]\n\t" \
    "ds_read_b64_tr_b16 v[44:45], v2 offset:160\n\t" \
    "ds_read_b64_tr_b16 v[46:47], v2 offset:416\n\t" \
    "s_waitcnt lgkmcnt(8)\n\t" \
    "v_mfma_f32_16x16x32_bf16 v[128:131], v[32:35], v[56:59], v[128:131]\n\t" \
    "ds_read_b64_tr_b16 v[244:245], v2 offset:192\n\t" \
    "ds_read_b64_tr_b16 v[246:247], v2 offset:448\n\t" \
    "v_mfma_f32_16x16x32_bf16 v[132:135], v[228:231], v[56:59], v[132:135]\n\t" \
    "ds_read_b64_tr_b16 v[28:29], v2 offset:224\n\t" \
    "ds_read_b64_tr_b16 v[30:31], v2 offset:480\n\t" \
    "s_waitcnt lgkmcnt(8)\n\t" \
    "v_mfma_f32_16x16x32_bf16 v[136:139], v[36:39], v[56:59], v[136:139]\n\t" \
    "v_add_u32 v2, 8448, v1\n\t" \
    "ds_read_b64_tr_b16 v[32:33], v2 offset:0\n\t" \
    "ds_read_b64_tr_b16 v[34:35], v2 offset:256\n\t" \
    "v_mfma_f32_16x16x32_bf16 v[140:143], v[236:239], v[56:59], v[140:143]\n\t" \
    "ds_read_b64_tr_b16 v[228:229], v2 offset:32\n\t" \
    "ds_read_b64_tr_b16 v[230:231], v2 offset:288\n\t" \
    "s_waitcnt lgkmcnt(8)\n\t" \
    "v_mfma_f32_16x16x32_bf16 v[144:147], v[40:43], v[56:59], v[144:147]\n\t" \
    "ds_read_b64_tr_b16 v[36:37], v2 offset:64\n\t" \
    "ds_read_b64_tr_b16 v[38:39], v2 offset:320\n\t" \
    "v_mfma_f32_16x16x32_bf16 v[148:151], v[44:47], v[56:59], v[148:151]\n\t" \
    "ds_read_b64_tr_b16 v[236:237], v2 offset:96\n\t" \
    "ds_read_b64_tr_b16 v[238:239], v2 offset:352\n\t" \
    "s_waitcnt lgkmcnt(8)\n\t" \
    "v_mfma_f32_16x16x32_bf16 v[152:155], v[244:247], v[56:59], v[152:155]\n\t" \
    "ds_read_b64_tr_b16 v[40:41], v2 offset:128\n\t" \
    "ds_read_b64_tr_b16 v[42:43], v2 offset:384\n\t" \
    "v_mfma_f32_16x16x32_bf16 v[156:159], v[28:31], v[56:59], v[156:159]\n\t" \
    "ds_read_b64_tr_b16 v[44:45], v2 offset:160\n\t" \
    "ds_read_b64_tr_b16 v[46:47], v2 offset:416\n\t" \
    "s_waitcnt lgkmcnt(8)\n\t" \
    "v_mfma_f32_16x16x32_bf16 v[128:131], v[32:35], v[60:63], v[128:131]\n\t" \
    "ds_read_b64_tr_b16 v[244:245], v2 offset:192\n\t" \
    "ds_read_b64_tr_b16 v[246:247], v2 offset:448\n\t" \
    "ds_read_b128 v[12:15], v8\n\t" \
    "v_mfma_f32_16x16x32_bf16 v[132:135], v[228:231], v[60:63], v[132:135]\n\t" \
    "ds_read_b64_tr_b16 v[28:29], v2 offset:224\n\t" \
    "ds_read_b64_tr_b16 v[30:31], v2 offset:480\n\t" \
    "s_waitcnt lgkmcnt(9)\n\t" \
    "v_mfma_f32_16x16x32_bf16 v[136:139], v[36:39], v[60:63], v[136:139]\n\t" \
    "v_add_u32 v2, 16896, v1\n\t" \
    "ds_read_b64_tr_b16 v[32:33], v2 offset:0\n\t" \
    "ds_read_b64_tr_b16 v[34:35], v2 offset:256\n\t" \
    "v_mfma_f32_16x16x32_bf16 v[140:143], v[236:239], v[60:63], v[140:143]\n\t" \
    "ds_read_b64_tr_b16 v[228:229], v2 offset:32\n\t" \
    "ds_read_b64_tr_b16 v[230:231], v2 offset:288\n\t" \
    "s_waitcnt lgkmcnt(9)\n\t" \
    "v_mfma_f32_16x16x32_bf16 v[144:147], v[40:43], v[60:63], v[144:147]\n\t" \
    "ds_read_b64_tr_b16 v[36:37], v2 offset:64\n\t" \
    "ds_read_b64_tr_b16 v[38:39], v2 offset:320\n\t" \
    "v_mfma_f32_16x16x32_bf16 v[148:151], v[44:47], v[60:63], v[148:151]\n\t" \
    "ds_read_b64_tr_b16 v[236:237], v2 offset:96\n\t" \
    "ds_read_b64_tr_b16 v[238:239], v2 offset:352\n\t" \
    "s_waitcnt lgkmcnt(10)\n\t" \
    "v_min_u32 v14, v14, v15\n\t" \
    "v_readfirstlane_b32 s42, v12\n\t" \
    "v_readfirstlane_b32 s43, v14\n\t" \
    "s_waitcnt lgkmcnt(8)\n\t" \
    "v_mfma_f32_16x16x32_bf16 v[152:155], v[244:247], v[60:63], v[152:155]\n\t" \
    "ds_read_b64_tr_b16 v[40:41], v2 offset:128\n\t" \
    "ds_read_b64_tr_b16 v[42:43], v2 offset:384\n\t" \
    "v_mfma_f32_16x16x32_bf16 v[156:159], v[28:31], v[60:63], v[156:159]\n\t" \
    "ds_read_b64_tr_b16 v[44:45], v2 offset:160\n\t" \
    "ds_read_b64_tr_b16 v[46:47], v2 offset:416\n\t" \
    "s_bitcmp1_b32 s44, 0\n\t" \
    "s_cbranch_scc0 Lna_%=\n\t" \
    "s_waitcnt vmcnt(0)\n\t" \
    "v_mfma_f32_16x16x32_bf16 v[128:131], v[16:19], v[192:195], v[128:131]\n\t" \
    "v_mfma_f32_16x16x32_bf16 v[132:135], v[20:23], v[192:195], v[132:135]\n\t" \
    "v_mfma_f32_16x16x32_bf16 v[136:139], v[16:19], v[196:199], v[136:139]\n\t" \
    "v_mfma_f32_16x16x32_bf16 v[140:143], v[20:23], v[196:199], v[140:143]\n\t" \
    "v_mfma_f32_16x16x32_bf16 v[144:147], v[16:19], v[200:203], v[144:147]\n\t" \
    "v_mfma_f32_16x16x32_bf16 v[148:151], v[20:23], v[200:203], v[148:151]\n\t" \
    "v_mfma_f32_16x16x32_bf16 v[152:155], v[16:19], v[204:207], v[152:155]\n\t" \
    "v_mfma_f32_16x16x32_bf16 v[156:159], v[20:23], v[204:207], v[156:159]\n\t" \
    "v_mfma_f32_16x16x32_bf16 v[128:131], v[16:19], v[208:211], v[128:131]\n\t" \
    "v_mfma_f32_16x16x32_bf16 v[132:135], v[20:23], v[208:211], v[132:135]\n\t" \
    "v_mfma_f32_16x16x32_bf16 v[136:139], v[16:19], v[212:215], v[136:139]\n\t" \
    "v_mfma_f32_16x16x32_bf16 v[140:143], v[20:23], v[212:215], v[140:143]\n\t" \
    "v_mfma_f32_16x16x32_bf16 v[144:147], v[16:19], v[216:219], v[144:147]\n\t" \
    "v_mfma_f32_16x16x32_bf16 v[148:151], v[20:23], v[216:219], v[148:151]\n\t" \
    "v_mfma_f32_16x16x32_bf16 v[152:155], v[16:19], v[220:223], v[152:155]\n\t" \
    "v_mfma_f32_16x16x32_bf16 v[156:159], v[20:23], v[220:223], v[156:159]\n\t" \
    "s_bitset1_b32 s44, 2\n\t" \
    "Lna_%=:\n\t" \
    "s_cmp_ge_u32 s41, 32\n\t" \
    "s_cselect_b32 s47, s55, 1\n\t" \
    "s_mov_b32 s55, 0\n\t" \
    "s_cmp_eq_u32 s47, 0\n\t" \
    "s_cbranch_scc0 LfB5_%=\n\t" \
    "s_waitcnt vmcnt(10)\n\t" \
    "LmB5_%=:\n\t" \
    "s_bitcmp1_b32 s44, 0\n\t" \
    "s_cbranch_scc1 Ly16_%=\n\t" \
    "Lx16_%=:\n\t" \
    "ds_read_b32 v25, v8 offset:16\n\t" \
    "s_waitcnt lgkmcnt(0)\n\t" \
    "v_readfirstlane_b32 s47, v25\n\t" \
    "s_and_b32 s47, s47, s54\n\t" \
    "s_cbranch_scc0 Ly16_%=\n\t" \
    "s_bitset1_b32 s44, 0\n\t" \
    "s_add_u32 s47, s53, 16384\n\t" \
    "buffer_load_dwordx4 v[192:195], v11, s[36:39], s47 offen sc0 sc1 nt\n\t" \
    "buffer_load_dwordx4 v[196:199], v11, s[36:39], s47 offen offset:1024 sc0 sc1 nt\n\t" \
    "buffer_load_dwordx4 v[200:203], v11, s[36:39], s47 offen offset:2048 sc0 sc1 nt\n\t" \
    "buffer_load_dwordx4 v[204:207], v11, s[36:39], s47 offen offset:3072 sc0 sc1 nt\n\t" \
    "s_add_u32 s47, s53, 32768\n\t" \
    "buffer_load_dwordx4 v[208:211], v11, s[36:39], s47 offen sc0 sc1 nt\n\t" \
    "buffer_load_dwordx4 v[212:215], v11, s[36:39], s47 offen offset:1024 sc0 sc1 nt\n\t" \
    "buffer_load_dwordx4 v[216:219], v11, s[36:39], s47 offen offset:2048 sc0 sc1 nt\n\t" \
    "buffer_load_dwordx4 v[220:223], v11, s[36:39], s47 offen offset:3072 sc0 sc1 nt\n\t" \
    "Ly16_%=:\n\t" \
    "s_bitcmp1_b32 s44, 1\n\t" \
    "s_cbranch_scc1 Ly17_%=\n\t" \
    "Lx17_%=:\n\t" \
    "ds_read_b32 v25, v8 offset:20\n\t" \
    "s_waitcnt lgkmcnt(0)\n\t" \
    "v_readfirstlane_b32 s47, v25\n\t" \
    "s_and_b32 s47, s47, s54\n\t" \
    "s_cbranch_scc0 Ly17_%=\n\t" \
    "s_bitset1_b32 s44, 1\n\t" \
    "s_add_u32 s47, s53, 0\n\t" \
    "buffer_load_dwordx4 v[160:163], v11, s[36:39], s47 offen sc0 sc1 nt\n\t" \
    "buffer_load_dwordx4 v[164:167], v11, s[36:39], s47 offen offset:1024 sc0 sc1 nt\n\t" \
    "buffer_load_dwordx4 v[168:171], v11, s[36:39], s47 offen offset:2048 sc0 sc1 nt\n\t" \
    "buffer_load_dwordx4 v[172:175], v11, s[36:39], s47 offen offset:3072 sc0 sc1 nt\n\t" \
    "Ly17_%=:\n\t" \
    "s_waitcnt lgkmcnt(8)\n\t" \
    "v_mfma_f32_16x16x32_bf16 v[128:131], v[32:35], v[64:67], v[128:131]\n\t" \
    "ds_read_b64_tr_b16 v[244:245], v2 offset:192\n\t" \
    "ds_read_b64_tr_b16 v[246:247], v2 offset:448\n\t" \
    "v_mfma_f32_16x16x32_bf16 v[132:135], v[228:231], v[64:67], v[132:135]\n\t" \
    "ds_read_b64_tr_b16 v[28:29], v2 offset:224\n\t" \
    "ds_read_b64_tr_b16 v[30:31], v2 offset:480\n\t" \
    "s_waitcnt lgkmcnt(8)\n\t" \
    "v_mfma_f32_16x16x32_bf16 v[136:139], v[36:39], v[64:67], v[136:139]\n\t" \
    "s_cmp_gt_u32 s43, 43\n\t" \
    "s_cbranch_scc0 Lwf18_%=\n\t" \
    "Lwb18_%=:\n\t" \
    "v_add_u32 v2, 25344, v1\n\t" \
    "ds_read_b64_tr_b16 v[32:33], v2 offset:0\n\t" \
    "ds_read_b64_tr_b16 v[34:35], v2 offset:256\n\t" \
    "v_mfma_f32_16x16x32_bf16 v[140:143], v[236:239], v[64:67], v[140:143]\n\t" \
    "ds_read_b64_tr_b16 v[228:229], v2 offset:32\n\t" \
    "ds_read_b64_tr_b16 v[230:231], v2 offset:288\n\t" \
    "s_waitcnt lgkmcnt(8)\n\t" \
    "v_mfma_f32_16x16x32_bf16 v[144:147], v[40:43], v[64:67], v[144:147]\n\t" \
    "ds_read_b64_tr_b16 v[36:37], v2 offset:64\n\t" \
    "ds_read_b64_tr_b16 v[38:39], v2 offset:320\n\t" \
    "v_mfma_f32_16x16x32_bf16 v[148:151], v[44:47], v[64:67], v[148:151]\n\t" \
    "ds_read_b64_tr_b16 v[236:237], v2 offset:96\n\t" \
    "ds_read_b64_tr_b16 v[238:239], v2 offset:352\n\t" \
    "s_waitcnt lgkmcnt(8)\n\t" \
    "v_mfma_f32_16x16x32_bf16 v[152:155], v[244:247], v[64:67], v[152:155]\n\t" \
    "ds_read_b64_tr_b16 v[40:41], v2 offset:128\n\t" \
    "ds_read_b64_tr_b16 v[42:43], v2 offset:384\n\t" \
    "v_mfma_f32_16x16x32_bf16 v[156:159], v[28:31], v[64:67], v[156:159]\n\t" \
    "ds_read_b64_tr_b16 v[44:45], v2 offset:160\n\t" \
    "ds_read_b64_tr_b16 v[46:47], v2 offset:416\n\t" \
    "s_waitcnt lgkmcnt(8)\n\t" \
    "v_mfma_f32_16x16x32_bf16 v[128:131], v[32:35], v[68:71], v[128:131]\n\t" \
    "ds_read_b64_tr_b16 v[244:245], v2 offset:192\n\t" \
    "ds_read_b64_tr_b16 v[246:247], v2 offset:448\n\t" \
    "v_mfma_f32_16x16x32_bf16 v[132:135], v[228:231], v[68:71], v[132:135]\n\t" \
    "ds_read_b64_tr_b16 v[28:29], v2 offset:224\n\t" \
    "ds_read_b64_tr_b16 v[30:31], v2 offset:480\n\t" \
    "s_waitcnt lgkmcnt(8)\n\t" \
    "v_mfma_f32_16x16x32_bf16 v[136:139], v[36:39], v[68:71], v[136:139]\n\t" \
    "v_add_u32 v2, 33792, v1\n\t" \
    "ds_read_b64_tr_b16 v[32:33], v2 offset:0\n\t" \
    "ds_read_b64_tr_b16 v[34:35], v2 offset:256\n\t" \
    "v_mfma_f32_16x16x32_bf16 v[140:143], v[236:239], v[68:71], v[140:143]\n\t" \
    "ds_read_b64_tr_b16 v[228:229], v2 offset:32\n\t" \
    "ds_read_b64_tr_b16 v[230:231], v2 offset:288\n\t" \
    "s_waitcnt lgkmcnt(8)\n\t" \
    "v_mfma_f32_16x16x32_bf16 v[144:147], v[40:43], v[68:71], v[144:147]\n\t" \
    "ds_read_b64_tr_b16 v[36:37], v2 offset:64\n\t" \
    "ds_read_b64_tr_b16 v[38:39], v2 offset:320\n\t" \
    "v_mfma_f32_16x16x32_bf16 v[148:151], v[44:47], v[68:71], v[148:151]\n\t" \
    "ds_read_b64_tr_b16 v[236:237], v2 offset:96\n\t" \
    "ds_read_b64_tr_b16 v[238:239], v2 offset:352\n\t" \
    "s_waitcnt lgkmcnt(8)\n\t" \
    "v_mfma_f32_16x16x32_bf16 v[152:155], v[244:247], v[68:71], v[152:155]\n\t" \
    "ds_read_b64_tr_b16 v[40:41], v2 offset:128\n\t" \
    "ds_read_b64_tr_b16 v[42:43], v2 offset:384\n\t" \
    "v_mfma_f32_16x16x32_bf16 v[156:159], v[28:31], v[68:71], v[156:159]\n\t" \
    "ds_read_b64_tr_b16 v[44:45], v2 offset:160\n\t" \
    "ds_read_b64_tr_b16 v[46:47], v2 offset:416\n\t" \
    "s_waitcnt lgkmcnt(8)\n\t" \
    "v_mfma_f32_16x16x32_bf16 v[128:131], v[32:35], v[72:75], v[128:131]\n\t" \
    "ds_read_b64_tr_b16 v[244:245], v2 offset:192\n\t" \
    "ds_read_b64_tr_b16 v[246:247], v2 offset:448\n\t" \
    "v_mfma_f32_16x16x32_bf16 v[132:135], v[228:231], v[72:75], v[132:135]\n\t" \
    "ds_read_b64_tr_b16 v[28:29], v2 offset:224\n\t" \
    "ds_read_b64_tr_b16 v[30:31], v2 offset:480\n\t" \
    "s_waitcnt lgkmcnt(8)\n\t" \
    "v_mfma_f32_16x16x32_bf16 v[136:139], v[36:39], v[72:75], v[136:139]\n\t" \
    "v_add_u32 v2, 42240, v1\n\t" \
    "ds_read_b64_tr_b16 v[32:33], v2 offset:0\n\t" \
    "ds_read_b64_tr_b16 v[34:35], v2 offset:256\n\t" \
    "v_mfma_f32_16x16x32_bf16 v[140:143], v[236:239], v[72:75], v[140:143]\n\t" \
    "ds_read_b64_tr_b16 v[228:229], v2 offset:32\n\t" \
    "ds_read_b64_tr_b16 v[230:231], v2 offset:288\n\t" \
    "s_waitcnt lgkmcnt(8)\n\t" \
    "v_mfma_f32_16x16x32_bf16 v[144:147], v[40:43], v[72:75], v[144:147]\n\t" \
    "ds_read_b64_tr_b16 v[36:37], v2 offset:64\n\t" \
    "ds_read_b64_tr_b16 v[38:39], v2 offset:320\n\t" \
    "v_mfma_f32_16x16x32_bf16 v[148:151], v[44:47], v[72:75], v[148:151]\n\t" \
    "ds_read_b64_tr_b16 v[236:237], v2 offset:96\n\t" \
    "ds_read_b64_tr_b16 v[238:239], v2 offset:352\n\t" \
    "s_waitcnt lgkmcnt(8)\n\t" \
    "v_mfma_f32_16x16x32_bf16 v[152:155], v[244:247], v[72:75], v[152:155]\n\t" \
    "ds_read_b64_tr_b16 v[40:41], v2 offset:128\n\t" \
    "ds_read_b64_tr_b16 v[42:43], v2 offset:384\n\t" \
    "v_mfma_f32_16x16x32_bf16 v[156:159], v[28:31], v[72:75], v[156:159]\n\t" \
    "ds_read_b64_tr_b16 v[44:45], v2 offset:160\n\t" \
    "ds_read_b64_tr_b16 v[46:47], v2 offset:416\n\t" \
    "s_waitcnt lgkmcnt(8)\n\t" \
    "v_mfma_f32_16x16x32_bf16 v[128:131], v[32:35], v[76:79], v[128:131]\n\t" \
    "ds_read_b64_tr_b16 v[244:245], v2 offset:192\n\t" \
    "ds_read_b64_tr_b16 v[246:247], v2 offset:448\n\t" \
    "v_mfma_f32_16x16x32_bf16 v[132:135], v[228:231], v[76:79], v[132:135]\n\t" \
    "ds_read_b64_tr_b16 v[28:29], v2 offset:224\n\t" \
    "ds_read_b64_tr_b16 v[30:31], v2 offset:480\n\t" \
    "s_waitcnt lgkmcnt(8)\n\t" \
    "v_mfma_f32_16x16x32_bf16 v[136:139], v[36:39], v[76:79], v[136:139]\n\t" \
    "v_mfma_f32_16x16x32_bf16 v[140:143], v[236:239], v[76:79], v[140:143]\n\t" \
    "s_waitcnt lgkmcnt(6)\n\t" \
    "v_mfma_f32_16x16x32_bf16 v[144:147], v[40:43], v[76:79], v[144:147]\n\t" \
    "s_waitcnt lgkmcnt(4)\n\t" \
    "v_mfma_f32_16x16x32_bf16 v[148:151], v[44:47], v[76:79], v[148:151]\n\t" \
    "s_waitcnt lgkmcnt(2)\n\t" \
    "v_mfma_f32_16x16x32_bf16 v[152:155], v[244:247], v[76:79], v[152:155]\n\t" \
    "s_waitcnt lgkmcnt(0)\n\t" \
    "v_mfma_f32_16x16x32_bf16 v[156:159], v[28:31], v[76:79], v[156:159]\n\t" \
    "s_bitcmp1_b32 s44, 2\n\t" \
    "s_cbranch_scc1 Lnb_%=\n\t" \
    "s_bitcmp1_b32 s44, 0\n\t" \
    "s_cbranch_scc1 Ly19_%=\n\t" \
    "Lx19_%=:\n\t" \
    "ds_read_b32 v25, v8 offset:16\n\t" \
    "s_waitcnt lgkmcnt(0)\n\t" \
    "v_readfirstlane_b32 s47, v25\n\t" \
    "s_and_b32 s47, s47, s54\n\t" \
    "s_cbranch_scc0 Lx19_%=\n\t" \
    "s_bitset1_b32 s44, 0\n\t" \
    "s_add_u32 s47, s53, 16384\n\t" \
    "buffer_load_dwordx4 v[192:195], v11, s[36:39], s47 offen sc0 sc1 nt\n\t" \
    "buffer_load_dwordx4 v[196:199], v11, s[36:39], s47 offen offset:1024 sc0 sc1 nt\n\t" \
    "buffer_load_dwordx4 v[200:203], v11, s[36:39], s47 offen offset:2048 sc0 sc1 nt\n\t" \
    "buffer_load_dwordx4 v[204:207], v11, s[36:39], s47 offen offset:3072 sc0 sc1 nt\n\t" \
    "s_add_u32 s47, s53, 32768\n\t" \
    "buffer_load_dwordx4 v[208:211], v11, s[36:39], s47 offen sc0 sc1 nt\n\t" \
    "buffer_load_dwordx4 v[212:215], v11, s[36:39], s47 offen offset:1024 sc0 sc1 nt\n\t" \
    "buffer_load_dwordx4 v[216:219], v11, s[36:39], s47 offen offset:2048 sc0 sc1 nt\n\t" \
    "buffer_load_dwordx4 v[220:223], v11, s[36:39], s47 offen offset:3072 sc0 sc1 nt\n\t" \
    "Ly19_%=:\n\t" \
    "s_bitcmp1_b32 s44, 1\n\t" \
    "s_cbranch_scc1 Ly20_%=\n\t" \
    "Lx20_%=:\n\t" \
    "ds_read_b32 v25, v8 offset:20\n\t" \
    "s_waitcnt lgkmcnt(0)\n\t" \
    "v_readfirstlane_b32 s47, v25\n\t" \
    "s_and_b32 s47, s47, s54\n\t" \
    "s_cbranch_scc0 Lx20_%=\n\t" \
    "s_bitset1_b32 s44, 1\n\t" \
    "s_add_u32 s47, s53, 0\n\t" \
    "buffer_load_dwordx4 v[160:163], v11, s[36:39], s47 offen sc0 sc1 nt\n\t" \
    "buffer_load_dwordx4 v[164:167], v11, s[36:39], s47 offen offset:1024 sc0 sc1 nt\n\t" \
    "buffer_load_dwordx4 v[168:171], v11, s[36:39], s47 offen offset:2048 sc0 sc1 nt\n\t" \
    "buffer_load_dwordx4 v[172:175], v11, s[36:39], s47 offen offset:3072 sc0 sc1 nt\n\t" \
    "Ly20_%=:\n\t" \
    "s_waitcnt vmcnt(0)\n\t" \
    "v_mfma_f32_16x16x32_bf16 v[128:131], v[16:19], v[192:195], v[128:131]\n\t" \
    "v_mfma_f32_16x16x32_bf16 v[132:135], v[20:23], v[192:195], v[132:135]\n\t" \
    "v_mfma_f32_16x16x32_bf16 v[136:139], v[16:19], v[196:199], v[136:139]\n\t" \
    "v_mfma_f32_16x16x32_bf16 v[140:143], v[20:23], v[196:199], v[140:143]\n\t" \
    "v_mfma_f32_16x16x32_bf16 v[144:147], v[16:19], v[200:203], v[144:147]\n\t" \
    "v_mfma_f32_16x16x32_bf16 v[148:151], v[20:23], v[200:203], v[148:151]\n\t" \
    "v_mfma_f32_16x16x32_bf16 v[152:155], v[16:19], v[204:207], v[152:155]\n\t" \
    "v_mfma_f32_16x16x32_bf16 v[156:159], v[20:23], v[204:207], v[156:159]\n\t" \
    "v_mfma_f32_16x16x32_bf16 v[128:131], v[16:19], v[208:211], v[128:131]\n\t" \
    "v_mfma_f32_16x16x32_bf16 v[132:135], v[20:23], v[208:211], v[132:135]\n\t" \
    "v_mfma_f32_16x16x32_bf16 v[136:139], v[16:19], v[212:215], v[136:139]\n\t" \
    "v_mfma_f32_16x16x32_bf16 v[140:143], v[20:23], v[212:215], v[140:143]\n\t" \
    "v_mfma_f32_16x16x32_bf16 v[144:147], v[16:19], v[216:219], v[144:147]\n\t" \
    "v_mfma_f32_16x16x32_bf16 v[148:151], v[20:23], v[216:219], v[148:151]\n\t" \
    "v_mfma_f32_16x16x32_bf16 v[152:155], v[16:19], v[220:223], v[152:155]\n\t" \
    "v_mfma_f32_16x16x32_bf16 v[156:159], v[20:23], v[220:223], v[156:159]\n\t" \
    "Lnb_%=:\n\t" \
    "s_bitcmp1_b32 s44, 1\n\t" \
    "s_cbranch_scc1 Ly21_%=\n\t" \
    "Lx21_%=:\n\t" \
    "ds_read_b32 v25, v8 offset:20\n\t" \
    "s_waitcnt lgkmcnt(0)\n\t" \
    "v_readfirstlane_b32 s47, v25\n\t" \
    "s_and_b32 s47, s47, s54\n\t" \
    "s_cbranch_scc0 Lx21_%=\n\t" \
    "s_bitset1_b32 s44, 1\n\t" \
    "s_add_u32 s47, s53, 0\n\t" \
    "buffer_load_dwordx4 v[160:163], v11, s[36:39], s47 offen sc0 sc1 nt\n\t" \
    "buffer_load_dwordx4 v[164:167], v11, s[36:39], s47 offen offset:1024 sc0 sc1 nt\n\t" \
    "buffer_load_dwordx4 v[168:171], v11, s[36:39], s47 offen offset:2048 sc0 sc1 nt\n\t" \
    "buffer_load_dwordx4 v[172:175], v11, s[36:39], s47 offen offset:3072 sc0 sc1 nt\n\t" \
    "Ly21_%=:\n\t" \
    "s_waitcnt vmcnt(0)\n\t" \
    "v_mfma_f32_16x16x32_bf16 v[128:131], v[16:19], v[160:163], v[128:131]\n\t" \
    "v_mfma_f32_16x16x32_bf16 v[132:135], v[20:23], v[160:163], v[132:135]\n\t" \
    "v_mfma_f32_16x16x32_bf16 v[136:139], v[16:19], v[164:167], v[136:139]\n\t" \
    "v_mfma_f32_16x16x32_bf16 v[140:143], v[20:23], v[164:167], v[140:143]\n\t" \
    "v_mfma_f32_16x16x32_bf16 v[144:147], v[16:19], v[168:171], v[144:147]\n\t" \
    "v_mfma_f32_16x16x32_bf16 v[148:151], v[20:23], v[168:171], v[148:151]\n\t" \
    "v_mfma_f32_16x16x32_bf16 v[152:155], v[16:19], v[172:175], v[152:155]\n\t" \
    "v_mfma_f32_16x16x32_bf16 v[156:159], v[20:23], v[172:175], v[156:159]\n\t" \
    "s_nop 0\n\t" \
    "v_max3_f32 v26, v128, v129, 0\n\t" \
    "v_max3_f32 v26, v130, v131, v26\n\t" \
    "v_max3_f32 v26, v132, v133, v26\n\t" \
    "v_max3_f32 v26, v134, v135, v26\n\t" \
    "v_max3_f32 v26, v136, v137, v26\n\t" \
    "v_max3_f32 v26, v138, v139, v26\n\t" \
    "v_max3_f32 v26, v140, v141, v26\n\t" \
    "v_max3_f32 v26, v142, v143, v26\n\t" \
    "v_max3_f32 v26, v144, v145, v26\n\t" \
    "v_max3_f32 v26, v146, v147, v26\n\t" \
    "v_max3_f32 v26, v148, v149, v26\n\t" \
    "v_max3_f32 v26, v150, v151, v26\n\t" \
    "s_nop 1\n\t" \
    "v_max3_f32 v26, v152, v153, v26\n\t" \
    "v_max3_f32 v26, v154, v155, v26\n\t" \
    "v_max3_f32 v26, v156, v157, v26\n\t" \
    "v_max3_f32 v26, v158, v159, v26\n\t" \
    "s_branch LEND_%=\n\t" \
    "Lwf1_%=:\n\t" \
    "ds_read_b128 v[12:15], v8\n\t" \
    "s_waitcnt lgkmcnt(0)\n\t" \
    "v_min_u32 v14, v14, v15\n\t" \
    "s_nop 0\n\t" \
    "v_readfirstlane_b32 s43, v14\n\t" \
    "s_cmp_gt_u32 s43, s56\n\t" \
    "s_cbranch_scc1 Lwb1_%=\n\t" \
    "s_sleep 1\n\t" \
    "s_branch Lwf1_%=\n\t" \
    "Lwf4_%=:\n\t" \
    "ds_read_b128 v[12:15], v8\n\t" \
    "s_waitcnt lgkmcnt(0)\n\t" \
    "v_min_u32 v14, v14, v15\n\t" \
    "s_nop 0\n\t" \
    "v_readfirstlane_b32 s43, v14\n\t" \
    "s_cmp_gt_u32 s43, s56\n\t" \
    "s_cbranch_scc1 Lwb4_%=\n\t" \
    "s_sleep 1\n\t" \
    "s_branch Lwf4_%=\n\t" \
    "Lwf5_%=:\n\t" \
    "ds_read_b128 v[12:15], v8\n\t" \
    "s_waitcnt lgkmcnt(0)\n\t" \
    "v_min_u32 v14, v14, v15\n\t" \
    "s_nop 0\n\t" \
    "v_readfirstlane_b32 s43, v14\n\t" \
    "s_cmp_gt_u32 s43, s56\n\t" \
    "s_cbranch_scc1 Lwb5_%=\n\t" \
    "s_sleep 1\n\t" \
    "s_branch Lwf5_%=\n\t" \
    "Lrp1_%=:\n\t" \
    "s_sub_u32 s48, s40, 4\n\t" \
    "s_mov_b32 s57, 1\n\t" \
    "s_cmp_eq_u32 s41, s48\n\t" \
    "s_cbranch_scc0 Lr11_%=\n\t" \
    "Lr01_%=:\n\t" \
    "ds_read_b128 v[12:15], v8\n\t" \
    "s_waitcnt lgkmcnt(0)\n\t" \
    "v_min_u32 v14, v14, v15\n\t" \
    "v_readfirstlane_b32 s42, v12\n\t" \
    "s_cmp_gt_u32 s42, s41\n\t" \
    "s_cbranch_scc1 Li1_0_%=\n\t" \
    "s_sleep 1\n\t" \
    "s_branch Lr01_%=\n\t" \
    "Lr11_%=:\n\t" \
    "s_cmp_gt_u32 s42, s41\n\t" \
    "s_cbranch_scc0 Lbk1_%=\n\t" \
    "s_waitcnt vmcnt(12)\n\t" \
    "s_mov_b32 s57, 0\n\t" \
    "s_add_u32 s48, s48, 1\n\t" \
    "s_cmp_eq_u32 s41, s48\n\t" \
    "s_cbranch_scc1 Li1_1_%=\n\t" \
    "s_add_u32 s48, s48, 1\n\t" \
    "s_cmp_eq_u32 s41, s48\n\t" \
    "s_cbranch_scc1 Li1_2_%=\n\t" \
    "s_add_u32 s48, s48, 1\n\t" \
    "s_branch Li1_3_%=\n\t" \
    "Li1_0_%=:\n\t" \
    "v_readlane_b32 s60, v4, s41\n\t" \
    "v_readlane_b32 s61, v5, s41\n\t" \
    "v_readlane_b32 s62, v6, s41\n\t" \
    "v_readlane_b32 s63, v7, s41\n\t" \
    "s_cmp_eq_u32 s41, s48\n\t" \
    "s_cselect_b32 s47, s42, 0\n\t" \
    "s_cmp_gt_u32 s47, s41\n\t" \
    "s_cselect_b32 s66, s46, 0\n\t" \
    "s_addc_u32 s41, s41, 0\n\t" \
    "buffer_load_dwordx4 v[64:67], v3, s[64:67], s60 offen sc0 sc1\n\t" \
    "buffer_load_dwordx4 v[68:71], v3, s[64:67], s61 offen sc0 sc1\n\t" \
    "buffer_load_dwordx4 v[72:75], v3, s[64:67], s62 offen sc0 sc1\n\t" \
    "buffer_load_dwordx4 v[76:79], v3, s[64:67], s63 offen sc0 sc1\n\t" \
    "s_add_u32 s48, s48, 1\n\t" \
    "Li1_1_%=:\n\t" \
    "v_readlane_b32 s60, v4, s41\n\t" \
    "v_readlane_b32 s61, v5, s41\n\t" \
    "v_readlane_b32 s62, v6, s41\n\t" \
    "v_readlane_b32 s63, v7, s41\n\t" \
    "s_cmp_eq_u32 s41, s48\n\t" \
    "s_cselect_b32 s47, s42, 0\n\t" \
    "s_cmp_gt_u32 s47, s41\n\t" \
    "s_cselect_b32 s66, s46, 0\n\t" \
    "s_addc_u32 s41, s41, 0\n\t" \
    "buffer_load_dwordx4 v[80:83], v3, s[64:67], s60 offen sc0 sc1\n\t" \
    "buffer_load_dwordx4 v[84:87], v3, s[64:67], s61 offen sc0 sc1\n\t" \
    "buffer_load_dwordx4 v[88:91], v3, s[64:67], s62 offen sc0 sc1\n\t" \
    "buffer_load_dwordx4 v[92:95], v3, s[64:67], s63 offen sc0 sc1\n\t" \
    "s_add_u32 s48, s48, 1\n\t" \
    "Li1_2_%=:\n\t" \
    "v_readlane_b32 s60, v4, s41\n\t" \
    "v_readlane_b32 s61, v5, s41\n\t" \
    "v_readlane_b32 s62, v6, s41\n\t" \
    "v_readlane_b32 s63, v7, s41\n\t" \
    "s_cmp_eq_u32 s41, s48\n\t" \
    "s_cselect_b32 s47, s42, 0\n\t" \
    "s_cmp_gt_u32 s47, s41\n\t" \
    "s_cselect_b32 s66, s46, 0\n\t" \
    "s_addc_u32 s41, s41, 0\n\t" \
    "buffer_load_dwordx4 v[96:99], v3, s[64:67], s60 offen sc0 sc1\n\t" \
    "buffer_load_dwordx4 v[100:103], v3, s[64:67], s61 offen sc0 sc1\n\t" \
    "buffer_load_dwordx4 v[104:107], v3, s[64:67], s62 offen sc0 sc1\n\t" \
    "buffer_load_dwordx4 v[108:111], v3, s[64:67], s63 offen sc0 sc1\n\t" \
    "s_add_u32 s48, s48, 1\n\t" \
    "Li1_3_%=:\n\t" \
    "v_readlane_b32 s60, v4, s41\n\t" \
    "v_readlane_b32 s61, v5, s41\n\t" \
    "v_readlane_b32 s62, v6, s41\n\t" \
    "v_readlane_b32 s63, v7, s41\n\t" \
    "s_cmp_eq_u32 s41, s48\n\t" \
    "s_cselect_b32 s47, s42, 0\n\t" \
    "s_cmp_gt_u32 s47, s41\n\t" \
    "s_cselect_b32 s66, s46, 0\n\t" \
    "s_addc_u32 s41, s41, 0\n\t" \
    "buffer_load_dwordx4 v[112:115], v3, s[64:67], s60 offen sc0 sc1\n\t" \
    "buffer_load_dwordx4 v[116:119], v3, s[64:67], s61 offen sc0 sc1\n\t" \
    "buffer_load_dwordx4 v[120:123], v3, s[64:67], s62 offen sc0 sc1\n\t" \
    "buffer_load_dwordx4 v[124:127], v3, s[64:67], s63 offen sc0 sc1\n\t" \
    "s_add_u32 s48, s48, 1\n\t" \
    "v_readlane_b32 s60, v4, s41\n\t" \
    "v_readlane_b32 s61, v5, s41\n\t" \
    "v_readlane_b32 s62, v6, s41\n\t" \
    "v_readlane_b32 s63, v7, s41\n\t" \
    "s_cmp_eq_u32 s57, 0\n\t" \
    "s_cbranch_scc1 Lbj1_%=\n\t" \
    "s_branch Lbk1_%=\n\t" \
    "Lwf6_%=:\n\t" \
    "ds_read_b128 v[12:15], v8\n\t" \
    "s_waitcnt lgkmcnt(0)\n\t" \
    "v_min_u32 v14, v14, v15\n\t" \
    "s_nop 0\n\t" \
    "v_readfirstlane_b32 s43, v14\n\t" \
    "s_cmp_gt_u32 s43, s56\n\t" \
    "s_cbranch_scc1 Lwb6_%=\n\t" \
    "s_sleep 1\n\t" \
    "s_branch Lwf6_%=\n\t" \
    "Lrp2_%=:\n\t" \
    "s_sub_u32 s48, s40, 4\n\t" \
    "s_mov_b32 s57, 1\n\t" \
    "s_cmp_eq_u32 s41, s48\n\t" \
    "s_cbranch_scc0 Lr12_%=\n\t" \
    "Lr02_%=:\n\t" \
    "ds_read_b128 v[12:15], v8\n\t" \
    "s_waitcnt lgkmcnt(0)\n\t" \
    "v_min_u32 v14, v14, v15\n\t" \
    "v_readfirstlane_b32 s42, v12\n\t" \
    "s_cmp_gt_u32 s42, s41\n\t" \
    "s_cbranch_scc1 Li2_0_%=\n\t" \
    "s_sleep 1\n\t" \
    "s_branch Lr02_%=\n\t" \
    "Lr12_%=:\n\t" \
    "s_cmp_gt_u32 s42, s41\n\t" \
    "s_cbranch_scc0 Lbk2_%=\n\t" \
    "s_waitcnt vmcnt(12)\n\t" \
    "s_mov_b32 s57, 0\n\t" \
    "s_add_u32 s48, s48, 1\n\t" \
    "s_cmp_eq_u32 s41, s48\n\t" \
    "s_cbranch_scc1 Li2_1_%=\n\t" \
    "s_add_u32 s48, s48, 1\n\t" \
    "s_cmp_eq_u32 s41, s48\n\t" \
    "s_cbranch_scc1 Li2_2_%=\n\t" \
    "s_add_u32 s48, s48, 1\n\t" \
    "s_branch Li2_3_%=\n\t" \
    "Li2_0_%=:\n\t" \
    "v_readlane_b32 s60, v4, s41\n\t" \
    "v_readlane_b32 s61, v5, s41\n\t" \
    "v_readlane_b32 s62, v6, s41\n\t" \
    "v_readlane_b32 s63, v7, s41\n\t" \
    "s_cmp_eq_u32 s41, s48\n\t" \
    "s_cselect_b32 s47, s42, 0\n\t" \
    "s_cmp_gt_u32 s47, s41\n\t" \
    "s_cselect_b32 s66, s46, 0\n\t" \
    "s_addc_u32 s41, s41, 0\n\t" \
    "buffer_load_dwordx4 v[80:83], v3, s[64:67], s60 offen sc0 sc1\n\t" \
    "buffer_load_dwordx4 v[84:87], v3, s[64:67], s61 offen sc0 sc1\n\t" \
    "buffer_load_dwordx4 v[88:91], v3, s[64:67], s62 offen sc0 sc1\n\t" \
    "buffer_load_dwordx4 v[92:95], v3, s[64:67], s63 offen sc0 sc1\n\t" \
    "s_add_u32 s48, s48, 1\n\t" \
    "Li2_1_%=:\n\t" \
    "v_readlane_b32 s60, v4, s41\n\t" \
    "v_readlane_b32 s61, v5, s41\n\t" \
    "v_readlane_b32 s62, v6, s41\n\t" \
    "v_readlane_b32 s63, v7, s41\n\t" \
    "s_cmp_eq_u32 s41, s48\n\t" \
    "s_cselect_b32 s47, s42, 0\n\t" \
    "s_cmp_gt_u32 s47, s41\n\t" \
    "s_cselect_b32 s66, s46, 0\n\t" \
    "s_addc_u32 s41, s41, 0\n\t" \
    "buffer_load_dwordx4 v[96:99], v3, s[64:67], s60 offen sc0 sc1\n\t" \
    "buffer_load_dwordx4 v[100:103], v3, s[64:67], s61 offen sc0 sc1\n\t" \
    "buffer_load_dwordx4 v[104:107], v3, s[64:67], s62 offen sc0 sc1\n\t" \
    "buffer_load_dwordx4 v[108:111], v3, s[64:67], s63 offen sc0 sc1\n\t" \
    "s_add_u32 s48, s48, 1\n\t" \
    "Li2_2_%=:\n\t" \
    "v_readlane_b32 s60, v4, s41\n\t" \
    "v_readlane_b32 s61, v5, s41\n\t" \
    "v_readlane_b32 s62, v6, s41\n\t" \
    "v_readlane_b32 s63, v7, s41\n\t" \
    "s_cmp_eq_u32 s41, s48\n\t" \
    "s_cselect_b32 s47, s42, 0\n\t" \
    "s_cmp_gt_u32 s47, s41\n\t" \
    "s_cselect_b32 s66, s46, 0\n\t" \
    "s_addc_u32 s41, s41, 0\n\t" \
    "buffer_load_dwordx4 v[112:115], v3, s[64:67], s60 offen sc0 sc1\n\t" \
    "buffer_load_dwordx4 v[116:119], v3, s[64:67], s61 offen sc0 sc1\n\t" \
    "buffer_load_dwordx4 v[120:123], v3, s[64:67], s62 offen sc0 sc1\n\t" \
    "buffer_load_dwordx4 v[124:127], v3, s[64:67], s63 offen sc0 sc1\n\t" \
    "s_add_u32 s48, s48, 1\n\t" \
    "Li2_3_%=:\n\t" \
    "v_readlane_b32 s60, v4, s41\n\t" \
    "v_readlane_b32 s61, v5, s41\n\t" \
    "v_readlane_b32 s62, v6, s41\n\t" \
    "v_readlane_b32 s63, v7, s41\n\t" \
    "s_cmp_eq_u32 s41, s48\n\t" \
    "s_cselect_b32 s47, s42, 0\n\t" \
    "s_cmp_gt_u32 s47, s41\n\t" \
    "s_cselect_b32 s66, s46, 0\n\t" \
    "s_addc_u32 s41, s41, 0\n\t" \
    "buffer_load_dwordx4 v[48:51], v3, s[64:67], s60 offen sc0 sc1\n\t" \
    "buffer_load_dwordx4 v[52:55], v3, s[64:67], s61 offen sc0 sc1\n\t" \
    "buffer_load_dwordx4 v[56:59], v3, s[64:67], s62 offen sc0 sc1\n\t" \
    "buffer_load_dwordx4 v[60:63], v3, s[64:67], s63 offen sc0 sc1\n\t" \
    "s_add_u32 s48, s48, 1\n\t" \
    "v_readlane_b32 s60, v4, s41\n\t" \
    "v_readlane_b32 s61, v5, s41\n\t" \
    "v_readlane_b32 s62, v6, s41\n\t" \
    "v_readlane_b32 s63, v7, s41\n\t" \
    "s_cmp_eq_u32 s57, 0\n\t" \
    "s_cbranch_scc1 Lbj2_%=\n\t" \
    "s_branch Lbk2_%=\n\t" \
    "Lwf7_%=:\n\t" \
    "ds_read_b128 v[12:15], v8\n\t" \
    "s_waitcnt lgkmcnt(0)\n\t" \
    "v_min_u32 v14, v14, v15\n\t" \
    "s_nop 0\n\t" \
    "v_readfirstlane_b32 s43, v14\n\t" \
    "s_cmp_gt_u32 s43, s56\n\t" \
    "s_cbranch_scc1 Lwb7_%=\n\t" \
    "s_sleep 1\n\t" \
    "s_branch Lwf7_%=\n\t" \
    "Lrp3_%=:\n\t" \
    "s_sub_u32 s48, s40, 4\n\t" \
    "s_mov_b32 s57, 1\n\t" \
    "s_cmp_eq_u32 s41, s48\n\t" \
    "s_cbranch_scc0 Lr13_%=\n\t" \
    "Lr03_%=:\n\t" \
    "ds_read_b128 v[12:15], v8\n\t" \
    "s_waitcnt lgkmcnt(0)\n\t" \
    "v_min_u32 v14, v14, v15\n\t" \
    "v_readfirstlane_b32 s42, v12\n\t" \
    "s_cmp_gt_u32 s42, s41\n\t" \
    "s_cbranch_scc1 Li3_0_%=\n\t" \
    "s_sleep 1\n\t" \
    "s_branch Lr03_%=\n\t" \
    "Lr13_%=:\n\t" \
    "s_cmp_gt_u32 s42, s41\n\t" \
    "s_cbranch_scc0 Lbk3_%=\n\t" \
    "s_waitcnt vmcnt(12)\n\t" \
    "s_mov_b32 s57, 0\n\t" \
    "s_add_u32 s48, s48, 1\n\t" \
    "s_cmp_eq_u32 s41, s48\n\t" \
    "s_cbranch_scc1 Li3_1_%=\n\t" \
    "s_add_u32 s48, s48, 1\n\t" \
    "s_cmp_eq_u32 s41, s48\n\t" \
    "s_cbranch_scc1 Li3_2_%=\n\t" \
    "s_add_u32 s48, s48, 1\n\t" \
    "s_branch Li3_3_%=\n\t" \
    "Li3_0_%=:\n\t" \
    "v_readlane_b32 s60, v4, s41\n\t" \
    "v_readlane_b32 s61, v5, s41\n\t" \
    "v_readlane_b32 s62, v6, s41\n\t" \
    "v_readlane_b32 s63, v7, s41\n\t" \
    "s_cmp_eq_u32 s41, s48\n\t" \
    "s_cselect_b32 s47, s42, 0\n\t" \
    "s_cmp_gt_u32 s47, s41\n\t" \
    "s_cselect_b32 s66, s46, 0\n\t" \
    "s_addc_u32 s41, s41, 0\n\t" \
    "buffer_load_dwordx4 v[96:99], v3, s[64:67], s60 offen sc0 sc1\n\t" \
    "buffer_load_dwordx4 v[100:103], v3, s[64:67], s61 offen sc0 sc1\n\t" \
    "buffer_load_dwordx4 v[104:107], v3, s[64:67], s62 offen sc0 sc1\n\t" \
    "buffer_load_dwordx4 v[108:111], v3, s[64:67], s63 offen sc0 sc1\n\t" \
    "s_add_u32 s48, s48, 1\n\t" \
    "Li3_1_%=:\n\t" \
    "v_readlane_b32 s60, v4, s41\n\t" \
    "v_readlane_b32 s61, v5, s41\n\t" \
    "v_readlane_b32 s62, v6, s41\n\t" \
    "v_readlane_b32 s63, v7, s41\n\t" \
    "s_cmp_eq_u32 s41, s48\n\t" \
    "s_cselect_b32 s47, s42, 0\n\t" \
    "s_cmp_gt_u32 s47, s41\n\t" \
    "s_cselect_b32 s66, s46, 0\n\t" \
    "s_addc_u32 s41, s41, 0\n\t" \
    "buffer_load_dwordx4 v[112:115], v3, s[64:67], s60 offen sc0 sc1\n\t" \
    "buffer_load_dwordx4 v[116:119], v3, s[64:67], s61 offen sc0 sc1\n\t" \
    "buffer_load_dwordx4 v[120:123], v3, s[64:67], s62 offen sc0 sc1\n\t" \
    "buffer_load_dwordx4 v[124:127], v3, s[64:67], s63 offen sc0 sc1\n\t" \
    "s_add_u32 s48, s48, 1\n\t" \
    "Li3_2_%=:\n\t" \
    "v_readlane_b32 s60, v4, s41\n\t" \
    "v_readlane_b32 s61, v5, s41\n\t" \
    "v_readlane_b32 s62, v6, s41\n\t" \
    "v_readlane_b32 s63, v7, s41\n\t" \
    "s_cmp_eq_u32 s41, s48\n\t" \
    "s_cselect_b32 s47, s42, 0\n\t" \
    "s_cmp_gt_u32 s47, s41\n\t" \
    "s_cselect_b32 s66, s46, 0\n\t" \
    "s_addc_u32 s41, s41, 0\n\t" \
    "buffer_load_dwordx4 v[48:51], v3, s[64:67], s60 offen sc0 sc1\n\t" \
    "buffer_load_dwordx4 v[52:55], v3, s[64:67], s61 offen sc0 sc1\n\t" \
    "buffer_load_dwordx4 v[56:59], v3, s[64:67], s62 offen sc0 sc1\n\t" \
    "buffer_load_dwordx4 v[60:63], v3, s[64:67], s63 offen sc0 sc1\n\t" \
    "s_add_u32 s48, s48, 1\n\t" \
    "Li3_3_%=:\n\t" \
    "v_readlane_b32 s60, v4, s41\n\t" \
    "v_readlane_b32 s61, v5, s41\n\t" \
    "v_readlane_b32 s62, v6, s41\n\t" \
    "v_readlane_b32 s63, v7, s41\n\t" \
    "s_cmp_eq_u32 s41, s48\n\t" \
    "s_cselect_b32 s47, s42, 0\n\t" \
    "s_cmp_gt_u32 s47, s41\n\t" \
    "s_cselect_b32 s66, s46, 0\n\t" \
    "s_addc_u32 s41, s41, 0\n\t" \
    "buffer_load_dwordx4 v[64:67], v3, s[64:67], s60 offen sc0 sc1\n\t" \
    "buffer_load_dwordx4 v[68:71], v3, s[64:67], s61 offen sc0 sc1\n\t" \
    "buffer_load_dwordx4 v[72:75], v3, s[64:67], s62 offen sc0 sc1\n\t" \
    "buffer_load_dwordx4 v[76:79], v3, s[64:67], s63 offen sc0 sc1\n\t" \
    "s_add_u32 s48, s48, 1\n\t" \
    "v_readlane_b32 s60, v4, s41\n\t" \
    "v_readlane_b32 s61, v5, s41\n\t" \
    "v_readlane_b32 s62, v6, s41\n\t" \
    "v_readlane_b32 s63, v7, s41\n\t" \
    "s_cmp_eq_u32 s57, 0\n\t" \
    "s_cbranch_scc1 Lbj3_%=\n\t" \
    "s_branch Lbk3_%=\n\t" \
    "Lwf8_%=:\n\t" \
    "ds_read_b128 v[12:15], v8\n\t" \
    "s_waitcnt lgkmcnt(0)\n\t" \
    "v_min_u32 v14, v14, v15\n\t" \
    "s_nop 0\n\t" \
    "v_readfirstlane_b32 s43, v14\n\t" \
    "s_cmp_gt_u32 s43, s56\n\t" \
    "s_cbranch_scc1 Lwb8_%=\n\t" \
    "s_sleep 1\n\t" \
    "s_branch Lwf8_%=\n\t" \
    "Lrp4_%=:\n\t" \
    "s_sub_u32 s48, s40, 4\n\t" \
    "s_mov_b32 s57, 1\n\t" \
    "s_cmp_eq_u32 s41, s48\n\t" \
    "s_cbranch_scc0 Lr14_%=\n\t" \
    "Lr04_%=:\n\t" \
    "ds_read_b128 v[12:15], v8\n\t" \
    "s_waitcnt lgkmcnt(0)\n\t" \
    "v_min_u32 v14, v14, v15\n\t" \
    "v_readfirstlane_b32 s42, v12\n\t" \
    "s_cmp_gt_u32 s42, s41\n\t" \
    "s_cbranch_scc1 Li4_0_%=\n\t" \
    "s_sleep 1\n\t" \
    "s_branch Lr04_%=\n\t" \
    "Lr14_%=:\n\t" \
    "s_cmp_gt_u32 s42, s41\n\t" \
    "s_cbranch_scc0 Lbk4_%=\n\t" \
    "s_waitcnt vmcnt(12)\n\t" \
    "s_mov_b32 s57, 0\n\t" \
    "s_add_u32 s48, s48, 1\n\t" \
    "s_cmp_eq_u32 s41, s48\n\t" \
    "s_cbranch_scc1 Li4_1_%=\n\t" \
    "s_add_u32 s48, s48, 1\n\t" \
    "s_cmp_eq_u32 s41, s48\n\t" \
    "s_cbranch_scc1 Li4_2_%=\n\t" \
    "s_add_u32 s48, s48, 1\n\t" \
    "s_branch Li4_3_%=\n\t" \
    "Li4_0_%=:\n\t" \
    "v_readlane_b32 s60, v4, s41\n\t" \
    "v_readlane_b32 s61, v5, s41\n\t" \
    "v_readlane_b32 s62, v6, s41\n\t" \
    "v_readlane_b32 s63, v7, s41\n\t" \
    "s_cmp_eq_u32 s41, s48\n\t" \
    "s_cselect_b32 s47, s42, 0\n\t" \
    "s_cmp_gt_u32 s47, s41\n\t" \
    "s_cselect_b32 s66, s46, 0\n\t" \
    "s_addc_u32 s41, s41, 0\n\t" \
    "buffer_load_dwordx4 v[112:115], v3, s[64:67], s60 offen sc0 sc1\n\t" \
    "buffer_load_dwordx4 v[116:119], v3, s[64:67], s61 offen sc0 sc1\n\t" \
    "buffer_load_dwordx4 v[120:123], v3, s[64:67], s62 offen sc0 sc1\n\t" \
    "buffer_load_dwordx4 v[124:127], v3, s[64:67], s63 offen sc0 sc1\n\t" \
    "s_add_u32 s48, s48, 1\n\t" \
    "Li4_1_%=:\n\t" \
    "v_readlane_b32 s60, v4, s41\n\t" \
    "v_readlane_b32 s61, v5, s41\n\t" \
    "v_readlane_b32 s62, v6, s41\n\t" \
    "v_readlane_b32 s63, v7, s41\n\t" \
    "s_cmp_eq_u32 s41, s48\n\t" \
    "s_cselect_b32 s47, s42, 0\n\t" \
    "s_cmp_gt_u32 s47, s41\n\t" \
    "s_cselect_b32 s66, s46, 0\n\t" \
    "s_addc_u32 s41, s41, 0\n\t" \
    "buffer_load_dwordx4 v[48:51], v3, s[64:67], s60 offen sc0 sc1\n\t" \
    "buffer_load_dwordx4 v[52:55], v3, s[64:67], s61 offen sc0 sc1\n\t" \
    "buffer_load_dwordx4 v[56:59], v3, s[64:67], s62 offen sc0 sc1\n\t" \
    "buffer_load_dwordx4 v[60:63], v3, s[64:67], s63 offen sc0 sc1\n\t" \
    "s_add_u32 s48, s48, 1\n\t" \
    "Li4_2_%=:\n\t" \
    "v_readlane_b32 s60, v4, s41\n\t" \
    "v_readlane_b32 s61, v5, s41\n\t" \
    "v_readlane_b32 s62, v6, s41\n\t" \
    "v_readlane_b32 s63, v7, s41\n\t" \
    "s_cmp_eq_u32 s41, s48\n\t" \
    "s_cselect_b32 s47, s42, 0\n\t" \
    "s_cmp_gt_u32 s47, s41\n\t" \
    "s_cselect_b32 s66, s46, 0\n\t" \
    "s_addc_u32 s41, s41, 0\n\t" \
    "buffer_load_dwordx4 v[64:67], v3, s[64:67], s60 offen sc0 sc1\n\t" \
    "buffer_load_dwordx4 v[68:71], v3, s[64:67], s61 offen sc0 sc1\n\t" \
    "buffer_load_dwordx4 v[72:75], v3, s[64:67], s62 offen sc0 sc1\n\t" \
    "buffer_load_dwordx4 v[76:79], v3, s[64:67], s63 offen sc0 sc1\n\t" \
    "s_add_u32 s48, s48, 1\n\t" \
    "Li4_3_%=:\n\t" \
    "v_readlane_b32 s60, v4, s41\n\t" \
    "v_readlane_b32 s61, v5, s41\n\t" \
    "v_readlane_b32 s62, v6, s41\n\t" \
    "v_readlane_b32 s63, v7, s41\n\t" \
    "s_cmp_eq_u32 s41, s48\n\t" \
    "s_cselect_b32 s47, s42, 0\n\t" \
    "s_cmp_gt_u32 s47, s41\n\t" \
    "s_cselect_b32 s66, s46, 0\n\t" \
    "s_addc_u32 s41, s41, 0\n\t" \
    "buffer_load_dwordx4 v[80:83], v3, s[64:67], s60 offen sc0 sc1\n\t" \
    "buffer_load_dwordx4 v[84:87], v3, s[64:67], s61 offen sc0 sc1\n\t" \
    "buffer_load_dwordx4 v[88:91], v3, s[64:67], s62 offen sc0 sc1\n\t" \
    "buffer_load_dwordx4 v[92:95], v3, s[64:67], s63 offen sc0 sc1\n\t" \
    "s_add_u32 s48, s48, 1\n\t" \
    "v_readlane_b32 s60, v4, s41\n\t" \
    "v_readlane_b32 s61, v5, s41\n\t" \
    "v_readlane_b32 s62, v6, s41\n\t" \
    "v_readlane_b32 s63, v7, s41\n\t" \
    "s_cmp_eq_u32 s57, 0\n\t" \
    "s_cbranch_scc1 Lbj4_%=\n\t" \
    "s_branch Lbk4_%=\n\t" \
    "Lwf9_%=:\n\t" \
    "ds_read_b128 v[12:15], v8\n\t" \
    "s_waitcnt lgkmcnt(0)\n\t" \
    "v_min_u32 v14, v14, v15\n\t" \
    "s_nop 0\n\t" \
    "v_readfirstlane_b32 s43, v14\n\t" \
    "s_cmp_gt_u32 s43, s56\n\t" \
    "s_cbranch_scc1 Lwb9_%=\n\t" \
    "s_sleep 1\n\t" \
    "s_branch Lwf9_%=\n\t" \
    "Lrp0_%=:\n\t" \
    "s_sub_u32 s48, s40, 4\n\t" \
    "s_mov_b32 s57, 1\n\t" \
    "s_cmp_eq_u32 s41, s48\n\t" \
    "s_cbranch_scc0 Lr10_%=\n\t" \
    "Lr00_%=:\n\t" \
    "ds_read_b128 v[12:15], v8\n\t" \
    "s_waitcnt lgkmcnt(0)\n\t" \
    "v_min_u32 v14, v14, v15\n\t" \
    "v_readfirstlane_b32 s42, v12\n\t" \
    "s_cmp_gt_u32 s42, s41\n\t" \
    "s_cbranch_scc1 Li0_0_%=\n\t" \
    "s_sleep 1\n\t" \
    "s_branch Lr00_%=\n\t" \
    "Lr10_%=:\n\t" \
    "s_cmp_gt_u32 s42, s41\n\t" \
    "s_cbranch_scc0 Lbk0_%=\n\t" \
    "s_waitcnt vmcnt(12)\n\t" \
    "s_mov_b32 s57, 0\n\t" \
    "s_add_u32 s48, s48, 1\n\t" \
    "s_cmp_eq_u32 s41, s48\n\t" \
    "s_cbranch_scc1 Li0_1_%=\n\t" \
    "s_add_u32 s48, s48, 1\n\t" \
    "s_cmp_eq_u32 s41, s48\n\t" \
    "s_cbranch_scc1 Li0_2_%=\n\t" \
    "s_add_u32 s48, s48, 1\n\t" \
    "s_branch Li0_3_%=\n\t" \
    "Li0_0_%=:\n\t" \
    "v_readlane_b32 s60, v4, s41\n\t" \
    "v_readlane_b32 s61, v5, s41\n\t" \
    "v_readlane_b32 s62, v6, s41\n\t" \
    "v_readlane_b32 s63, v7, s41\n\t" \
    "s_cmp_eq_u32 s41, s48\n\t" \
    "s_cselect_b32 s47, s42, 0\n\t" \
    "s_cmp_gt_u32 s47, s41\n\t" \
    "s_cselect_b32 s66, s46, 0\n\t" \
    "s_addc_u32 s41, s41, 0\n\t" \
    "buffer_load_dwordx4 v[48:51], v3, s[64:67], s60 offen sc0 sc1\n\t" \
    "buffer_load_dwordx4 v[52:55], v3, s[64:67], s61 offen sc0 sc1\n\t" \
    "buffer_load_dwordx4 v[56:59], v3, s[64:67], s62 offen sc0 sc1\n\t" \
    "buffer_load_dwordx4 v[60:63], v3, s[64:67], s63 offen sc0 sc1\n\t" \
    "s_add_u32 s48, s48, 1\n\t" \
    "Li0_1_%=:\n\t" \
    "v_readlane_b32 s60, v4, s41\n\t" \
    "v_readlane_b32 s61, v5, s41\n\t" \
    "v_readlane_b32 s62, v6, s41\n\t" \
    "v_readlane_b32 s63, v7, s41\n\t" \
    "s_cmp_eq_u32 s41, s48\n\t" \
    "s_cselect_b32 s47, s42, 0\n\t" \
    "s_cmp_gt_u32 s47, s41\n\t" \
    "s_cselect_b32 s66, s46, 0\n\t" \
    "s_addc_u32 s41, s41, 0\n\t" \
    "buffer_load_dwordx4 v[64:67], v3, s[64:67], s60 offen sc0 sc1\n\t" \
    "buffer_load_dwordx4 v[68:71], v3, s[64:67], s61 offen sc0 sc1\n\t" \
    "buffer_load_dwordx4 v[72:75], v3, s[64:67], s62 offen sc0 sc1\n\t" \
    "buffer_load_dwordx4 v[76:79], v3, s[64:67], s63 offen sc0 sc1\n\t" \
    "s_add_u32 s48, s48, 1\n\t" \
    "Li0_2_%=:\n\t" \
    "v_readlane_b32 s60, v4, s41\n\t" \
    "v_readlane_b32 s61, v5, s41\n\t" \
    "v_readlane_b32 s62, v6, s41\n\t" \
    "v_readlane_b32 s63, v7, s41\n\t" \
    "s_cmp_eq_u32 s41, s48\n\t" \
    "s_cselect_b32 s47, s42, 0\n\t" \
    "s_cmp_gt_u32 s47, s41\n\t" \
    "s_cselect_b32 s66, s46, 0\n\t" \
    "s_addc_u32 s41, s41, 0\n\t" \
    "buffer_load_dwordx4 v[80:83], v3, s[64:67], s60 offen sc0 sc1\n\t" \
    "buffer_load_dwordx4 v[84:87], v3, s[64:67], s61 offen sc0 sc1\n\t" \
    "buffer_load_dwordx4 v[88:91], v3, s[64:67], s62 offen sc0 sc1\n\t" \
    "buffer_load_dwordx4 v[92:95], v3, s[64:67], s63 offen sc0 sc1\n\t" \
    "s_add_u32 s48, s48, 1\n\t" \
    "Li0_3_%=:\n\t" \
    "v_readlane_b32 s60, v4, s41\n\t" \
    "v_readlane_b32 s61, v5, s41\n\t" \
    "v_readlane_b32 s62, v6, s41\n\t" \
    "v_readlane_b32 s63, v7, s41\n\t" \
    "s_cmp_eq_u32 s41, s48\n\t" \
    "s_cselect_b32 s47, s42, 0\n\t" \
    "s_cmp_gt_u32 s47, s41\n\t" \
    "s_cselect_b32 s66, s46, 0\n\t" \
    "s_addc_u32 s41, s41, 0\n\t" \
    "buffer_load_dwordx4 v[96:99], v3, s[64:67], s60 offen sc0 sc1\n\t" \
    "buffer_load_dwordx4 v[100:103], v3, s[64:67], s61 offen sc0 sc1\n\t" \
    "buffer_load_dwordx4 v[104:107], v3, s[64:67], s62 offen sc0 sc1\n\t" \
    "buffer_load_dwordx4 v[108:111], v3, s[64:67], s63 offen sc0 sc1\n\t" \
    "s_add_u32 s48, s48, 1\n\t" \
    "v_readlane_b32 s60, v4, s41\n\t" \
    "v_readlane_b32 s61, v5, s41\n\t" \
    "v_readlane_b32 s62, v6, s41\n\t" \
    "v_readlane_b32 s63, v7, s41\n\t" \
    "s_cmp_eq_u32 s57, 0\n\t" \
    "s_cbranch_scc1 Lbj0_%=\n\t" \
    "s_branch Lbk0_%=\n\t" \
    "Lwf10_%=:\n\t" \
    "ds_read_b128 v[12:15], v8\n\t" \
    "s_waitcnt lgkmcnt(0)\n\t" \
    "v_min_u32 v14, v14, v15\n\t" \
    "s_nop 0\n\t" \
    "v_readfirstlane_b32 s43, v14\n\t" \
    "s_cmp_gt_u32 s43, 28\n\t" \
    "s_cbranch_scc1 Lwb10_%=\n\t" \
    "s_sleep 1\n\t" \
    "s_branch Lwf10_%=\n\t" \
    "LfB1_%=:\n\t" \
    "s_waitcnt vmcnt(0)\n\t" \
    "s_mov_b32 s48, s41\n\t" \
    "s_cmp_gt_u32 s41, 27\n\t" \
    "s_cbranch_scc1 LgB1_%=\n\t" \
    "LhB1_%=:\n\t" \
    "ds_read_b128 v[12:15], v8\n\t" \
    "s_waitcnt lgkmcnt(0)\n\t" \
    "v_min_u32 v14, v14, v15\n\t" \
    "v_readfirstlane_b32 s42, v12\n\t" \
    "s_cmp_gt_u32 s42, s41\n\t" \
    "s_cbranch_scc1 LgB1_%=\n\t" \
    "s_sleep 1\n\t" \
    "s_branch LhB1_%=\n\t" \
    "LgB1_%=:\n\t" \
    "s_cmp_eq_u32 s41, 27\n\t" \
    "s_cbranch_scc0 LjB1_27_%=\n\t" \
    "s_cmp_gt_u32 s42, s41\n\t" \
    "s_cbranch_scc0 LkB1_%=\n\t" \
    "v_readlane_b32 s60, v4, s41\n\t" \
    "v_readlane_b32 s61, v5, s41\n\t" \
    "v_readlane_b32 s62, v6, s41\n\t" \
    "v_readlane_b32 s63, v7, s41\n\t" \
    "s_mov_b32 s66, s46\n\t" \
    "s_add_u32 s41, s41, 1\n\t" \
    "s_mov_b32 s55, 1\n\t" \
    "s_nop 1\n\t" \
    "buffer_load_dwordx4 v[80:83], v3, s[64:67], s60 offen sc0 sc1\n\t" \
    "buffer_load_dwordx4 v[84:87], v3, s[64:67], s61 offen sc0 sc1\n\t" \
    "buffer_load_dwordx4 v[88:91], v3, s[64:67], s62 offen sc0 sc1\n\t" \
    "buffer_load_dwordx4 v[92:95], v3, s[64:67], s63 offen sc0 sc1\n\t" \
    "LjB1_27_%=:\n\t" \
    "s_cmp_eq_u32 s41, 28\n\t" \
    "s_cbranch_scc0 LjB1_28_%=\n\t" \
    "s_cmp_gt_u32 s42, s41\n\t" \
    "s_cbranch_scc0 LkB1_%=\n\t" \
    "v_readlane_b32 s60, v4, s41\n\t" \
    "v_readlane_b32 s61, v5, s41\n\t" \
    "v_readlane_b32 s62, v6, s41\n\t" \
    "v_readlane_b32 s63, v7, s41\n\t" \
    "s_mov_b32 s66, s46\n\t" \
    "s_add_u32 s41, s41, 1\n\t" \
    "s_mov_b32 s55, 1\n\t" \
    "s_nop 1\n\t" \
    "buffer_load_dwordx4 v[96:99], v3, s[64:67], s60 offen sc0 sc1\n\t" \
    "buffer_load_dwordx4 v[100:103], v3, s[64:67], s61 offen sc0 sc1\n\t" \
    "buffer_load_dwordx4 v[104:107], v3, s[64:67], s62 offen sc0 sc1\n\t" \
    "buffer_load_dwordx4 v[108:111], v3, s[64:67], s63 offen sc0 sc1\n\t" \
    "LjB1_28_%=:\n\t" \
    "s_cmp_eq_u32 s41, 29\n\t" \
    "s_cbranch_scc0 LjB1_29_%=\n\t" \
    "s_cmp_gt_u32 s42, s41\n\t" \
    "s_cbranch_scc0 LkB1_%=\n\t" \
    "v_readlane_b32 s60, v4, s41\n\t" \
    "v_readlane_b32 s61, v5, s41\n\t" \
    "v_readlane_b32 s62, v6, s41\n\t" \
    "v_readlane_b32 s63, v7, s41\n\t" \
    "s_mov_b32 s66, s46\n\t" \
    "s_add_u32 s41, s41, 1\n\t" \
    "s_mov_b32 s55, 1\n\t" \
    "s_nop 1\n\t" \
    "buffer_load_dwordx4 v[112:115], v3, s[64:67], s60 offen sc0 sc1\n\t" \
    "buffer_load_dwordx4 v[116:119], v3, s[64:67], s61 offen sc0 sc1\n\t" \
    "buffer_load_dwordx4 v[120:123], v3, s[64:67], s62 offen sc0 sc1\n\t" \
    "buffer_load_dwordx4 v[124:127], v3, s[64:67], s63 offen sc0 sc1\n\t" \
    "LjB1_29_%=:\n\t" \
    "s_cmp_eq_u32 s41, 30\n\t" \
    "s_cbranch_scc0 LjB1_30_%=\n\t" \
    "s_cmp_gt_u32 s42, s41\n\t" \
    "s_cbranch_scc0 LkB1_%=\n\t" \
    "v_readlane_b32 s60, v4, s41\n\t" \
    "v_readlane_b32 s61, v5, s41\n\t" \
    "v_readlane_b32 s62, v6, s41\n\t" \
    "v_readlane_b32 s63, v7, s41\n\t" \
    "s_mov_b32 s66, s46\n\t" \
    "s_add_u32 s41, s41, 1\n\t" \
    "s_mov_b32 s55, 1\n\t" \
    "s_nop 1\n\t" \
    "buffer_load_dwordx4 v[48:51], v3, s[64:67], s60 offen sc0 sc1\n\t" \
    "buffer_load_dwordx4 v[52:55], v3, s[64:67], s61 offen sc0 sc1\n\t" \
    "buffer_load_dwordx4 v[56:59], v3, s[64:67], s62 offen sc0 sc1\n\t" \
    "buffer_load_dwordx4 v[60:63], v3, s[64:67], s63 offen sc0 sc1\n\t" \
    "LjB1_30_%=:\n\t" \
    "LkB1_%=:\n\t" \
    "v_readlane_b32 s60, v4, s41\n\t" \
    "v_readlane_b32 s61, v5, s41\n\t" \
    "v_readlane_b32 s62, v6, s41\n\t" \
    "v_readlane_b32 s63, v7, s41\n\t" \
    "s_cmp_gt_u32 s48, 27\n\t" \
    "s_cbranch_scc1 LmB1_%=\n\t" \
    "s_waitcnt vmcnt(0)\n\t" \
    "s_branch LmB1_%=\n\t" \
    "Lwf11_%=:\n\t" \
    "ds_read_b128 v[12:15], v8\n\t" \
    "s_waitcnt lgkmcnt(0)\n\t" \
    "v_min_u32 v14, v14, v15\n\t" \
    "s_nop 0\n\t" \
    "v_readfirstlane_b32 s43, v14\n\t" \
    "s_cmp_gt_u32 s43, 30\n\t" \
    "s_cbranch_scc1 Lwb11_%=\n\t" \
    "s_sleep 1\n\t" \
    "s_branch Lwf11_%=\n\t" \
    "LfB2_%=:\n\t" \
    "s_waitcnt vmcnt(0)\n\t" \
    "s_mov_b32 s48, s41\n\t" \
    "s_cmp_gt_u32 s41, 28\n\t" \
    "s_cbranch_scc1 LgB2_%=\n\t" \
    "LhB2_%=:\n\t" \
    "ds_read_b128 v[12:15], v8\n\t" \
    "s_waitcnt lgkmcnt(0)\n\t" \
    "v_min_u32 v14, v14, v15\n\t" \
    "v_readfirstlane_b32 s42, v12\n\t" \
    "s_cmp_gt_u32 s42, s41\n\t" \
    "s_cbranch_scc1 LgB2_%=\n\t" \
    "s_sleep 1\n\t" \
    "s_branch LhB2_%=\n\t" \
    "LgB2_%=:\n\t" \
    "s_cmp_eq_u32 s41, 28\n\t" \
    "s_cbranch_scc0 LjB2_28_%=\n\t" \
    "s_cmp_gt_u32 s42, s41\n\t" \
    "s_cbranch_scc0 LkB2_%=\n\t" \
    "v_readlane_b32 s60, v4, s41\n\t" \
    "v_readlane_b32 s61, v5, s41\n\t" \
    "v_readlane_b32 s62, v6, s41\n\t" \
    "v_readlane_b32 s63, v7, s41\n\t" \
    "s_mov_b32 s66, s46\n\t" \
    "s_add_u32 s41, s41, 1\n\t" \
    "s_mov_b32 s55, 1\n\t" \
    "s_nop 1\n\t" \
    "buffer_load_dwordx4 v[96:99], v3, s[64:67], s60 offen sc0 sc1\n\t" \
    "buffer_load_dwordx4 v[100:103], v3, s[64:67], s61 offen sc0 sc1\n\t" \
    "buffer_load_dwordx4 v[104:107], v3, s[64:67], s62 offen sc0 sc1\n\t" \
    "buffer_load_dwordx4 v[108:111], v3, s[64:67], s63 offen sc0 sc1\n\t" \
    "LjB2_28_%=:\n\t" \
    "s_cmp_eq_u32 s41, 29\n\t" \
    "s_cbranch_scc0 LjB2_29_%=\n\t" \
    "s_cmp_gt_u32 s42, s41\n\t" \
    "s_cbranch_scc0 LkB2_%=\n\t" \
    "v_readlane_b32 s60, v4, s41\n\t" \
    "v_readlane_b32 s61, v5, s41\n\t" \
    "v_readlane_b32 s62, v6, s41\n\t" \
    "v_readlane_b32 s63, v7, s41\n\t" \
    "s_mov_b32 s66, s46\n\t" \
    "s_add_u32 s41, s41, 1\n\t" \
    "s_mov_b32 s55, 1\n\t" \
    "s_nop 1\n\t" \
    "buffer_load_dwordx4 v[112:115], v3, s[64:67], s60 offen sc0 sc1\n\t" \
    "buffer_load_dwordx4 v[116:119], v3, s[64:67], s61 offen sc0 sc1\n\t" \
    "buffer_load_dwordx4 v[120:123], v3, s[64:67], s62 offen sc0 sc1\n\t" \
    "buffer_load_dwordx4 v[124:127], v3, s[64:67], s63 offen sc0 sc1\n\t" \
    "LjB2_29_%=:\n\t" \
    "s_cmp_eq_u32 s41, 30\n\t" \
    "s_cbranch_scc0 LjB2_30_%=\n\t" \
    "s_cmp_gt_u32 s42, s41\n\t" \
    "s_cbranch_scc0 LkB2_%=\n\t" \
    "v_readlane_b32 s60, v4, s41\n\t" \
    "v_readlane_b32 s61, v5, s41\n\t" \
    "v_readlane_b32 s62, v6, s41\n\t" \
    "v_readlane_b32 s63, v7, s41\n\t" \
    "s_mov_b32 s66, s46\n\t" \
    "s_add_u32 s41, s41, 1\n\t" \
    "s_mov_b32 s55, 1\n\t" \
    "s_nop 1\n\t" \
    "buffer_load_dwordx4 v[48:51], v3, s[64:67], s60 offen sc0 sc1\n\t" \
    "buffer_load_dwordx4 v[52:55], v3, s[64:67], s61 offen sc0 sc1\n\t" \
    "buffer_load_dwordx4 v[56:59], v3, s[64:67], s62 offen sc0 sc1\n\t" \
    "buffer_load_dwordx4 v[60:63], v3, s[64:67], s63 offen sc0 sc1\n\t" \
    "LjB2_30_%=:\n\t" \
    "s_cmp_eq_u32 s41, 31\n\t" \
    "s_cbranch_scc0 LjB2_31_%=\n\t" \
    "s_cmp_gt_u32 s42, s41\n\t" \
    "s_cbranch_scc0 LkB2_%=\n\t" \
    "v_readlane_b32 s60, v4, s41\n\t" \
    "v_readlane_b32 s61, v5, s41\n\t" \
    "v_readlane_b32 s62, v6, s41\n\t" \
    "v_readlane_b32 s63, v7, s41\n\t" \
    "s_mov_b32 s66, s46\n\t" \
    "s_add_u32 s41, s41, 1\n\t" \
    "s_mov_b32 s55, 1\n\t" \
    "s_nop 1\n\t" \
    "buffer_load_dwordx4 v[64:67], v3, s[64:67], s60 offen sc0 sc1\n\t" \
    "buffer_load_dwordx4 v[68:71], v3, s[64:67], s61 offen sc0 sc1\n\t" \
    "buffer_load_dwordx4 v[72:75], v3, s[64:67], s62 offen sc0 sc1\n\t" \
    "buffer_load_dwordx4 v[76:79], v3, s[64:67], s63 offen sc0 sc1\n\t" \
    "LjB2_31_%=:\n\t" \
    "LkB2_%=:\n\t" \
    "v_readlane_b32 s60, v4, s41\n\t" \
    "v_readlane_b32 s61, v5, s41\n\t" \
    "v_readlane_b32 s62, v6, s41\n\t" \
    "v_readlane_b32 s63, v7, s41\n\t" \
    "s_cmp_gt_u32 s48, 28\n\t" \
    "s_cbranch_scc1 LmB2_%=\n\t" \
    "s_waitcnt vmcnt(0)\n\t" \
    "s_branch LmB2_%=\n\t" \
    "Lwf12_%=:\n\t" \
    "ds_read_b128 v[12:15], v8\n\t" \
    "s_waitcnt lgkmcnt(0)\n\t" \
    "v_min_u32 v14, v14, v15\n\t" \
    "s_nop 0\n\t" \
    "v_readfirstlane_b32 s43, v14\n\t" \
    "s_cmp_gt_u32 s43, 32\n\t" \
    "s_cbranch_scc1 Lwb12_%=\n\t" \
    "s_sleep 1\n\t" \
    "s_branch Lwf12_%=\n\t" \
    "LfB3_%=:\n\t" \
    "s_waitcnt vmcnt(0)\n\t" \
    "s_mov_b32 s48, s41\n\t" \
    "s_cmp_gt_u32 s41, 29\n\t" \
    "s_cbranch_scc1 LgB3_%=\n\t" \
    "LhB3_%=:\n\t" \
    "ds_read_b128 v[12:15], v8\n\t" \
    "s_waitcnt lgkmcnt(0)\n\t" \
    "v_min_u32 v14, v14, v15\n\t" \
    "v_readfirstlane_b32 s42, v12\n\t" \
    "s_cmp_gt_u32 s42, s41\n\t" \
    "s_cbranch_scc1 LgB3_%=\n\t" \
    "s_sleep 1\n\t" \
    "s_branch LhB3_%=\n\t" \
    "LgB3_%=:\n\t" \
    "s_cmp_eq_u32 s41, 29\n\t" \
    "s_cbranch_scc0 LjB3_29_%=\n\t" \
    "s_cmp_gt_u32 s42, s41\n\t" \
    "s_cbranch_scc0 LkB3_%=\n\t" \
    "v_readlane_b32 s60, v4, s41\n\t" \
    "v_readlane_b32 s61, v5, s41\n\t" \
    "v_readlane_b32 s62, v6, s41\n\t" \
    "v_readlane_b32 s63, v7, s41\n\t" \
    "s_mov_b32 s66, s46\n\t" \
    "s_add_u32 s41, s41, 1\n\t" \
    "s_mov_b32 s55, 1\n\t" \
    "s_nop 1\n\t" \
    "buffer_load_dwordx4 v[112:115], v3, s[64:67], s60 offen sc0 sc1\n\t" \
    "buffer_load_dwordx4 v[116:119], v3, s[64:67], s61 offen sc0 sc1\n\t" \
    "buffer_load_dwordx4 v[120:123], v3, s[64:67], s62 offen sc0 sc1\n\t" \
    "buffer_load_dwordx4 v[124:127], v3, s[64:67], s63 offen sc0 sc1\n\t" \
    "LjB3_29_%=:\n\t" \
    "s_cmp_eq_u32 s41, 30\n\t" \
    "s_cbranch_scc0 LjB3_30_%=\n\t" \
    "s_cmp_gt_u32 s42, s41\n\t" \
    "s_cbranch_scc0 LkB3_%=\n\t" \
    "v_readlane_b32 s60, v4, s41\n\t" \
    "v_readlane_b32 s61, v5, s41\n\t" \
    "v_readlane_b32 s62, v6, s41\n\t" \
    "v_readlane_b32 s63, v7, s41\n\t" \
    "s_mov_b32 s66, s46\n\t" \
    "s_add_u32 s41, s41, 1\n\t" \
    "s_mov_b32 s55, 1\n\t" \
    "s_nop 1\n\t" \
    "buffer_load_dwordx4 v[48:51], v3, s[64:67], s60 offen sc0 sc1\n\t" \
    "buffer_load_dwordx4 v[52:55], v3, s[64:67], s61 offen sc0 sc1\n\t" \
    "buffer_load_dwordx4 v[56:59], v3, s[64:67], s62 offen sc0 sc1\n\t" \
    "buffer_load_dwordx4 v[60:63], v3, s[64:67], s63 offen sc0 sc1\n\t" \
    "LjB3_30_%=:\n\t" \
    "s_cmp_eq_u32 s41, 31\n\t" \
    "s_cbranch_scc0 LjB3_31_%=\n\t" \
    "s_cmp_gt_u32 s42, s41\n\t" \
    "s_cbranch_scc0 LkB3_%=\n\t" \
    "v_readlane_b32 s60, v4, s41\n\t" \
    "v_readlane_b32 s61, v5, s41\n\t" \
    "v_readlane_b32 s62, v6, s41\n\t" \
    "v_readlane_b32 s63, v7, s41\n\t" \
    "s_mov_b32 s66, s46\n\t" \
    "s_add_u32 s41, s41, 1\n\t" \
    "s_mov_b32 s55, 1\n\t" \
    "s_nop 1\n\t" \
    "buffer_load_dwordx4 v[64:67], v3, s[64:67], s60 offen sc0 sc1\n\t" \
    "buffer_load_dwordx4 v[68:71], v3, s[64:67], s61 offen sc0 sc1\n\t" \
    "buffer_load_dwordx4 v[72:75], v3, s[64:67], s62 offen sc0 sc1\n\t" \
    "buffer_load_dwordx4 v[76:79], v3, s[64:67], s63 offen sc0 sc1\n\t" \
    "LjB3_31_%=:\n\t" \
    "LkB3_%=:\n\t" \
    "v_readlane_b32 s60, v4, s41\n\t" \
    "v_readlane_b32 s61, v5, s41\n\t" \
    "v_readlane_b32 s62, v6, s41\n\t" \
    "v_readlane_b32 s63, v7, s41\n\t" \
    "s_cmp_gt_u32 s48, 29\n\t" \
    "s_cbranch_scc1 LmB3_%=\n\t" \
    "s_waitcnt vmcnt(0)\n\t" \
    "s_branch LmB3_%=\n\t" \
    "Lwf13_%=:\n\t" \
    "ds_read_b128 v[12:15], v8\n\t" \
    "s_waitcnt lgkmcnt(0)\n\t" \
    "v_min_u32 v14, v14, v15\n\t" \
    "s_nop 0\n\t" \
    "v_readfirstlane_b32 s43, v14\n\t" \
    "s_cmp_gt_u32 s43, 36\n\t" \
    "s_cbranch_scc1 Lwb13_%=\n\t" \
    "s_sleep 1\n\t" \
    "s_branch Lwf13_%=\n\t" \
    "LfB4_%=:\n\t" \
    "s_waitcnt vmcnt(0)\n\t" \
    "s_mov_b32 s48, s41\n\t" \
    "s_cmp_gt_u32 s41, 30\n\t" \
    "s_cbranch_scc1 LgB4_%=\n\t" \
    "LhB4_%=:\n\t" \
    "ds_read_b128 v[12:15], v8\n\t" \
    "s_waitcnt lgkmcnt(0)\n\t" \
    "v_min_u32 v14, v14, v15\n\t" \
    "v_readfirstlane_b32 s42, v12\n\t" \
    "s_cmp_gt_u32 s42, s41\n\t" \
    "s_cbranch_scc1 LgB4_%=\n\t" \
    "s_sleep 1\n\t" \
    "s_branch LhB4_%=\n\t" \
    "LgB4_%=:\n\t" \
    "s_cmp_eq_u32 s41, 30\n\t" \
    "s_cbranch_scc0 LjB4_30_%=\n\t" \
    "s_cmp_gt_u32 s42, s41\n\t" \
    "s_cbranch_scc0 LkB4_%=\n\t" \
    "v_readlane_b32 s60, v4, s41\n\t" \
    "v_readlane_b32 s61, v5, s41\n\t" \
    "v_readlane_b32 s62, v6, s41\n\t" \
    "v_readlane_b32 s63, v7, s41\n\t" \
    "s_mov_b32 s66, s46\n\t" \
    "s_add_u32 s41, s41, 1\n\t" \
    "s_mov_b32 s55, 1\n\t" \
    "s_nop 1\n\t" \
    "buffer_load_dwordx4 v[48:51], v3, s[64:67], s60 offen sc0 sc1\n\t" \
    "buffer_load_dwordx4 v[52:55], v3, s[64:67], s61 offen sc0 sc1\n\t" \
    "buffer_load_dwordx4 v[56:59], v3, s[64:67], s62 offen sc0 sc1\n\t" \
    "buffer_load_dwordx4 v[60:63], v3, s[64:67], s63 offen sc0 sc1\n\t" \
    "LjB4_30_%=:\n\t" \
    "s_cmp_eq_u32 s41, 31\n\t" \
    "s_cbranch_scc0 LjB4_31_%=\n\t" \
    "s_cmp_gt_u32 s42, s41\n\t" \
    "s_cbranch_scc0 LkB4_%=\n\t" \
    "v_readlane_b32 s60, v4, s41\n\t" \
    "v_readlane_b32 s61, v5, s41\n\t" \
    "v_readlane_b32 s62, v6, s41\n\t" \
    "v_readlane_b32 s63, v7, s41\n\t" \
    "s_mov_b32 s66, s46\n\t" \
    "s_add_u32 s41, s41, 1\n\t" \
    "s_mov_b32 s55, 1\n\t" \
    "s_nop 1\n\t" \
    "buffer_load_dwordx4 v[64:67], v3, s[64:67], s60 offen sc0 sc1\n\t" \
    "buffer_load_dwordx4 v[68:71], v3, s[64:67], s61 offen sc0 sc1\n\t" \
    "buffer_load_dwordx4 v[72:75], v3, s[64:67], s62 offen sc0 sc1\n\t" \
    "buffer_load_dwordx4 v[76:79], v3, s[64:67], s63 offen sc0 sc1\n\t" \
    "LjB4_31_%=:\n\t" \
    "LkB4_%=:\n\t" \
    "v_readlane_b32 s60, v4, s41\n\t" \
    "v_readlane_b32 s61, v5, s41\n\t" \
    "v_readlane_b32 s62, v6, s41\n\t" \
    "v_readlane_b32 s63, v7, s41\n\t" \
    "s_cmp_gt_u32 s48, 30\n\t" \
    "s_cbranch_scc1 LmB4_%=\n\t" \
    "s_waitcnt vmcnt(0)\n\t" \
    "s_branch LmB4_%=\n\t" \
    "Lwf15_%=:\n\t" \
    "ds_read_b128 v[12:15], v8\n\t" \
    "s_waitcnt lgkmcnt(0)\n\t" \
    "v_min_u32 v14, v14, v15\n\t" \
    "s_nop 0\n\t" \
    "v_readfirstlane_b32 s43, v14\n\t" \
    "s_cmp_gt_u32 s43, 40\n\t" \
    "s_cbranch_scc1 Lwb15_%=\n\t" \
    "s_sleep 1\n\t" \
    "s_branch Lwf15_%=\n\t" \
    "LfB5_%=:\n\t" \
    "s_waitcnt vmcnt(0)\n\t" \
    "s_mov_b32 s48, s41\n\t" \
    "s_cmp_gt_u32 s41, 31\n\t" \
    "s_cbranch_scc1 LgB5_%=\n\t" \
    "LhB5_%=:\n\t" \
    "ds_read_b128 v[12:15], v8\n\t" \
    "s_waitcnt lgkmcnt(0)\n\t" \
    "v_min_u32 v14, v14, v15\n\t" \
    "v_readfirstlane_b32 s42, v12\n\t" \
    "s_cmp_gt_u32 s42, s41\n\t" \
    "s_cbranch_scc1 LgB5_%=\n\t" \
    "s_sleep 1\n\t" \
    "s_branch LhB5_%=\n\t" \
    "LgB5_%=:\n\t" \
    "s_cmp_eq_u32 s41, 31\n\t" \
    "s_cbranch_scc0 LjB5_31_%=\n\t" \
    "s_cmp_gt_u32 s42, s41\n\t" \
    "s_cbranch_scc0 LkB5_%=\n\t" \
    "v_readlane_b32 s60, v4, s41\n\t" \
    "v_readlane_b32 s61, v5, s41\n\t" \
    "v_readlane_b32 s62, v6, s41\n\t" \
    "v_readlane_b32 s63, v7, s41\n\t" \
    "s_mov_b32 s66, s46\n\t" \
    "s_add_u32 s41, s41, 1\n\t" \
    "s_mov_b32 s55, 1\n\t" \
    "s_nop 1\n\t" \
    "buffer_load_dwordx4 v[64:67], v3, s[64:67], s60 offen sc0 sc1\n\t" \
    "buffer_load_dwordx4 v[68:71], v3, s[64:67], s61 offen sc0 sc1\n\t" \
    "buffer_load_dwordx4 v[72:75], v3, s[64:67], s62 offen sc0 sc1\n\t" \
    "buffer_load_dwordx4 v[76:79], v3, s[64:67], s63 offen sc0 sc1\n\t" \
    "LjB5_31_%=:\n\t" \
    "LkB5_%=:\n\t" \
    "v_readlane_b32 s60, v4, s41\n\t" \
    "v_readlane_b32 s61, v5, s41\n\t" \
    "v_readlane_b32 s62, v6, s41\n\t" \
    "v_readlane_b32 s63, v7, s41\n\t" \
    "s_cmp_gt_u32 s48, 31\n\t" \
    "s_cbranch_scc1 LmB5_%=\n\t" \
    "s_waitcnt vmcnt(0)\n\t" \
    "s_branch LmB5_%=\n\t" \
    "Lwf18_%=:\n\t" \
    "ds_read_b128 v[12:15], v8\n\t" \
    "s_waitcnt lgkmcnt(0)\n\t" \
    "v_min_u32 v14, v14, v15\n\t" \
    "s_nop 0\n\t" \
    "v_readfirstlane_b32 s43, v14\n\t" \
    "s_cmp_gt_u32 s43, 43\n\t" \
    "s_cbranch_scc1 Lwb18_%=\n\t" \
    "s_sleep 1\n\t" \
    "s_branch Lwf18_%=\n\t" \
    "LEND_%=:\n\t"

}

__global__ __launch_bounds__(512, 2) void a2a_gemm_kernel(Params P) {
    extern __shared__ __attribute__((aligned(16))) char shm[];
    const int wid = rfl(threadIdx.x >> 6);
#define LANE_ID(v) int v; asm volatile("v_mbcnt_lo_u32_b32 %0, -1, 0\n\tv_mbcnt_hi_u32_b32 %0, -1, %0" : "=v"(v))
    const int b = blockIdx.x;
    const int xcd = b & 7, idx = b >> 3;
    int rank = P.rank;
    char* my = P.ws[0];
    char* pw = P.ws[xcd];
    const char* px = P.x;
    const char* pwt = P.w;
    char* pout = P.out;
    asm volatile("" : "+s"(rank), "+s"(my), "+s"(pw), "+s"(px), "+s"(pwt), "+s"(pout));
    const int kc = xcd & 3, nt = (xcd >> 2) * 16 + (idx & 15), mt = idx >> 4;
#define PARTNER(r) (b ^ (r))
    AS1 unsigned* const cntp = (AS1 unsigned*)(my + WS_CNT) + b;
    AS1 unsigned* const tmop = (AS1 unsigned*)(my + WS_TMO);

    const unsigned long long t0 = __builtin_amdgcn_s_memrealtime();
    const int slq = ((3 - (int)((unsigned long long)pwt >> 7)) & 7) >> 1, i3 = idx & 3;
    const int nrm = (idx >> 2) * 3 + i3 - (i3 > slq);
    const int slot = i3 == slq ? SLOW0 + (idx >> 2) : (nrm < SLOW0 ? nrm : nrm + 8);
    const unsigned tq = xcd ? POFF + (slot >> 1) * PACE + (slot & 1) * (PACE / 2) : 0x7fffffffu;
    if (wid == 0) {
        LANE_ID(lane);
        __builtin_amdgcn_s_setprio(3);
        const unsigned p = (unsigned)(rank + xcd) & 7u;
        const int q = slot >> 1, G = slot & 1;
        const int kb = (q + KROT * rank) & 15;
        const char* src = px + (size_t)(p * 512 + G * 256) * 1024 + kb * 64;
        const unsigned so = (lane >> 2) * 1024 + (lane & 3) * 16;
        char* dst = pw + WS_STG + (size_t)(rank * NQ + kb) * BLK + G * (BLK / 2);
        char* fl = pw + WS_FLG + (G * 16 + q) * 2048 + (7 - xcd) * 4;
        if (tq > PLEAD) {
            __builtin_amdgcn_s_barrier();
            if (xcd == 0) return;
            while ((unsigned)(__builtin_amdgcn_s_memrealtime() - t0) < min(tq - PLEAD, (unsigned)PLOAD)) __builtin_amdgcn_s_sleep(8);
        }
        unsigned sov = so;
        u32x32 va, vb;
        asm volatile(
            "global_load_dwordx4 v[64:67], %[so], %[sr]\n\t"
            "v_add_u32 %[so], 0x4000, %[so]\n\t"
            "global_load_dwordx4 v[68:71], %[so], %[sr]\n\t"
            "v_add_u32 %[so], 0x4000, %[so]\n\t"
            "global_load_dwordx4 v[72:75], %[so], %[sr]\n\t"
            "v_add_u32 %[so], 0x4000, %[so]\n\t"
            "global_load_dwordx4 v[76:79], %[so], %[sr]\n\t"
            "v_add_u32 %[so], 0x4000, %[so]\n\t"
            "global_load_dwordx4 v[80:83], %[so], %[sr]\n\t"
            "v_add_u32 %[so], 0x4000, %[so]\n\t"
            "global_load_dwordx4 v[84:87], %[so], %[sr]\n\t"
            "v_add_u32 %[so], 0x4000, %[so]\n\t"
            "global_load_dwordx4 v[88:91], %[so], %[sr]\n\t"
            "v_add_u32 %[so], 0x4000, %[so]\n\t"
            "global_load_dwordx4 v[92:95], %[so], %[sr]\n\t"
            "v_add_u32 %[so], 0x4000, %[so]\n\t"
            "global_load_dwordx4 v[96:99], %[so], %[sr]\n\t"
            "v_add_u32 %[so], 0x4000, %[so]\n\t"
            "global_load_dwordx4 v[100:103], %[so], %[sr]\n\t"
            "v_add_u32 %[so], 0x4000, %[so]\n\t"
            "global_load_dwordx4 v[104:107], %[so], %[sr]\n\t"
            "v_add_u32 %[so], 0x4000, %[so]\n\t"
            "global_load_dwordx4 v[108:111], %[so], %[sr]\n\t"
            "v_add_u32 %[so], 0x4000, %[so]\n\t"
            "global_load_dwordx4 v[112:115], %[so], %[sr]\n\t"
            "v_add_u32 %[so], 0x4000, %[so]\n\t"
            "global_load_dwordx4 v[116:119], %[so], %[sr]\n\t"
            "v_add_u32 %[so], 0x4000, %[so]\n\t"
            "global_load_dwordx4 v[120:123], %[so], %[sr]\n\t"
            "v_add_u32 %[so], 0x4000, %[so]\n\t"
            "global_load_dwordx4 v[124:127], %[so], %[sr]\n\t"
            "s_cmp_gt_u32 %[eb], %[pl]\n\t"
            "s_cbranch_scc1 Lpb_%=\n\t"
            "s_barrier\n\t"
            "Lpb_%=:\n\t"
            "s_waitcnt vmcnt(15)\n\t"
            "ds_bpermute_b32 v64, %[pa], v64\n\t"
            "ds_bpermute_b32 v65, %[pa], v65\n\t"
            "ds_bpermute_b32 v66, %[pa], v66\n\t"
            "ds_bpermute_b32 v67, %[pa], v67\n\t"
            "s_waitcnt vmcnt(14)\n\t"
            "ds_bpermute_b32 v68, %[pa], v68\n\t"
            "ds_bpermute_b32 v69, %[pa], v69\n\t"
            "ds_bpermute_b32 v70, %[pa], v70\n\t"
            "ds_bpermute_b32 v71, %[pa], v71\n\t"
            "s_waitcnt vmcnt(13)\n\t"
            "ds_bpermute_b32 v72, %[pa], v72\n\t"
            "ds_bpermute_b32 v73, %[pa], v73\n\t"
            "ds_bpermute_b32 v74, %[pa], v74\n\t"
            "ds_bpermute_b32 v75, %[pa], v75\n\t"
            "s_waitcnt vmcnt(12)\n\t"
            "ds_bpermute_b32 v76, %[pa], v76\n\t"
            "ds_bpermute_b32 v77, %[pa], v77\n\t"
            "ds_bpermute_b32 v78, %[pa], v78\n\t"
            "ds_bpermute_b32 v79, %[pa], v79\n\t"
            "s_waitcnt vmcnt(11)\n\t"
            "ds_bpermute_b32 v80, %[pa], v80\n\t"
            "ds_bpermute_b32 v81, %[pa], v81\n\t"
            "ds_bpermute_b32 v82, %[pa], v82\n\t"
            "ds_bpermute_b32 v83, %[pa], v83\n\t"
            "s_waitcnt vmcnt(10)\n\t"
            "ds_bpermute_b32 v84, %[pa], v84\n\t"
            "ds_bpermute_b32 v85, %[pa], v85\n\t"
            "ds_bpermute_b32 v86, %[pa], v86\n\t"
            "ds_bpermute_b32 v87, %[pa], v87\n\t"
            "s_waitcnt vmcnt(9)\n\t"
            "ds_bpermute_b32 v88, %[pa], v88\n\t"
            "ds_bpermute_b32 v89, %[pa], v89\n\t"
            "ds_bpermute_b32 v90, %[pa], v90\n\t"
            "ds_bpermute_b32 v91, %[pa], v91\n\t"
            "s_waitcnt vmcnt(8)\n\t"
            "ds_bpermute_b32 v92, %[pa], v92\n\t"
            "ds_bpermute_b32 v93, %[pa], v93\n\t"
            "ds_bpermute_b32 v94, %[pa], v94\n\t"
            "ds_bpermute_b32 v95, %[pa], v95\n\t"
            "s_waitcnt vmcnt(7)\n\t"
            "ds_bpermute_b32 v96, %[pa], v96\n\t"
            "ds_bpermute_b32 v97, %[pa], v97\n\t"
            "ds_bpermute_b32 v98, %[pa], v98\n\t"
            "ds_bpermute_b32 v99, %[pa], v99\n\t"
            "s_waitcnt vmcnt(6)\n\t"
            "ds_bpermute_b32 v100, %[pa], v100\n\t"
            "ds_bpermute_b32 v101, %[pa], v101\n\t"
            "ds_bpermute_b32 v102, %[pa], v102\n\t"
            "ds_bpermute_b32 v103, %[pa], v103\n\t"
            "s_waitcnt vmcnt(5)\n\t"
            "ds_bpermute_b32 v104, %[pa], v104\n\t"
            "ds_bpermute_b32 v105, %[pa], v105\n\t"
            "ds_bpermute_b32 v106, %[pa], v106\n\t"
            "ds_bpermute_b32 v107, %[pa], v107\n\t"
            "s_waitcnt vmcnt(4)\n\t"
            "ds_bpermute_b32 v108, %[pa], v108\n\t"
            "ds_bpermute_b32 v109, %[pa], v109\n\t"
            "ds_bpermute_b32 v110, %[pa], v110\n\t"
            "ds_bpermute_b32 v111, %[pa], v111\n\t"
            "s_waitcnt vmcnt(3)\n\t"
            "ds_bpermute_b32 v112, %[pa], v112\n\t"
            "ds_bpermute_b32 v113, %[pa], v113\n\t"
            "ds_bpermute_b32 v114, %[pa], v114\n\t"
            "ds_bpermute_b32 v115, %[pa], v115\n\t"
            "s_waitcnt vmcnt(2)\n\t"
            "ds_bpermute_b32 v116, %[pa], v116\n\t"
            "ds_bpermute_b32 v117, %[pa], v117\n\t"
            "ds_bpermute_b32 v118, %[pa], v118\n\t"
            "ds_bpermute_b32 v119, %[pa], v119\n\t"
            "s_waitcnt vmcnt(1)\n\t"
            "ds_bpermute_b32 v120, %[pa], v120\n\t"
            "ds_bpermute_b32 v121, %[pa], v121\n\t"
            "ds_bpermute_b32 v122, %[pa], v122\n\t"
            "ds_bpermute_b32 v123, %[pa], v123\n\t"
            "s_waitcnt vmcnt(0)\n\t"
            "ds_bpermute_b32 v124, %[pa], v124\n\t"
            "ds_bpermute_b32 v125, %[pa], v125\n\t"
            "ds_bpermute_b32 v126, %[pa], v126\n\t"
            "ds_bpermute_b32 v127, %[pa], v127\n\t"
            "s_waitcnt lgkmcnt(0)\n\t"
            : "={v[64:95]}"(va), "={v[96:127]}"(vb), [so] "+{v2}"(sov)
            : [sr] "s"(src), [eb] "s"(tq), [pl] "i"(PLEAD),
              [pa] "{v3}"(((lane & 15) * 4 + (lane >> 4)) * 4)
            : "memory", "scc");
        STAMP(4);
        while ((unsigned)(__builtin_amdgcn_s_memrealtime() - t0) + PHOLD < tq) __builtin_amdgcn_s_sleep(4);
        lds_st(L_HLD, 1);
        while ((unsigned)(__builtin_amdgcn_s_memrealtime() - t0) < tq) __builtin_amdgcn_s_sleep(1);
        STAMP(28);
        unsigned sto = lane_off(lane);
        asm volatile(
            "global_store_dwordx4 %[o], v[64:67], %[ds] sc0 sc1\n\t"
            "v_add_u32 %[o], %[fr], %[o]\n\t"
            "global_store_dwordx4 %[o], v[68:71], %[ds] sc0 sc1\n\t"
            "v_add_u32 %[o], %[fr], %[o]\n\t"
            "global_store_dwordx4 %[o], v[72:75], %[ds] sc0 sc1\n\t"
            "v_add_u32 %[o], %[fr], %[o]\n\t"
            "global_store_dwordx4 %[o], v[76:79], %[ds] sc0 sc1\n\t"
            "v_add_u32 %[o], %[fr], %[o]\n\t"
            "global_store_dwordx4 %[o], v[80:83], %[ds] sc0 sc1\n\t"
            "v_add_u32 %[o], %[fr], %[o]\n\t"
            "global_store_dwordx4 %[o], v[84:87], %[ds] sc0 sc1\n\t"
            "v_add_u32 %[o], %[fr], %[o]\n\t"
            "global_store_dwordx4 %[o], v[88:91], %[ds] sc0 sc1\n\t"
            "v_add_u32 %[o], %[fr], %[o]\n\t"
            "global_store_dwordx4 %[o], v[92:95], %[ds] sc0 sc1\n\t"
            "v_add_u32 %[o], %[fr], %[o]\n\t"
            "global_store_dwordx4 %[o], v[96:99], %[ds] sc0 sc1\n\t"
            "v_add_u32 %[o], %[fr], %[o]\n\t"
            "global_store_dwordx4 %[o], v[100:103], %[ds] sc0 sc1\n\t"
            "v_add_u32 %[o], %[fr], %[o]\n\t"
            "global_store_dwordx4 %[o], v[104:107], %[ds] sc0 sc1\n\t"
            "v_add_u32 %[o], %[fr], %[o]\n\t"
            "global_store_dwordx4 %[o], v[108:111], %[ds] sc0 sc1\n\t"
            "v_add_u32 %[o], %[fr], %[o]\n\t"
            "global_store_dwordx4 %[o], v[112:115], %[ds] sc0 sc1\n\t"
            "v_add_u32 %[o], %[fr], %[o]\n\t"
            "global_store_dwordx4 %[o], v[116:119], %[ds] sc0 sc1\n\t"
            "v_add_u32 %[o], %[fr], %[o]\n\t"
            "global_store_dwordx4 %[o], v[120:123], %[ds] sc0 sc1\n\t"
            "v_add_u32 %[o], %[fr], %[o]\n\t"
            "global_store_dwordx4 %[o], v[124:127], %[ds] sc0 sc1\n\t"
            "s_nop 1\n\t"
            : [o] "+v"(sto) : "{v[64:95]}"(va), "{v[96:127]}"(vb), [ds] "s"(dst), [fr] "i"(FRAG) : "memory");
        if (lane == 0) lds_st(L_PST, 1);
        STAMP(29);
        unsigned call;
        while ((call = rfl(lds_ld1(L_CALL))) == 0) {}
        wait_vm<0>();
        if (lane < FREP) st_sys32(fl + lane * 64, call);
        if (lane == 0) lds_st(L_PST, 2);
        STAMP(2);
        return;
    }
    {
        if (wid == 1) {
            LANE_ID(lane);
            if (lane < 36) lds_st(L_SYNC + lane * 4, 0);
            asm volatile("s_waitcnt lgkmcnt(0)" ::: "memory");
        }
        __builtin_amdgcn_s_barrier();
    }
    if (wid == 1) {
        LANE_ID(lane);
        const unsigned call = rfl(__hip_atomic_load(cntp, __ATOMIC_RELAXED, __HIP_MEMORY_SCOPE_AGENT)) + 1;
        __builtin_amdgcn_s_setprio(MPRIO);
        if (lane == 0) lds_st(L_CALL, call);
        STAMP(1);
        const char* fp = my + WS_FLG + mt * 32768 + (b % FREP) * 64;
        const char* xp = my + WS_XFL + b * 128;
        unsigned avi = 0, ava = 4, xrdy = 0, spins = 0;
        bool dead = false;
        auto publish = [&](unsigned a, unsigned i) { lds_st(L_SYNC, a); };
        auto expired = [&]() {
            return (++spins & 255u) == 0 &&
                   (__hip_atomic_load(tmop, __ATOMIC_RELAXED, __HIP_MEMORY_SCOPE_AGENT) == call ||
                    __builtin_amdgcn_s_memrealtime() - t0 > DEADLINE);
        };
        unsigned rounds = avi / 7;
        while ((unsigned)(__builtin_amdgcn_s_memrealtime() - t0) < MSTART) __builtin_amdgcn_s_sleep(2);
        while (rounds < (unsigned)NQ) {
            u32x8 g0, g1;
            unsigned fr;
            asm volatile("s_load_dwordx8 %0, %3, %4 glc\n\ts_load_dwordx8 %1, %3, %5 glc\n\tds_read_b32 %2, %6\n\ts_waitcnt lgkmcnt(0)"
                         : "=&s"(g0), "=&s"(g1), "=&v"(fr) : "s"(fp), "s"(rounds * 2048), "s"(min(rounds + 1, 15u) * 2048), "v"(L_FREE) : "memory");
            unsigned mk = 0;
#pragma unroll
            for (int j = 0; j < 7; ++j) mk |= (g0[j] >= call ? 1u << j : 0u) | (g1[j] >= call ? 128u << j : 0u);
            const unsigned base = rounds * 7;
            const unsigned other = base + 14 <= (unsigned)IA && FINE ? ~(0x1111u << ((kc - base) & 3)) : 0u;
            const unsigned ni = min(base + __builtin_ctz(~((mk | other) & 0x3fffu)), 112u);
            if (ni > avi) {
                avi = ni;
                rounds = avi / 7;
                ava = avi < (unsigned)IA + 4 ? min(4u + ((avi + 3 - kc) >> 2), (unsigned)NA) : NA + (avi - IA) / 4;
                publish(ava, avi);
                STAMP(8 + rounds);
            } else if (expired()) {
                dead = true;
                break;
            }
            if (ava >= rfl(fr) + MLOOK) __builtin_amdgcn_s_sleep(MSLEEP);
        }
        while (!dead && rfl(lds_ld1(L_FREE)) <= (unsigned)NA) __builtin_amdgcn_s_sleep(2);
        while (!dead && xrdy != 255) {
            u32x16 g;
            asm volatile("s_load_dwordx16 %0, %1, 0x0 glc\n\ts_waitcnt lgkmcnt(0)" : "=&s"(g) : "s"(xp) : "memory");
            unsigned nx = 0;
#pragma unroll
            for (int w = 0; w < 4; ++w) {
                const unsigned e = min(g[4 + w], g[8 + w]);
                nx |= (e >= call ? 1u << w : 0u) | ((N2 ? g[w] : min(g[w], e)) >= call ? 16u << w : 0u);
            }
            if (nx != xrdy) {
                xrdy = nx;
                if (lane == 0) lds_st2(L_XRDY, xrdy & 15u, xrdy >> 4);
            } else {
                if (expired()) { dead = true; break; }
            }
        }
        STAMP(26);
        if (dead) {
            if (lane == 0) {
                __hip_atomic_store(tmop, call, __ATOMIC_RELAXED, __HIP_MEMORY_SCOPE_AGENT);
                lds_st(L_TMO, 1);
                lds_st2(L_XRDY, 15, 15);
            }
            publish(NA + (112 - IA) / 4, 112);
        }
        if (b == 0) {
            char* const dp = P.ws[lane >> 3] + WS_AM2 + (lane & 7) * 1024 + rank * 8;
            const char* gp = my + WS_AMX + lane * 16;
            unsigned m;
            while (true) {
                u32x4 g0, g1;
                asm volatile("global_load_dwordx4 %0, %2, off sc0 sc1\n\t"
                             "global_load_dwordx4 %1, %2, off offset:1024 sc0 sc1\n\t"
                             "s_waitcnt vmcnt(0)"
                             : "=&v"(g0), "=&v"(g1) : "v"(gp) : "memory");
                m = max(max(g0.x, g0.z), max(g1.x, g1.z));
                if (__all(min(min(g0.y, g0.w), min(g1.y, g1.w)) >= call)) break;
                if (expired()) {
                    if (lane == 0) __hip_atomic_store(tmop, call, __ATOMIC_RELAXED, __HIP_MEMORY_SCOPE_AGENT);
                    m = 0x7fffffffu;
                    break;
                }
            }
            st_gran(dp, wave_max_u32(m), call);
        }
        return;
    }
    if (wid < 4) {
        LANE_ID(lane);
        const int lw = wid - 2;
        const auto wr = __builtin_amdgcn_make_buffer_rsrc((void*)pwt, 0, W_BYTES, 0x00020000);
        const int r0 = lane >> 4;
        const unsigned voff = (16 * (r0 >> 1) + 4 * (r0 & 1)) * 8192 + nt * 256 + (lane & 15) * 16;
#define ROWOFF(i) (8 * ((i) >> 2) + ((i) & 3))
        bool pushed = xcd == 0;
        __builtin_amdgcn_s_setprio(LPRIO);
        unsigned voff1 = voff + 8192, voff2 = voff + 16384, voff3 = voff + 24576;
        asm volatile("" : "+v"(voff1), "+v"(voff2), "+v"(voff3));
        auto item = [&](unsigned so, unsigned lb) {
            __builtin_amdgcn_raw_ptr_buffer_load_lds(wr, (AS3 void*)(lb), 16, voff, so, 0, 0);
            __builtin_amdgcn_raw_ptr_buffer_load_lds(wr, (AS3 void*)(lb + 1056), 16, voff1, so, 0, 0);
            __builtin_amdgcn_raw_ptr_buffer_load_lds(wr, (AS3 void*)(lb + 2112), 16, voff2, so, 0, 0);
            __builtin_amdgcn_raw_ptr_buffer_load_lds(wr, (AS3 void*)(lb + 3168), 16, voff3, so, 0, 0);
        };
        if (tq < GHOLD) {
            while (rfl(lds_ld1(L_PST)) < (unsigned)GREL) __builtin_amdgcn_s_sleep(1);
            pushed = true;
        }
        if (lw == 0) STAMP(384);
        {
            const unsigned so = (rank * 16 + 4 * kc + lw) * 262144u, lb = lw * SLOT;
            item(so, lb); item(so + 65536, lb + 4224); item(so + 524288, lb + 2 * SLOT); item(so + 589824, lb + 2 * SLOT + 4224);
        }
        const unsigned tab = step_block(lane, kc, rank);
        const unsigned sh0 = mt * 4, stag = (nt & 3) * LSTAG, fa = L_FULL + lw * 4;
        unsigned c = 4, junk = 0, room = XL_SLOT, pub = 0;
        unsigned hist = 0x00020204u;
        unsigned tabw = tab * 262144u, tabl = (lane % NS) * SLOT;
        asm volatile("" : "+v"(tabw), "+v"(tabl));
        const u32x4 wsrd = {(unsigned)(unsigned long long)pwt, (unsigned)((unsigned long long)pwt >> 32) & 0xffffu, W_BYTES, 0x00020000u};
        auto progress = [&]() {
            const u32x4 f = lds_ld4(L_FREE);
            return rfl(min(min(f.x, f.y), min(f.z, f.w)));
        };
        auto drain = [&]() {
            wait_vm<8>();
            lds_st(fa, max((hist >> 16) & 255u, pub));
            wait_vm<0>();
            pub = hist & 255u;
            lds_st(fa, pub);
            hist = (hist & 255u) * 0x01010101u;
        };
        auto issued = [&](unsigned val, bool cold) {
            unsigned v;
            if (cold) { wait_vm<4 * LIF>(); v = (hist >> (8 * LIF - 8)) & 255u; }
            else { wait_vm<16>(); v = hist >> 24; }
            if (v > pub) { pub = v; lds_st(fa, v); }
            hist = hist << 8 | val;
        };
        constexpr unsigned long long SB = LSB, ST = LST;
        for (unsigned g = 0; c < (unsigned)NST; ++g) {
            const unsigned B = min((unsigned)(SB >> (4 * g)) & 15u, NST - c);
            unsigned trig = ((ST >> (4 * g)) & 15u) * 2;
            trig += trig ? stag : 0;
            if (trig && progress() < trig) {
                drain();
                while (progress() < trig) __builtin_amdgcn_s_sleep(LSLEEP);
            }
            if (lw == 0) STAMP(384 + c);
            if (g == LTOUCH) {
                const unsigned sh8 = mt * 4 + kc, tr = (lane >> 1) & 3;
                const unsigned tv = (16 * (tr >> 1) + 4 * (tr & 1) + ROWOFF(sh8)) * 8192 + nt * 256 + (lane & 1) * 128;
#pragma unroll
                for (int i = 0; i < (NST - NB0 + 15) / 16; ++i) {
                    const unsigned a = tv + __shfl(tabw, min(NB0 + (2 * i + lw) * 8 + (lane >> 3), NST - 1));
                    asm volatile("buffer_load_dword %0, %1, %2, 0 offen" : "+v"(junk) : "v"(a), "s"(wsrd) : "memory");
                }
            }
            const unsigned first = c + ((c ^ lw) & 1);
            c += B;
#pragma unroll
            for (int h = 0; h < 2; ++h) {
                const unsigned ow = (sh0 ^ (h * 4)) * 16384u, ol = (sh0 ^ (h * 4)) * 1056u;
                for (unsigned st = first; st < c; st += 2) {
                    if (h == 0 && st >= room) {
                        unsigned fr = progress();
                        if (!fr || fr + NS - 1 <= st) {
                            drain();
                            do { __builtin_amdgcn_s_sleep(LSLEEP); fr = progress(); } while (!fr || fr + NS - 1 <= st);
                        }
                        room = fr + NS - 1;
                    }
                    if (!pushed && rfl(lds_ld1(L_HLD))) {
                        drain();
                        while (rfl(lds_ld1(L_PST)) < (unsigned)PREL) __builtin_amdgcn_s_sleep(1);
                        pushed = true;
                    }
                    item(__builtin_amdgcn_readlane(tabw, st) + ow, __builtin_amdgcn_readlane(tabl, st) + ol);
                    issued(h ? st + 2 : hist & 255u, h == 0 && st < (unsigned)NB0);
                }
            }
            if (lw == 0) STAMP(320 + c - 1);
        }
        drain();
        asm volatile("" ::"v"(junk));
        return;
    }

    const int gw = wid - 4;
    __builtin_amdgcn_s_setprio(GPRIO);
    f32x32 q0;
    float mx;
    unsigned call;
    {
    LANE_ID(lane);
    if (GDELAY) { while ((unsigned)(__builtin_amdgcn_s_memrealtime() - t0) < GDELAY) {} }
    if (tq < GHOLD) while (rfl(lds_ld1(L_PST)) < (unsigned)GREL1) __builtin_amdgcn_s_sleep(1);
    const auto lr = __builtin_amdgcn_make_buffer_rsrc((void*)px, 0, 4096u * 1024u, 0x00020000);
    auto local_x = [&](int i) {
        const unsigned j = (i + (nt & 3) * 4 * XROT) & 15, r4 = lane >> 4;
        __builtin_amdgcn_raw_ptr_buffer_load_lds(lr, (AS3 void*)(L_XL + gw * 16384 + j * 1024), 16,
                                                 r4 * 1024 + (((lane & 15) ^ ((i & 3) * 4 + r4)) * 16),
                                                 (rank * 512 + mt * 256 + gw * 64 + j * 4) * 1024 + kc * 256, 0, 0);
    };
#pragma unroll
    for (int i = 0; i < 4; ++i) local_x(i);
    const int m16 = lane & 15, G = lane >> 4;
    const unsigned tab = step_block(lane, kc, rank);
    {
        const unsigned long long base = (unsigned long long)my;
        const u32x4 srd = {(unsigned)base, (unsigned)(base >> 32) & 0xffffu, (unsigned)WS_TMO, 0x00020000u};
        const unsigned wl = (((G & 1) * 4 + (m16 >> 2)) * 1056) + (G >> 1) * 512 + (lane & 3) * 8;
        const unsigned xv = (mt * 16 + gw * 4) * FRAG + lane_off(lane);
        unsigned tb[4];
#pragma unroll
        for (int r = 0; r < 4; ++r) {
            const int k = lane < NA ? lane : (lane < NA + N2 ? NA + 2 * (lane - NA) + (r >> 1) : NB0 + 4 * (lane - NA - N2) + r);
            const int fr = lane < NA ? r : (lane < NA + N2 ? r & 1 : 0);
            tb[r] = __builtin_amdgcn_ds_bpermute((k & 63) * 4, tab) * BLK + (fr ^ kc) * FRAG;
        }
        float mxv;
        unsigned cno = (unsigned)WS_CNT + b * 4u;
        const unsigned one = (G == (m16 >> 2)) ? (0x3f80u << ((m16 & 1) * 16)) : 0u;
        const u32x4 sel0 = {(m16 & 2) ? 0u : one, (m16 & 2) ? one : 0u, 0u, 0u};
        const u32x4 sel1 = {0u, 0u, sel0.x, sel0.y};
        const unsigned l3 = lane < 3 ? lane : 0;
        const unsigned flg = (unsigned)WS_XFL + PARTNER(l3 + 1) * 128 + (l3 * 4 + gw) * 4;
        unsigned xa0 = L_XL + gw * 16384 + (kc * 16 + m16) * 256 + ((G ^ m16) * 16);
        {
            unsigned p0 = wl, p1 = flg, p2 = sel0.x, p3 = sel0.y;
            asm volatile("" : "+v"(tb[0]), "+v"(tb[1]), "+v"(tb[2]), "+v"(tb[3]), "+v"(xa0) : "v"(p0), "v"(p1), "v"(p2), "v"(p3), "v"(xv) : "memory");
        }
        if (tq < GHOLD) while (rfl(lds_ld1(L_PST)) < (unsigned)GREL) __builtin_amdgcn_s_sleep(1);
#pragma unroll
        for (int i = 4; i < 16; ++i) local_x(i);
        asm volatile(GEMM_ASM
                     : "={v[128:159]}"(q0), "={s49}"(call), "+{v2}"(xa0), "={v26}"(mxv), "+{s57}"(cno)
                     : "{s[36:39]}"(srd),
                       "{s[68:69]}"(my + WS_XCH + ((PARTNER(1) * 3 + 0) * 4 + gw) * 4096u),
                       "{s[70:71]}"(my + WS_XCH + ((PARTNER(2) * 3 + 1) * 4 + gw) * 4096u),
                       "{s[72:73]}"(my + WS_XCH + ((PARTNER(3) * 3 + 2) * 4 + gw) * 4096u),
                       "{s53}"((unsigned)WS_XCH + (b * 12 + gw) * 4096u), "{s54}"(1u << gw),
                       "{v1}"(wl), "{v3}"(xv), "{v4}"(tb[0]), "{v5}"(tb[1]), "{v6}"(tb[2]), "{v7}"(tb[3]),
                       "{v8}"(L_SYNC), "{v9}"(L_FREE + lane * 16 + gw * 4), "{v11}"(lane * 16),
                       "{v[16:19]}"(sel0), "{v[20:23]}"(sel1), "{v24}"(flg), "{v27}"(L_DBG + 256 + gw * 256)
                     : "memory", "scc", "vcc", "v10", "v12", "v13", "v14", "v15", "v25", "v0", "v28", "v29", "v30", "v31",
                       "v32", "v33", "v34", "v35", "v36", "v37", "v38", "v39", "v40", "v41", "v42", "v43", "v44", "v45", "v46",
                       "v47", "v48", "v49", "v50", "v51", "v52", "v53", "v54", "v55", "v56", "v57", "v58", "v59", "v60", "v61",
                       "v62", "v63", "v64", "v65", "v66", "v67", "v68", "v69", "v70", "v71", "v72", "v73", "v74", "v75", "v76",
                       "v77", "v78", "v79", "v80", "v81", "v82", "v83", "v84", "v85", "v86", "v87", "v88", "v89", "v90", "v91",
                       "v92", "v93", "v94", "v95", "v96", "v97", "v98", "v99", "v100", "v101", "v102", "v103", "v104", "v105",
                       "v106", "v107", "v108", "v109", "v110", "v111", "v112", "v113", "v114", "v115", "v116", "v117", "v118",
                       "v119", "v120", "v121", "v122", "v123", "v124", "v125", "v126", "v127", "v160", "v161", "v162", "v163",
                       "v164", "v165", "v166", "v167", "v168", "v169", "v170", "v171", "v172", "v173", "v174", "v175", "v176",
                       "v177", "v178", "v179", "v180", "v181", "v182", "v183", "v184", "v185", "v186", "v187", "v188", "v189",
                       "v190", "v191", "v192", "v193", "v194", "v195", "v196", "v197", "v198", "v199", "v200", "v201", "v202",
                       "v203", "v204", "v205", "v206", "v207", "v208", "v209", "v210", "v211", "v212", "v213", "v214", "v215",
                       "v216", "v217", "v218", "v219", "v220", "v221", "v222", "v223", "v224", "v225", "v226", "v227", "v228",
                       "v229", "v230", "v231", "v232", "v233", "v234", "v235", "v236", "v237", "v238", "v239", "v240", "v241",
                       "v242", "v243", "v244", "v245", "v246", "v247", "v248", "v249", "v250", "v251", "v252", "v253", "v254",
                       "v255",
                       "s40", "s41", "s42", "s43", "s44", "s45", "s46", "s47", "s48", "s55", "s56", "s58", "s59",
                       "s60", "s61", "s62", "s63", "s64", "s65", "s66", "s67", "s74", "s75");
        mx = mxv;
    }
    }
    LANE_ID(lane);
    const int m16 = lane & 15, G = lane >> 4;
    if (gw == 0) STAMP(5);

    float rl[32];
    auto relu = [&]() {
#pragma unroll
        for (int i = 0; i < 32; ++i) asm volatile("v_max_f32 %0, 0, %1" : "=v"(rl[i]) : "v"(q0[i]));
    };
    unsigned mu = wave_max_u32(__builtin_bit_cast(unsigned, mx));
    {
        u32x4 m4;
        unsigned tm;
        asm volatile("ds_write_b32 %2, %3\n\tds_read_b128 %0, %4 offset:96\n\tds_read_b32 %1, %4 offset:48\n\ts_waitcnt lgkmcnt(0)"
                     : "=&v"(m4), "=&v"(tm) : "v"(L_MAX1 + gw * 4), "v"(mu | 0x80000000u), "v"(L_SYNC) : "memory");
        if (rfl(m4.x & m4.y & m4.z & m4.w) >> 31) {
            unsigned m = max(max(m4.x, m4.y), max(m4.z, m4.w)) & 0x7fffffffu;
            if (tm) m = 0x7fffffffu;
            if (lane == 0) st_gran(my + WS_AMX + b * 8, m, call);
            STAMP(27);
            relu();
            const char* gp = my + WS_AM2 + xcd * 1024 + (lane & 7) * 8;
            unsigned spins = 0;
            while (true) {
                u32x2 g;
                asm volatile("global_load_dwordx2 %0, %1, off sc0 sc1\n\ts_waitcnt vmcnt(0)" : "=&v"(g) : "v"(gp) : "memory");
                m = g.x;
                if (__all(g.y >= call)) break;
                if ((++spins & 255u) == 0 &&
                    (__hip_atomic_load(tmop, __ATOMIC_RELAXED, __HIP_MEMORY_SCOPE_AGENT) == call ||
                     __builtin_amdgcn_s_memrealtime() - t0 > DEADLINE)) {
                    if (lane == 0) __hip_atomic_store(tmop, call, __ATOMIC_RELAXED, __HIP_MEMORY_SCOPE_AGENT);
                    m = 0x7fffffffu;
                    break;
                }
            }
            mu = wave_max_u32(m);
            if (lane == 0) lds_st2(L_MAX2, mu, 1u);
        } else {
            relu();
            u32x2 r2;
            do { r2 = lds_ld2(L_MAX2); } while (rfl(r2.y) == 0);
            mu = r2.x;
        }
    }
    if (gw == 0) STAMP(6);

    const float amax = __builtin_bit_cast(float, mu);
    float inv = 448.0f * __builtin_amdgcn_rcpf(amax);
    if (rfl(mu) - 0x08000000u >= 0x70000000u) {
        const float scale = amax / 448.0f;
        inv = scale > 0.f ? 1.0f / scale : 0.f;
    }
    unsigned d[8];
#pragma unroll
    for (int i = 0; i < 8; ++i) {
        int w = __builtin_amdgcn_cvt_pk_fp8_f32(rl[4 * i] * inv, rl[4 * i + 1] * inv, 0, false);
        w = __builtin_amdgcn_cvt_pk_fp8_f32(rl[4 * i + 2] * inv, rl[4 * i + 3] * inv, w, true);
        d[i] = mu > 0x7f800000u ? 0x7f7f7f7fu : (unsigned)w;
    }
    char* op = pout + (size_t)(mt * 256 + gw * 64 + kc * 16 + m16) * 4096 + nt * 128 + G * 16;
#pragma unroll
    for (int h = 0; h < 2; ++h) {
        const auto s0 = __builtin_amdgcn_permlane32_swap(d[4 * h], d[4 * h + 2], false, false);
        const auto s1 = __builtin_amdgcn_permlane32_swap(d[4 * h + 1], d[4 * h + 3], false, false);
        const auto a = __builtin_amdgcn_permlane16_swap(s0[0], s1[0], false, false);
        const auto c = __builtin_amdgcn_permlane16_swap(s0[1], s1[1], false, false);
        const u32x4 o4 = {a[0], a[1], c[0], c[1]};
        asm volatile("global_store_dwordx4 %0, %1, off sc0 sc1\n\ts_nop 1" ::"v"(op + h * 64), "v"(o4) : "memory");
    }
    if (gw == 0 && lane == 0) __hip_atomic_store(cntp, call, __ATOMIC_RELAXED, __HIP_MEMORY_SCOPE_AGENT);
}

extern "C" void kernel_launch(int rank, int nranks, ncclComm_t comm, hipStream_t stream,
                              void* const* d_in, const int* in_sizes, int n_in,
                              void* d_out, int out_size,
                              const int* in_dtypes, int out_dtype,
                              void* const* d_out_all,
                              void* const* d_ws_all, size_t ws_bytes) {
    if (nranks != NR || (unsigned)rank >= (unsigned)NR || ws_bytes < WS_END || n_in < 2) {
        fprintf(stderr, "unexpected configuration\n");
        return;
    }
    HIP_CHECK(hipFuncSetAttribute((const void*)a2a_gemm_kernel, hipFuncAttributeMaxDynamicSharedMemorySize, LDS_BYTES));
    Params P;
    P.x = (const char*)d_in[0];
    P.w = (const char*)d_in[1];
    P.out = (char*)d_out;
    for (int i = 0; i < NR; ++i) P.ws[i] = (char*)d_ws_all[(rank + i) % NR];
    P.rank = rank;
    P.pad = 0;
    a2a_gemm_kernel<<<NWG, 512, LDS_BYTES, stream>>>(P);
    HIP_CHECK(hipGetLastError());
}
```
